```python
import numpy as np

kernel_name = "gpu_gemm_rs_m4096_k4096_n4096_w8a8_dequant"


ABSMAX_RTOL = 0.001

COMPUTE_DTYPE = "int8"
EPILOGUE = "dequant"
IN_QUANT = "int8"
OUT_QUANT = None
IN_DTYPES = {"x": "int8", "w_mat": "int8",
             "scale_x": "float32", "scale_w": "float32"}
WORLD_SIZE = 8

def setup_inputs():

    x = np.random.randint(-127, 128, size=(4096, 4096)).astype(np.int8)
    w_mat = np.random.randint(-127, 128, size=(4096, 4096)).astype(np.int8)
    scale_x = (0.02 * (1.0 + np.random.rand(1))).astype(np.float32)
    scale_w = (0.02 * (1.0 + np.random.rand(1))).astype(np.float32)
    return {"x": x, "w_mat": w_mat, "scale_x": scale_x, "scale_w": scale_w}

def reference(x, w_mat, scale_x, scale_w):
    acc = x.astype(np.float64) @ w_mat.astype(np.float64)
    y = (acc.astype(np.float32) * (scale_x * scale_w)).astype(np.float32)
    return y
```

```cpp
#include <hip/hip_runtime.h>
#include <rccl/rccl.h>
#include <cstdio>
#include <cstdint>

#define HIP_CHECK(expr) do { hipError_t _e = (expr); if (_e != hipSuccess) { \
    fprintf(stderr, "HIP error %s at %s:%d\n", hipGetErrorString(_e), __FILE__, __LINE__); \
    return; } } while (0)

namespace {

typedef int v4i __attribute__((ext_vector_type(4)));
typedef unsigned v4u __attribute__((ext_vector_type(4)));
typedef float v4f __attribute__((ext_vector_type(4)));

constexpr int NR = 8;
constexpr int NN = 4096;
constexpr int MS = 512;
constexpr int KS = 512;
constexpr unsigned UNIT = 4096;
constexpr unsigned SHARD = 2u << 20;
constexpr unsigned XBLK = 256u << 10;

constexpr unsigned OFF_WST = 0;
constexpr unsigned OFF_X = 4u << 20;
constexpr unsigned OFF_WG = 8u << 20;
constexpr unsigned OFF_WFLAG = 24u << 20;
constexpr unsigned OFF_XFLAG = OFF_WFLAG + 16384;
constexpr unsigned OFF_SFLAG = OFF_XFLAG + 4096;
constexpr unsigned OFF_CTR = OFF_SFLAG + 4096;
constexpr unsigned OFF_TMO = OFF_CTR + 256 * 128;
constexpr unsigned WS_USED = OFF_TMO + 128;

constexpr int GRID = 256;
constexpr int BLOCK = 256;
#define PULLERS 18
#define GCELLS 4
constexpr unsigned long long DEADLINE = 100000000ull;

constexpr int AUX_SYS = 17;
constexpr int AUX_AGT = 16;

struct Params {
  const int8_t* x;
  const int8_t* w;
  const float* sx;
  const float* sw;
  float* out;
  char* ws[NR];
  int rank;
};

__device__ __forceinline__ __amdgpu_buffer_rsrc_t mk_rsrc(const void* p, unsigned bytes) {
  return __builtin_amdgcn_make_buffer_rsrc(const_cast<void*>(p), 0, bytes, 0x00020000);
}
__device__ __forceinline__ unsigned ld_sys(const unsigned* p) {
  return __hip_atomic_load(p, __ATOMIC_RELAXED, __HIP_MEMORY_SCOPE_SYSTEM);
}
__device__ __forceinline__ void st_sys(unsigned* p, unsigned v) {
  __hip_atomic_store(p, v, __ATOMIC_RELAXED, __HIP_MEMORY_SCOPE_SYSTEM);
}
__device__ __forceinline__ unsigned ld_agt(const unsigned* p) {
  return __hip_atomic_load(p, __ATOMIC_RELAXED, __HIP_MEMORY_SCOPE_AGENT);
}
__device__ __forceinline__ void st_agt(unsigned* p, unsigned v) {
  __hip_atomic_store(p, v, __ATOMIC_RELAXED, __HIP_MEMORY_SCOPE_AGENT);
}

__device__ __forceinline__ unsigned sched(unsigned t, unsigned d) {
  const unsigned grp = t / (8 * GCELLS), r = t % (8 * GCELLS);
  const unsigned ks = r / GCELLS, c = grp * GCELLS + r % GCELLS;
  const unsigned nb = (c + 8 * d) & 63;
  return nb * 8 + ks;
}
__device__ __forceinline__ unsigned sgroup(unsigned unit) {
  const unsigned nb = unit >> 3, ks = unit & 7;
  return (nb >> 1) * 4 + (ks >> 1);
}

__global__ void __launch_bounds__(BLOCK, 1) fused_kernel(Params p) {
  __shared__ unsigned lds_pad[21 * 1024];
  const unsigned b = blockIdx.x;
  const unsigned lane = threadIdx.x & 63;
  const unsigned wid = __builtin_amdgcn_readfirstlane(threadIdx.x >> 6);
  const unsigned rank = p.rank;
  char* const my = p.ws[rank];
  const unsigned role = (b >> 3) & 1;
  const unsigned idx = ((b >> 4) << 3) | (b & 7);
  const unsigned q = lane >> 4, c = lane & 15;

  unsigned* const ctr = (unsigned*)(my + OFF_CTR) + b * 32 + wid;
  unsigned* const tmo = (unsigned*)(my + OFF_TMO);
  const unsigned long long t_start = __builtin_amdgcn_s_memrealtime();

  if (role == 0) {
    const unsigned snb = (idx >> 2) * 2 + (wid & 1), sks = (idx & 3) * 2 + (wid >> 1);
    unsigned d[16];
    {
      const unsigned* src = (const unsigned*)(p.w + (size_t)(sks * 64 + q * 16) * NN + snb * 64 + c * 4);
#pragma unroll
      for (int r = 0; r < 16; ++r) d[r] = src[r * (NN / 4)];
    }
    const unsigned seq = __builtin_amdgcn_readfirstlane(ld_agt(ctr)) + 1;
    const unsigned par = seq & 1;
    v4u o[4];
#pragma unroll
    for (int g = 0; g < 4; ++g) {
      const unsigned t0 = __builtin_amdgcn_perm(d[4 * g + 1], d[4 * g + 0], 0x05010400u);
      const unsigned t1 = __builtin_amdgcn_perm(d[4 * g + 1], d[4 * g + 0], 0x07030602u);
      const unsigned t2 = __builtin_amdgcn_perm(d[4 * g + 3], d[4 * g + 2], 0x05010400u);
      const unsigned t3 = __builtin_amdgcn_perm(d[4 * g + 3], d[4 * g + 2], 0x07030602u);
      o[0][g] = __builtin_amdgcn_perm(t2, t0, 0x05040100u);
      o[1][g] = __builtin_amdgcn_perm(t2, t0, 0x07060302u);
      o[2][g] = __builtin_amdgcn_perm(t3, t1, 0x05040100u);
      o[3][g] = __builtin_amdgcn_perm(t3, t1, 0x07060302u);
    }
    {
      const __amdgpu_buffer_rsrc_t rs = mk_rsrc(my + OFF_WST + par * SHARD, SHARD);
      const unsigned off = (snb * 8 + sks) * UNIT + lane * 16;
#pragma unroll
      for (int j = 0; j < 4; ++j) __builtin_amdgcn_raw_buffer_store_b128(o[j], rs, off + j * 1024, 0, AUX_SYS);
    }
    asm volatile("s_waitcnt vmcnt(0)" ::: "memory");
    if (lane == 0) {
      st_sys((unsigned*)(my + OFF_WFLAG) + snb * 64 + rank * 8 + sks, seq);
      st_agt(ctr, seq);
    }
    __syncthreads();
    if (wid == 0 && lane >= 1 && lane < NR) {
      const unsigned peer = (rank + lane) & (NR - 1);
      st_sys((unsigned*)(p.ws[peer] + OFF_SFLAG) + rank * 128 + idx, seq);
    }
    asm volatile("" :: "v"(&lds_pad[threadIdx.x]) : "memory");
    __builtin_amdgcn_fence(__ATOMIC_ACQUIRE, "");
    asm volatile("s_waitcnt vmcnt(0)" ::: "memory");

    const unsigned xcd = idx & 7, sub = idx >> 3;
    const unsigned nb = (sub >> 1) * 8 + xcd;
    const unsigned mb = (sub & 1) * 4 + wid;
    const unsigned qn = nb >> 3;
    unsigned pd;
    {
      const unsigned ord = lane >> 3;
      unsigned dd = (qn + 9 - ord) & 7;
      const bool passed = (ord >= 1) && (ord - 1 >= qn);
      if (passed) dd = (dd + 7) & 7;
      pd = ord == 0 ? 0 : dd;
    }
    const unsigned psrc = (rank + pd) & 7;
    const unsigned* const wf = (const unsigned*)(my + OFF_WFLAG) + nb * 64 + psrc * 8 + (lane & 7);
    const unsigned* const xf = (const unsigned*)(my + OFF_XFLAG) + mb * 64 + psrc * 8 + (lane & 7);

    v4i acc[4][4];
#pragma unroll
    for (int i = 0; i < 4; ++i)
#pragma unroll
      for (int j = 0; j < 4; ++j) acc[i][j] = v4i{0, 0, 0, 0};
    v4i ra[4][4], rb[4][4];
#pragma unroll
    for (int s = 0; s < 4; ++s)
#pragma unroll
      for (int i = 0; i < 4; ++i) { ra[s][i] = v4i{0, 0, 0, 0}; rb[s][i] = v4i{0, 0, 0, 0}; }

    unsigned long long ready = 0;
    bool failed = false;
    const __amdgpu_buffer_rsrc_t rws = mk_rsrc(my, OFF_WFLAG);
    for (int ord = 0; ord < 8; ++ord) {
      const unsigned src = __builtin_amdgcn_readlane(psrc, ord * 8);
      const unsigned aoff = OFF_X + par * (NR * XBLK) + src * XBLK + mb * (8 * UNIT);
      const unsigned boff = (src == rank ? OFF_WST + par * SHARD : OFF_WG + src * SHARD) + nb * (8 * UNIT);
#pragma unroll
      for (int ks = 0; ks < 8; ++ks) {
        const unsigned long long bit = 1ull << (ord * 8 + ks);
        while (!(ready & bit)) {
          const unsigned fw = ld_sys(wf), fx = ld_sys(xf);
          ready = __ballot(fw >= seq && fx >= seq);
          if (ready & bit) break;
          if (ld_agt(tmo) == seq || __builtin_amdgcn_s_memrealtime() - t_start > DEADLINE) {
            st_agt(tmo, seq);
            failed = true;
            ready = ~0ull;
            break;
          }
          __builtin_amdgcn_s_sleep(1);
        }
        asm volatile("" ::: "memory");
        const int s = ks & 3;
#pragma unroll
        for (int i = 0; i < 4; ++i)
          rb[s][i] = (v4i)__builtin_amdgcn_raw_buffer_load_b128(rws, lane * 16 + i * 1024, boff + ks * UNIT, 0);
#pragma unroll
        for (int i = 0; i < 4; ++i)
          ra[s][i] = (v4i)__builtin_amdgcn_raw_buffer_load_b128(rws, lane * 16 + i * 1024, aoff + ks * UNIT, 0);
        const int s3 = (ks + 1) & 3;
#pragma unroll
        for (int i = 0; i < 4; ++i)
#pragma unroll
          for (int j = 0; j < 4; ++j)
            acc[i][j] = __builtin_amdgcn_mfma_i32_16x16x64_i8(ra[s3][i], rb[s3][j], acc[i][j], 0, 0, 0);
      }
    }
#pragma unroll
    for (int s = 1; s < 4; ++s)
#pragma unroll
      for (int i = 0; i < 4; ++i)
#pragma unroll
        for (int j = 0; j < 4; ++j)
          acc[i][j] = __builtin_amdgcn_mfma_i32_16x16x64_i8(ra[s][i], rb[s][j], acc[i][j], 0, 0, 0);

    float scale = p.sx[0] * p.sw[0];
    if (failed || ld_agt(tmo) == seq) scale = __builtin_nanf("");
    const __amdgpu_buffer_rsrc_t ro = mk_rsrc(p.out, (unsigned)MS * NN * 4);
#pragma unroll
    for (int i = 0; i < 4; ++i)
#pragma unroll
      for (int r = 0; r < 4; ++r) {
        v4f v;
#pragma unroll
        for (int j = 0; j < 4; ++j) v[j] = (float)acc[i][j][r] * scale;
        const unsigned row = mb * 64 + i * 16 + q * 4 + r;
        const unsigned off = (row * NN + nb * 64 + c * 4) * 4;
        __builtin_amdgcn_raw_buffer_store_b128((v4u)v, ro, off, 0, 0);
      }
  } else {
    const unsigned dd = ((idx & 7) + 1) & 7;
    const unsigned dst = (rank + dd) & 7;
    const unsigned xu = (idx >> 3) * 4 + wid;
    const unsigned xmb = xu >> 3, xks = xu & 7;
    v4u v[4];
    {
      const int8_t* src = p.x + (size_t)(dst * MS + xmb * 64 + c) * KS + xks * 64 + q * 16;
#pragma unroll
      for (int i = 0; i < 4; ++i) v[i] = *(const v4u*)(src + i * 16 * KS);
    }
    const unsigned seq = __builtin_amdgcn_readfirstlane(ld_agt(ctr)) + 1;
    const unsigned par = seq & 1;
    {
      const __amdgpu_buffer_rsrc_t rs = mk_rsrc(p.ws[dst] + OFF_X + par * (NR * XBLK) + rank * XBLK, XBLK);
      const unsigned off = xu * UNIT + lane * 16;
#pragma unroll
      for (int i = 0; i < 4; ++i) __builtin_amdgcn_raw_buffer_store_b128(v[i], rs, off + i * 1024, 0, AUX_SYS);
    }
    asm volatile("s_waitcnt vmcnt(0)" ::: "memory");
    if (lane == 0) {
      st_sys((unsigned*)(p.ws[dst] + OFF_XFLAG) + xmb * 64 + rank * 8 + xks, seq);
      st_agt(ctr, seq);
    }
    if (wid == 0 && idx < 7 * PULLERS) {
      const unsigned sd = 1 + idx % 7;
      const unsigned src = (rank + sd) & 7;
      const unsigned pi = idx / 7;
      const __amdgpu_buffer_rsrc_t rsrc = mk_rsrc(p.ws[src] + OFF_WST + par * SHARD, SHARD);
      const __amdgpu_buffer_rsrc_t rdst = mk_rsrc(my + OFF_WG + src * SHARD, SHARD);
      unsigned* const wfl = (unsigned*)(my + OFF_WFLAG) + src * 8;
      const unsigned* const sf = (const unsigned*)(my + OFF_SFLAG) + src * 128;
      unsigned long long m0 = 0, m1 = 0;
      bool failed = false;
      v4u cur[4];
      unsigned prev_unit = 0xffffffffu;
      for (unsigned t = pi; t < 512 + PULLERS; t += PULLERS) {
        if (prev_unit != 0xffffffffu) {
#pragma unroll
          for (int j = 0; j < 4; ++j)
            __builtin_amdgcn_raw_buffer_store_b128(cur[j], rdst, prev_unit * UNIT + j * 1024 + lane * 16, 0, AUX_AGT);
        }
        unsigned unit = 0xffffffffu;
        if (t < 512) {
          unit = sched(t, sd);
          const unsigned g = sgroup(unit);
          while (!(((g < 64 ? m0 : m1) >> (g & 63)) & 1)) {
            const unsigned f0 = ld_sys(sf + lane), f1 = ld_sys(sf + 64 + lane);
            m0 = __ballot(f0 >= seq);
            m1 = __ballot(f1 >= seq);
            if (((g < 64 ? m0 : m1) >> (g & 63)) & 1) break;
            if (ld_agt(tmo) == seq || __builtin_amdgcn_s_memrealtime() - t_start > DEADLINE) {
              st_agt(tmo, seq);
              failed = true;
              break;
            }
            __builtin_amdgcn_s_sleep(1);
          }
          if (failed) break;
          asm volatile("" ::: "memory");
          v4u nxt[4];
#pragma unroll
          for (int j = 0; j < 4; ++j)
            nxt[j] = __builtin_amdgcn_raw_buffer_load_b128(rsrc, unit * UNIT + j * 1024 + lane * 16, 0, AUX_SYS);
          asm volatile("s_waitcnt vmcnt(4)" ::: "memory");
          if (prev_unit != 0xffffffffu && lane == 0) st_agt(wfl + (prev_unit >> 3) * 64 + (prev_unit & 7), seq);
#pragma unroll
          for (int j = 0; j < 4; ++j) cur[j] = nxt[j];
        } else {
          asm volatile("s_waitcnt vmcnt(0)" ::: "memory");
          if (prev_unit != 0xffffffffu && lane == 0) st_agt(wfl + (prev_unit >> 3) * 64 + (prev_unit & 7), seq);
        }
        prev_unit = unit;
      }
    }
  }
}

}

extern "C" void kernel_launch(int rank, int nranks, ncclComm_t comm,
                              hipStream_t stream,
                              void* const* d_in, const int* in_sizes, int n_in,
                              void* d_out, int out_size,
                              const int* in_dtypes, int out_dtype,
                              void* const* d_out_all,
                              void* const* d_ws_all, size_t ws_bytes) {
  (void)comm; (void)in_dtypes; (void)out_dtype; (void)d_out_all;
  if (nranks != NR || (unsigned)rank >= (unsigned)NR || n_in != 4 || ws_bytes < WS_USED ||
      in_sizes[0] != NR * MS * KS || in_sizes[1] != KS * NN || out_size != MS * NN) {
    fprintf(stderr, "unexpected problem geometry\n");
    return;
  }
  Params p;
  p.x = (const int8_t*)d_in[0];
  p.w = (const int8_t*)d_in[1];
  p.sx = (const float*)d_in[2];
  p.sw = (const float*)d_in[3];
  p.out = (float*)d_out;
  for (int i = 0; i < NR; ++i) p.ws[i] = (char*)d_ws_all[i];
  p.rank = rank;
  fused_kernel<<<GRID, BLOCK, 0, stream>>>(p);
  HIP_CHECK(hipGetLastError());
}
```

```cpp
#include <hip/hip_runtime.h>
#include <rccl/rccl.h>
#include <cstdio>
#include <cstdint>

#define HIP_CHECK(expr) do { hipError_t _e = (expr); if (_e != hipSuccess) { \
    fprintf(stderr, "HIP error %s at %s:%d\n", hipGetErrorString(_e), __FILE__, __LINE__); \
    return; } } while (0)

namespace {

typedef int v4i __attribute__((ext_vector_type(4)));
typedef unsigned v4u __attribute__((ext_vector_type(4)));
typedef float v4f __attribute__((ext_vector_type(4)));

constexpr int NR = 8;
constexpr int NN = 4096;
constexpr int MS = 512;
constexpr int KS = 512;
constexpr unsigned UNIT = 4096;
constexpr unsigned SHARD = 2u << 20;
constexpr unsigned XBLK = 256u << 10;

#define PL 14
#define NXG 8
#define XPOS 32
#define GCELLS 4
#define HELP_MIN 48
#define POLLW 2
#define AFFINE 1
#define GSLEEP 1
#define QSLEEP 1
#define PSLEEP 1
#define OUT_AUX 16
#define WGPAD 0
#define SWAPROLE 0

constexpr unsigned STG = 4u << 20;
constexpr unsigned OFF_STG = 0;
constexpr unsigned OFF_X = 8u << 20;
constexpr unsigned OFF_WG = 12u << 20;
constexpr unsigned WGS = SHARD + WGPAD;
constexpr unsigned OFF_FLG = 29u << 20;
constexpr unsigned OFF_WFLAG = OFF_FLG;
constexpr unsigned OFF_XFLAG = OFF_WFLAG + 512 * 128;
constexpr unsigned OFF_SFLAG = OFF_XFLAG + 64 * 128;
constexpr unsigned OFF_SXFLAG = OFF_SFLAG + 1024 * 128;
constexpr unsigned OFF_CTR = OFF_SXFLAG + 128 * 128;
constexpr unsigned OFF_QCTR = OFF_CTR + 256 * 128;
constexpr unsigned OFF_TMO = OFF_QCTR + 16 * 128;
constexpr unsigned WS_USED = OFF_TMO + 128;
static_assert(OFF_WG + NR * WGS <= OFF_FLG, "layout");

constexpr unsigned NXU = (16 - NXG) * 4;
constexpr unsigned NT = 512 + NXU;

constexpr int GRID = 256;
constexpr int BLOCK = 384;
constexpr unsigned long long DEADLINE = 100000000ull;

constexpr int AUX_SYS = 17;
constexpr int AUX_AGT = 16;
constexpr unsigned OOB = 0xfffffff0u;

#define STAMP(i) do {} while (0)

struct Params {
  const int8_t* x;
  const int8_t* w;
  const float* sx;
  const float* sw;
  float* out;
  char* ws[NR];
  int rank;
};

__device__ __forceinline__ __amdgpu_buffer_rsrc_t mk_rsrc(const void* p, unsigned bytes) {
  return __builtin_amdgcn_make_buffer_rsrc(const_cast<void*>(p), 0, bytes, 0x00020000);
}
__device__ __forceinline__ unsigned ld_sys(const unsigned* p) {
  return __hip_atomic_load(p, __ATOMIC_RELAXED, __HIP_MEMORY_SCOPE_SYSTEM);
}
__device__ __forceinline__ void st_sys(unsigned* p, unsigned v) {
  __hip_atomic_store(p, v, __ATOMIC_RELAXED, __HIP_MEMORY_SCOPE_SYSTEM);
}
__device__ __forceinline__ unsigned ld_agt(const unsigned* p) {
  return __hip_atomic_load(p, __ATOMIC_RELAXED, __HIP_MEMORY_SCOPE_AGENT);
}
__device__ __forceinline__ void st_agt(unsigned* p, unsigned v) {
  __hip_atomic_store(p, v, __ATOMIC_RELAXED, __HIP_MEMORY_SCOPE_AGENT);
}
__device__ __forceinline__ unsigned ld_lds(const unsigned* p) {
  return __hip_atomic_load(p, __ATOMIC_RELAXED, __HIP_MEMORY_SCOPE_WORKGROUP);
}
__device__ __forceinline__ void st_lds(unsigned* p, unsigned v) {
  __hip_atomic_store(p, v, __ATOMIC_RELAXED, __HIP_MEMORY_SCOPE_WORKGROUP);
}

__device__ __forceinline__ unsigned slot_of(unsigned nb, unsigned ks) {
  return (nb / GCELLS) * (8 * GCELLS) + ks * GCELLS + nb % GCELLS;
}
__device__ __forceinline__ unsigned ord_dist(unsigned ord, unsigned nb) {
  const unsigned qn = nb >> 3;
  unsigned dd = (qn + 9 - ord) & 7;
  if (ord - 1 >= qn) dd = (dd + 7) & 7;
  return ord == 0 ? 0 : dd;
}

struct Move { unsigned soff, doff, foff; };
__device__ __forceinline__ Move decode(unsigned t, unsigned src, unsigned sd, unsigned rank, unsigned par) {
  const bool isx = t - XPOS < NXU;
  const unsigned xu = NXG * 4 + (t - XPOS);
  const unsigned tw = t < XPOS ? t : t - NXU;
  const unsigned slot = (tw + 64 * sd) & 511;
  const unsigned r = slot % (8 * GCELLS);
  const unsigned nb = (slot / (8 * GCELLS)) * GCELLS + r % GCELLS, ks = r / GCELLS;
  Move m;
  m.soff = (isx ? 512 + rank * 64 + xu : slot) * UNIT;
  m.doff = isx ? par * (NR * XBLK) + (src * 64 + xu) * UNIT : (OFF_WG - OFF_X) + src * WGS + slot * UNIT;
  m.foff = isx ? (OFF_XFLAG - OFF_FLG) + (((xu >> 3) * 8 + src) * 32 + (xu & 7)) * 4
               : (OFF_WFLAG - OFF_FLG) + ((nb * 8 + src) * 32 + ks) * 4;
  return m;
}

__global__ void __launch_bounds__(BLOCK, 2) fused_kernel(Params p) {
  __shared__ unsigned lds[21 * 1024];
  const unsigned b = blockIdx.x;
  const unsigned lane = threadIdx.x & 63;
  const unsigned wid = __builtin_amdgcn_readfirstlane(threadIdx.x >> 6);
  const unsigned rank = p.rank;
  char* const my = p.ws[rank];
  const unsigned sidx = (b >> 3) * 4 + (b & 3);
  const unsigned role = (b >> 3) & 1;
  const unsigned idx = ((b >> 4) << 3) | (b & 7);
  const unsigned q = lane >> 4, c = lane & 15;

  unsigned* const ctr = (unsigned*)(my + OFF_CTR) + b * 32 + wid;
  unsigned* const tmo = (unsigned*)(my + OFF_TMO);
  const unsigned long long t_start = __builtin_amdgcn_s_memrealtime();
  unsigned seq;

  STAMP(0);
  if (wid >= 4) {
    if (wid == 4 && lane < 4) st_lds(&lds[lane], 0);
    asm volatile("s_waitcnt lgkmcnt(0)" ::: "memory");
    __builtin_amdgcn_s_barrier();
    asm volatile("" ::: "memory");
    seq = __builtin_amdgcn_readfirstlane(ld_agt(ctr)) + 1;
  } else if (((b >> 2) & 1) == SWAPROLE) {
    const unsigned dd = (((sidx >> 2) & 7) + 1) & 7;
    const unsigned dst = (rank + dd) & 7;
    const unsigned xg = (sidx >> 5) * 4 + (sidx & 3);
    v4u v[4];
    const int8_t* src = p.x + (size_t)(dst * MS + (xg >> 1) * 64 + c) * KS + (xg & 1) * 256 + wid * 64 + q * 16;
#pragma unroll
    for (int i = 0; i < 4; ++i) v[i] = *(const v4u*)(src + i * 16 * KS);
    const unsigned cv = ld_agt(ctr);
    char* const dws = p.ws[dst];
    __builtin_amdgcn_s_barrier();
    seq = __builtin_amdgcn_readfirstlane(cv) + 1;
    const bool push = xg < NXG || dd == 0;
    char* const base = push ? dws + OFF_X + (seq & 1) * (NR * XBLK) + rank * XBLK
                            : my + OFF_STG + (seq & 1) * STG + SHARD + dst * XBLK;
    const __amdgpu_buffer_rsrc_t rs = mk_rsrc(base, XBLK);
    const unsigned off = (xg * 4 + wid) * UNIT + lane * 16;
#pragma unroll
    for (int i = 0; i < 4; ++i) __builtin_amdgcn_raw_buffer_store_b128(v[i], rs, off + i * 1024, 0, AUX_SYS);
    asm volatile("s_waitcnt vmcnt(0)" ::: "memory");
    STAMP(1);
    unsigned old = 0;
    if (lane == 0) old = __hip_atomic_fetch_add(&lds[3], 1u, __ATOMIC_RELAXED, __HIP_MEMORY_SCOPE_WORKGROUP);
    old = __builtin_amdgcn_readfirstlane(old);
    if (old == 3) {
      const unsigned fo = push ? OFF_XFLAG + (((xg >> 1) * 8 + rank) * 32 + (xg & 1) * 4) * 4
                               : OFF_SXFLAG + (rank * 16 + xg) * 128;
      if (lane < 4) st_sys((unsigned*)(dws + fo) + lane, seq);
    }
    if (lane == 0) st_agt(ctr, seq);
  } else {
    const unsigned snb = (sidx >> 2) * 2 + (wid & 1), sks = (sidx & 3) * 2 + (wid >> 1);
    char* const peer_ws = p.ws[(rank + lane) & (NR - 1)];
    unsigned d[16];
    {
      const unsigned* src = (const unsigned*)(p.w + (size_t)(sks * 64 + q * 16) * NN + snb * 64 + c * 4);
#pragma unroll
      for (int r = 0; r < 16; ++r) d[r] = src[r * (NN / 4)];
    }
    const unsigned cv = ld_agt(ctr);
    __builtin_amdgcn_s_barrier();
    seq = __builtin_amdgcn_readfirstlane(cv) + 1;
    v4u o[4];
#pragma unroll
    for (int g = 0; g < 4; ++g) {
      const unsigned t0 = __builtin_amdgcn_perm(d[4 * g + 1], d[4 * g + 0], 0x05010400u);
      const unsigned t1 = __builtin_amdgcn_perm(d[4 * g + 1], d[4 * g + 0], 0x07030602u);
      const unsigned t2 = __builtin_amdgcn_perm(d[4 * g + 3], d[4 * g + 2], 0x05010400u);
      const unsigned t3 = __builtin_amdgcn_perm(d[4 * g + 3], d[4 * g + 2], 0x07030602u);
      o[0][g] = __builtin_amdgcn_perm(t2, t0, 0x05040100u);
      o[1][g] = __builtin_amdgcn_perm(t2, t0, 0x07060302u);
      o[2][g] = __builtin_amdgcn_perm(t3, t1, 0x05040100u);
      o[3][g] = __builtin_amdgcn_perm(t3, t1, 0x07060302u);
    }
    const __amdgpu_buffer_rsrc_t rs = mk_rsrc(my + OFF_STG + (seq & 1) * STG, SHARD);
    const unsigned off = slot_of(snb, sks) * UNIT + lane * 16;
#pragma unroll
    for (int j = 0; j < 4; ++j) __builtin_amdgcn_raw_buffer_store_b128(o[j], rs, off + j * 1024, 0, AUX_SYS);
    asm volatile("s_waitcnt vmcnt(0)" ::: "memory");
    STAMP(1);
    unsigned old = 0;
    if (lane == 0) old = __hip_atomic_fetch_add(&lds[3], 1u, __ATOMIC_RELAXED, __HIP_MEMORY_SCOPE_WORKGROUP);
    old = __builtin_amdgcn_readfirstlane(old);
    if (old == 3 && lane >= 1 && lane < NR)
      st_sys((unsigned*)(peer_ws + OFF_SFLAG) + (rank * 128 + sidx) * 32, seq);
    if (lane == 0) {
      st_agt((unsigned*)(my + OFF_WFLAG) + (snb * 8 + rank) * 32 + sks, seq);
      st_agt(ctr, seq);
    }
  }
  seq = __builtin_amdgcn_readfirstlane(seq);
  const unsigned par = seq & 1;

  if (role == 0) {
    const unsigned sub = idx >> 3;
    const unsigned nb = (sub >> 1) * 8 + (idx & 7);
    const unsigned mb0 = (sub & 1) * 4;
    if (wid == 4) {
      const unsigned psrc = (rank + ord_dist(lane >> 3, nb)) & 7;
      const unsigned* const wf = (const unsigned*)(my + OFF_WFLAG) + (nb * 8 + psrc) * 32 + (lane & 7);
      const __amdgpu_buffer_rsrc_t rxf = mk_rsrc(my + OFF_XFLAG, 64 * 128);
      const unsigned xfo = (((mb0 + (lane & 3)) * 8 + psrc) * 32 + ((lane >> 2) & 1) * 4) * 4;
      bool wok = false, xok = false;
      unsigned long long last = 0;
      for (unsigned spin = 0;; ++spin) {
        const unsigned front = (unsigned)__builtin_ctzll(~last) >> 3;
        const bool near = (lane >> 3) < front + POLLW;
        if (!wok && near) wok = ld_agt(wf) >= seq;
        if (!xok && near) {
          const v4u f = __builtin_amdgcn_raw_buffer_load_b128(rxf, xfo, 0, AUX_SYS);
          xok = f[0] >= seq && f[1] >= seq && f[2] >= seq && f[3] >= seq;
        }
        asm volatile("" ::: "memory");
        const unsigned long long xm = __ballot(xok);
        const unsigned long long m = __ballot(wok && ((xm >> (lane & 56)) & 0xff) == 0xff);
        if (m != last) {
          last = m;
          if (lane < 2) st_lds(&lds[lane], lane ? (unsigned)(m >> 32) : (unsigned)m);
          if (m == ~0ull) break;
        }
        if ((spin & 63) == 63 &&
            (ld_agt(tmo) == seq || __builtin_amdgcn_s_memrealtime() - t_start > DEADLINE)) {
          st_agt(tmo, seq);
          if (lane == 0) st_lds(&lds[2], 1);
          asm volatile("s_waitcnt lgkmcnt(0)" ::: "memory");
          if (lane < 2) st_lds(&lds[lane], 0xffffffffu);
          break;
        }
        if (QSLEEP) __builtin_amdgcn_s_sleep(QSLEEP);
      }
    } else if (wid < 4) {
      const unsigned mb = mb0 + wid;
      const float scale0 = p.sx[0] * p.sw[0];
      v4i acc[4][4];
#pragma unroll
      for (int i = 0; i < 4; ++i)
#pragma unroll
        for (int j = 0; j < 4; ++j) acc[i][j] = v4i{0, 0, 0, 0};
      v4i ra[4][4], rb[4][4];
#pragma unroll
      for (int s = 0; s < 4; ++s)
#pragma unroll
        for (int i = 0; i < 4; ++i) { ra[s][i] = v4i{0, 0, 0, 0}; rb[s][i] = v4i{0, 0, 0, 0}; }

      unsigned tmo_val = 0;
      const __amdgpu_buffer_rsrc_t rws = mk_rsrc(my, OFF_FLG);
      for (unsigned ord = 0; ord < 8; ++ord) {
        const unsigned src = (rank + ord_dist(ord, nb)) & 7;
        const unsigned aoff = OFF_X + par * (NR * XBLK) + src * XBLK + mb * (8 * UNIT);
        const unsigned boff = (src == rank ? OFF_STG + par * STG : OFF_WG + src * WGS) + slot_of(nb, 0) * UNIT;
        const unsigned* const mword = &lds[ord >> 2];
        const unsigned sh = (ord & 3) * 8;
        unsigned ready = 0;
#pragma unroll
        for (int ks = 0; ks < 8; ++ks) {
          while (!((ready >> ks) & 1)) {
            ready = __builtin_amdgcn_readfirstlane(ld_lds(mword)) >> sh;
            if ((ready >> ks) & 1) break;
            if (GSLEEP) __builtin_amdgcn_s_sleep(GSLEEP);
          }
          asm volatile("" ::: "memory");
          if (ks == 7) STAMP(3 + ord);
          const int s = ks & 3;
#pragma unroll
          for (int i = 0; i < 4; ++i)
            rb[s][i] = (v4i)__builtin_amdgcn_raw_buffer_load_b128(rws, lane * 16 + i * 1024, boff + ks * (GCELLS * UNIT), 0);
#pragma unroll
          for (int i = 0; i < 4; ++i)
            ra[s][i] = (v4i)__builtin_amdgcn_raw_buffer_load_b128(rws, lane * 16 + i * 1024, aoff + ks * UNIT, 0);
          if (ks == 7) tmo_val = ld_agt(tmo);
          const int s3 = (ks + 1) & 3;
#pragma unroll
          for (int i = 0; i < 4; ++i)
#pragma unroll
            for (int j = 0; j < 4; ++j)
              acc[i][j] = __builtin_amdgcn_mfma_i32_16x16x64_i8(ra[s3][i], rb[s3][j], acc[i][j], 0, 0, 0);
        }
      }
#pragma unroll
      for (int s = 1; s < 4; ++s)
#pragma unroll
        for (int i = 0; i < 4; ++i)
#pragma unroll
          for (int j = 0; j < 4; ++j)
            acc[i][j] = __builtin_amdgcn_mfma_i32_16x16x64_i8(ra[s][i], rb[s][j], acc[i][j], 0, 0, 0);

      STAMP(11);
      const bool failed = ld_lds(&lds[2]) != 0;
      const float scale = (failed || tmo_val == seq) ? __builtin_nanf("") : scale0;
      const __amdgpu_buffer_rsrc_t ro = mk_rsrc(p.out, (unsigned)MS * NN * 4);
#pragma unroll
      for (int i = 0; i < 4; ++i)
#pragma unroll
        for (int r = 0; r < 4; ++r) {
          v4f v;
#pragma unroll
          for (int j = 0; j < 4; ++j) v[j] = (float)acc[i][j][r] * scale;
          const unsigned row = mb * 64 + i * 16 + q * 4 + r;
          const unsigned off = (row * NN + nb * 64 + c * 4) * 4;
          __builtin_amdgcn_raw_buffer_store_b128((v4u)v, ro, off, 0, OUT_AUX);
        }
    }
  } else if (wid >= 4) {
    const unsigned xcc = __builtin_amdgcn_s_getreg((3 << 11) | 20) & 7;
    const unsigned row = (wid - 4) * 16 + (b >> 4);
    const unsigned CLQ = 0xB2u;
    const unsigned same = (((CLQ >> rank) & 1) ? CLQ : (~CLQ & 0xffu)) & ~(1u << rank);
    const unsigned other = ~same & ~(1u << rank) & 0xffu;
    unsigned cand = AFFINE == 1 ? ((xcc & 4) ? other : same) : ((xcc & 4) ? same : other);
    unsigned li = (row + xcc) & 3;
    if (AFFINE == 0) { cand = 0xffu & ~(1u << rank); li = (row + xcc) & 7; }
    unsigned src0 = 8;
    {
      unsigned m = cand;
      for (unsigned i = 0; i < li && m; ++i) m &= m - 1;
      if (m) src0 = __builtin_ctz(m);
    }
    if (row < PL && src0 < 8) {
      unsigned* const qh0 = (unsigned*)(my + OFF_QCTR) + par * (8 * 32);
      const __amdgpu_buffer_rsrc_t rdst = mk_rsrc(my + OFF_X, OFF_FLG - OFF_X);
      const __amdgpu_buffer_rsrc_t rflag = mk_rsrc(my + OFF_FLG, OFF_SFLAG - OFF_FLG);
      unsigned vzero = 0, vone = 1;
      asm volatile("" : "+v"(vzero), "+v"(vone));
      bool failed = false;
      const unsigned lane16 = lane * 16;
      unsigned src = src0;
      for (unsigned hop = 0; hop < 16 && !failed; ++hop) {
        if (hop > 0) {
          const unsigned head = ld_agt(qh0 + (lane & 7) * 32);
          unsigned m = (unsigned)__ballot(head + HELP_MIN < NT) & 0xffu & ~(1u << rank);
          if (!m) break;
          m = (m | (m << 8)) >> ((src + 1) & 7);
          src = (src + 1 + __builtin_ctz(m)) & 7;
        }
        const unsigned sd = (src - rank) & 7;
        unsigned* const qh = qh0 + src * 32;
        unsigned tk = 0;
        if (lane == 0)
          asm volatile("global_atomic_add %0, %1, %2, %3 sc0\n\ts_waitcnt vmcnt(0)"
                       : "=&v"(tk) : "v"(vzero), "v"(vone), "s"(qh) : "memory");
        unsigned t = __builtin_amdgcn_readfirstlane(tk);
        if (t >= NT) continue;
        const unsigned* const sf = (const unsigned*)(my + OFF_SFLAG) + src * (128 * 32);
        const unsigned g0 = ((lane + 16 * sd) & 127) * 32, g1 = ((lane + 64 + 16 * sd) & 127) * 32;
        const unsigned* const sxf = (const unsigned*)(my + OFF_SXFLAG) + (src * 16 + (lane & 15)) * 32;
        unsigned known = 0;
        const __amdgpu_buffer_rsrc_t rsrc = mk_rsrc(p.ws[src] + OFF_STG + par * STG, STG);
        auto wait_staged = [&](unsigned tt) {
          for (unsigned spin = 0;; ++spin) {
            const unsigned f0 = ld_sys(sf + g0), f1 = ld_sys(sf + g1), fx = ld_sys(sxf);
            const unsigned long long m0 = __ballot(f0 >= seq), m1 = __ballot(f1 >= seq);
            const bool xall = (__ballot(fx >= seq || (lane & 15) < NXG) & 0xffffull) == 0xffffull;
            const unsigned lead = ~m0 ? (unsigned)__builtin_ctzll(~m0) : 64 + (~m1 ? (unsigned)__builtin_ctzll(~m1) : 64);
            constexpr unsigned GG = GCELLS < 2 ? 2 : GCELLS;
            const unsigned kw = lead / (2 * GG) * (8 * GG);
            known = kw <= XPOS ? kw : (xall ? kw + NXU : XPOS);
            if (tt < known) break;
            if ((spin & 63) == 63 &&
                (ld_agt(tmo) == seq || __builtin_amdgcn_s_memrealtime() - t_start > DEADLINE)) {
              st_agt(tmo, seq);
              failed = true;
              break;
            }
            if (PSLEEP) __builtin_amdgcn_s_sleep(PSLEEP);
          }
          asm volatile("" ::: "memory");
        };
        Move mv = decode(t, src, sd, rank, par);
        wait_staged(t);
        if (failed) break;
        if (hop == 0) STAMP(2);
        v4u cur[4];
        if (lane == 0)
          asm volatile("global_atomic_add %0, %1, %2, %3 sc0" : "=v"(tk) : "v"(vzero), "v"(vone), "s"(qh) : "memory");
#pragma unroll
        for (int j = 0; j < 4; ++j)
          cur[j] = __builtin_amdgcn_raw_buffer_load_b128(rsrc, lane16 + j * 1024, mv.soff, AUX_SYS);
        asm volatile("s_waitcnt vmcnt(4)" : "+v"(tk) :: "memory");
        __builtin_amdgcn_raw_buffer_store_b32(seq, rflag, OOB, 0, AUX_AGT);
        t = __builtin_amdgcn_readfirstlane(tk);
        while (t < NT) {
          if (t >= known) {
            wait_staged(t);
            if (failed) break;
          }
          const Move nx = decode(t, src, sd, rank, par);
#pragma unroll
          for (int j = 0; j < 4; ++j) __builtin_amdgcn_raw_buffer_store_b128(cur[j], rdst, mv.doff + lane16 + j * 1024, 0, AUX_AGT);
          if (lane == 0)
            asm volatile("global_atomic_add %0, %1, %2, %3 sc0" : "=v"(tk) : "v"(vzero), "v"(vone), "s"(qh) : "memory");
#pragma unroll
          for (int j = 0; j < 4; ++j)
            cur[j] = __builtin_amdgcn_raw_buffer_load_b128(rsrc, lane16 + j * 1024, nx.soff, AUX_SYS);
          asm volatile("s_waitcnt vmcnt(4)" : "+v"(tk) :: "memory");
          __builtin_amdgcn_raw_buffer_store_b32(seq, rflag, lane == 0 ? mv.foff : OOB, 0, AUX_AGT);
          t = __builtin_amdgcn_readfirstlane(tk);
          mv = nx;
        }
        if (failed) break;
#pragma unroll
        for (int j = 0; j < 4; ++j) __builtin_amdgcn_raw_buffer_store_b128(cur[j], rdst, mv.doff + lane16 + j * 1024, 0, AUX_AGT);
        asm volatile("s_waitcnt vmcnt(0)" ::: "memory");
        __builtin_amdgcn_raw_buffer_store_b32(seq, rflag, lane == 0 ? mv.foff : OOB, 0, AUX_AGT);
        if (hop == 0) STAMP(4);
        STAMP(3);
      }
    }
  }
  if (wid >= 4) {
    if (lane == 0) st_agt(ctr, seq);
    if (b == 8 && wid == 5 && lane < 8) st_agt((unsigned*)(my + OFF_QCTR) + ((par ^ 1) * 8 + lane) * 32, 0);
  }
}

}

extern "C" void kernel_launch(int rank, int nranks, ncclComm_t comm,
                              hipStream_t stream,
                              void* const* d_in, const int* in_sizes, int n_in,
                              void* d_out, int out_size,
                              const int* in_dtypes, int out_dtype,
                              void* const* d_out_all,
                              void* const* d_ws_all, size_t ws_bytes) {
  (void)comm; (void)in_dtypes; (void)out_dtype; (void)d_out_all;
  if (nranks != NR || (unsigned)rank >= (unsigned)NR || n_in != 4 || ws_bytes < WS_USED ||
      in_sizes[0] != NR * MS * KS || in_sizes[1] != KS * NN || out_size != MS * NN) {
    fprintf(stderr, "unexpected problem geometry\n");
    return;
  }
  Params p;
  p.x = (const int8_t*)d_in[0];
  p.w = (const int8_t*)d_in[1];
  p.sx = (const float*)d_in[2];
  p.sw = (const float*)d_in[3];
  p.out = (float*)d_out;
  for (int i = 0; i < NR; ++i) p.ws[i] = (char*)d_ws_all[i];
  p.rank = rank;
  fused_kernel<<<GRID, BLOCK, 0, stream>>>(p);
  HIP_CHECK(hipGetLastError());
}
```

```cpp
#include <hip/hip_runtime.h>
#include <rccl/rccl.h>
#include <cstdio>
#include <cstdint>

#define HIP_CHECK(expr) do { hipError_t _e = (expr); if (_e != hipSuccess) { \
    fprintf(stderr, "HIP error %s at %s:%d\n", hipGetErrorString(_e), __FILE__, __LINE__); \
    return; } } while (0)

namespace {

typedef int v4i __attribute__((ext_vector_type(4)));
typedef unsigned v4u __attribute__((ext_vector_type(4)));
typedef float v4f __attribute__((ext_vector_type(4)));

constexpr int NR = 8;
constexpr int NN = 4096;
constexpr int MS = 512;
constexpr int KS = 512;
constexpr unsigned UNIT = 4096;
constexpr unsigned SHARD = 2u << 20;
constexpr unsigned XBLK = 256u << 10;

#define PL 14
#define NXG 8
#define XPOS 32
#define GCELLS 8
#define HELP_MIN 64
#define POLLW 2
#define AFFINE 1
#define GSLEEP 1
#define QSLEEP 1
#define PSLEEP 1
#define OUT_AUX 16
#define PLAN 1
#define SNAP_AT 128
#define PLAN_DEAD 4
#define SWAPROLE 0

constexpr unsigned STG = 4u << 20;
constexpr unsigned OFF_STG = 0;
constexpr unsigned OFF_X = 8u << 20;
constexpr unsigned OFF_WG = 12u << 20;
constexpr unsigned WGS = SHARD;
constexpr unsigned WGB = NR * WGS;
constexpr unsigned OFF_FLG = 44u << 20;
constexpr unsigned OFF_WFLAG = OFF_FLG;
constexpr unsigned OFF_XFLAG = OFF_WFLAG + 512 * 128;
constexpr unsigned OFF_DENSE = OFF_XFLAG + 64 * 128;
constexpr unsigned OFF_SNAP = OFF_DENSE + 8 * 512 * 4;
constexpr unsigned OFF_SOWN = OFF_SNAP + 128;
constexpr unsigned OFF_SFLAG = OFF_SOWN + 128;
constexpr unsigned OFF_SXFLAG = OFF_SFLAG + 1024 * 128;
constexpr unsigned OFF_CTR = OFF_SXFLAG + 128 * 128;
constexpr unsigned OFF_QCTR = OFF_CTR + 256 * 128;
constexpr unsigned OFF_TMO = OFF_QCTR + 16 * 128;
constexpr unsigned WS_USED = OFF_TMO + 128;
static_assert(OFF_WG + 2 * WGB <= OFF_FLG, "layout");

constexpr unsigned NXU = (16 - NXG) * 4;
constexpr unsigned NT = 512 + NXU;

constexpr int GRID = 256;
constexpr int BLOCK = 384;
constexpr unsigned long long DEADLINE = 100000000ull;

constexpr int AUX_SYS = 17;
constexpr int AUX_AGT = 16;
constexpr unsigned OOB = 0xfffffff0u;

#define STAMP(i) do {} while (0)

struct Params {
  const int8_t* x;
  const int8_t* w;
  const float* sx;
  const float* sw;
  float* out;
  char* ws[NR];
  int rank;
};

__device__ __forceinline__ __amdgpu_buffer_rsrc_t mk_rsrc(const void* p, unsigned bytes) {
  return __builtin_amdgcn_make_buffer_rsrc(const_cast<void*>(p), 0, bytes, 0x00020000);
}
__device__ __forceinline__ unsigned ld_sys(const unsigned* p) {
  return __hip_atomic_load(p, __ATOMIC_RELAXED, __HIP_MEMORY_SCOPE_SYSTEM);
}
__device__ __forceinline__ void st_sys(unsigned* p, unsigned v) {
  __hip_atomic_store(p, v, __ATOMIC_RELAXED, __HIP_MEMORY_SCOPE_SYSTEM);
}
__device__ __forceinline__ unsigned ld_agt(const unsigned* p) {
  return __hip_atomic_load(p, __ATOMIC_RELAXED, __HIP_MEMORY_SCOPE_AGENT);
}
__device__ __forceinline__ void st_agt(unsigned* p, unsigned v) {
  __hip_atomic_store(p, v, __ATOMIC_RELAXED, __HIP_MEMORY_SCOPE_AGENT);
}
__device__ __forceinline__ unsigned ld_lds(const unsigned* p) {
  return __hip_atomic_load(p, __ATOMIC_RELAXED, __HIP_MEMORY_SCOPE_WORKGROUP);
}
__device__ __forceinline__ void st_lds(unsigned* p, unsigned v) {
  __hip_atomic_store(p, v, __ATOMIC_RELAXED, __HIP_MEMORY_SCOPE_WORKGROUP);
}

__device__ __forceinline__ unsigned slot_of(unsigned nb, unsigned ks) {
  return (nb / GCELLS) * (8 * GCELLS) + ks * GCELLS + nb % GCELLS;
}
__device__ __forceinline__ unsigned ord_dist(unsigned ord, unsigned nb) {
  const unsigned qn = nb >> 3;
  unsigned dd = (qn + 9 - ord) & 7;
  if (ord - 1 >= qn) dd = (dd + 7) & 7;
  return ord == 0 ? 0 : dd;
}

struct Move { unsigned soff, doff, foff, goff; };
__device__ __forceinline__ Move decode(unsigned t, unsigned src, unsigned rank, unsigned par) {
  const unsigned sd = (src - rank) & 7;
  const bool isx = t - XPOS < NXU;
  const unsigned xu = NXG * 4 + (t - XPOS);
  const unsigned tw = t < XPOS ? t : t - NXU;
  const unsigned slot = (tw + 64 * sd) & 511;
  const unsigned r = slot % (8 * GCELLS);
  const unsigned nb = (slot / (8 * GCELLS)) * GCELLS + r % GCELLS, ks = r / GCELLS;
  Move m;
  m.soff = (isx ? 512 + rank * 64 + xu : slot) * UNIT;
  m.doff = isx ? par * (NR * XBLK) + (src * 64 + xu) * UNIT : (OFF_WG - OFF_X) + par * WGB + src * WGS + slot * UNIT;
  m.foff = isx ? (OFF_XFLAG - OFF_FLG) + (((xu >> 3) * 8 + src) * 32 + (xu & 7)) * 4
               : (OFF_WFLAG - OFF_FLG) + ((nb * 8 + src) * 32 + ks) * 4;
  m.goff = isx ? OOB : (OFF_DENSE - OFF_FLG) + (src * 512 + slot) * 4;
  return m;
}

__global__ void __launch_bounds__(BLOCK, 2) fused_kernel(Params p) {
  __shared__ unsigned lds[21 * 1024];
  const unsigned b = blockIdx.x;
  const unsigned lane = threadIdx.x & 63;
  const unsigned wid = __builtin_amdgcn_readfirstlane(threadIdx.x >> 6);
  const unsigned rank = p.rank;
  char* const my = p.ws[rank];
  const unsigned sidx = (b >> 3) * 4 + (b & 3);
  const unsigned role = (b >> 3) & 1;
  const unsigned idx = ((b >> 4) << 3) | (b & 7);
  const unsigned q = lane >> 4, c = lane & 15;

  unsigned* const ctr = (unsigned*)(my + OFF_CTR) + b * 32 + wid;
  unsigned* const tmo = (unsigned*)(my + OFF_TMO);
  const unsigned long long t_start = __builtin_amdgcn_s_memrealtime();
  unsigned seq;

  STAMP(0);
  if (wid >= 4) {
    if (wid == 4 && lane < 4) st_lds(&lds[lane], 0);
    asm volatile("s_waitcnt lgkmcnt(0)" ::: "memory");
    __builtin_amdgcn_s_barrier();
    asm volatile("" ::: "memory");
    seq = __builtin_amdgcn_readfirstlane(ld_agt(ctr)) + 1;
  } else if (((b >> 2) & 1) == SWAPROLE) {
    const unsigned dd = (((sidx >> 2) & 7) + 1) & 7;
    const unsigned dst = (rank + dd) & 7;
    const unsigned xg = (sidx >> 5) * 4 + (sidx & 3);
    v4u v[4];
    const int8_t* src = p.x + (size_t)(dst * MS + (xg >> 1) * 64 + c) * KS + (xg & 1) * 256 + wid * 64 + q * 16;
#pragma unroll
    for (int i = 0; i < 4; ++i) v[i] = *(const v4u*)(src + i * 16 * KS);
    const unsigned cv = ld_agt(ctr);
    char* const dws = p.ws[dst];
    __builtin_amdgcn_s_barrier();
    seq = __builtin_amdgcn_readfirstlane(cv) + 1;
    const bool push = xg < NXG || dd == 0;
    char* const base = push ? dws + OFF_X + (seq & 1) * (NR * XBLK) + rank * XBLK
                            : my + OFF_STG + (seq & 1) * STG + SHARD + dst * XBLK;
    const __amdgpu_buffer_rsrc_t rs = mk_rsrc(base, XBLK);
    const unsigned off = (xg * 4 + wid) * UNIT + lane * 16;
#pragma unroll
    for (int i = 0; i < 4; ++i) __builtin_amdgcn_raw_buffer_store_b128(v[i], rs, off + i * 1024, 0, AUX_SYS);
    asm volatile("s_waitcnt vmcnt(0)" ::: "memory");
    STAMP(1);
    unsigned old = 0;
    if (lane == 0) old = __hip_atomic_fetch_add(&lds[3], 1u, __ATOMIC_RELAXED, __HIP_MEMORY_SCOPE_WORKGROUP);
    old = __builtin_amdgcn_readfirstlane(old);
    if (old == 3) {
      const unsigned fo = push ? OFF_XFLAG + (((xg >> 1) * 8 + rank) * 32 + (xg & 1) * 4) * 4
                               : OFF_SXFLAG + (rank * 16 + xg) * 128;
      if (lane < 4) st_sys((unsigned*)(dws + fo) + lane, seq);
    }
    if (lane == 0) st_agt(ctr, seq);
  } else {
    const unsigned snb = (sidx >> 2) * 2 + (wid & 1), sks = (sidx & 3) * 2 + (wid >> 1);
    char* const peer_ws = p.ws[(rank + lane) & (NR - 1)];
    unsigned d[16];
    {
      const unsigned* src = (const unsigned*)(p.w + (size_t)(sks * 64 + q * 16) * NN + snb * 64 + c * 4);
#pragma unroll
      for (int r = 0; r < 16; ++r) d[r] = src[r * (NN / 4)];
    }
    const unsigned cv = ld_agt(ctr);
    __builtin_amdgcn_s_barrier();
    seq = __builtin_amdgcn_readfirstlane(cv) + 1;
    v4u o[4];
#pragma unroll
    for (int g = 0; g < 4; ++g) {
      const unsigned t0 = __builtin_amdgcn_perm(d[4 * g + 1], d[4 * g + 0], 0x05010400u);
      const unsigned t1 = __builtin_amdgcn_perm(d[4 * g + 1], d[4 * g + 0], 0x07030602u);
      const unsigned t2 = __builtin_amdgcn_perm(d[4 * g + 3], d[4 * g + 2], 0x05010400u);
      const unsigned t3 = __builtin_amdgcn_perm(d[4 * g + 3], d[4 * g + 2], 0x07030602u);
      o[0][g] = __builtin_amdgcn_perm(t2, t0, 0x05040100u);
      o[1][g] = __builtin_amdgcn_perm(t2, t0, 0x07060302u);
      o[2][g] = __builtin_amdgcn_perm(t3, t1, 0x05040100u);
      o[3][g] = __builtin_amdgcn_perm(t3, t1, 0x07060302u);
    }
    const __amdgpu_buffer_rsrc_t rs = mk_rsrc(my + OFF_STG + (seq & 1) * STG, SHARD);
    const unsigned off = slot_of(snb, sks) * UNIT + lane * 16;
#pragma unroll
    for (int j = 0; j < 4; ++j) __builtin_amdgcn_raw_buffer_store_b128(o[j], rs, off + j * 1024, 0, AUX_SYS);
    asm volatile("s_waitcnt vmcnt(0)" ::: "memory");
    STAMP(1);
    unsigned old = 0;
    if (lane == 0) old = __hip_atomic_fetch_add(&lds[3], 1u, __ATOMIC_RELAXED, __HIP_MEMORY_SCOPE_WORKGROUP);
    old = __builtin_amdgcn_readfirstlane(old);
    if (old == 3 && lane >= 1 && lane < NR)
      st_sys((unsigned*)(peer_ws + OFF_SFLAG) + (rank * 128 + sidx) * 32, seq);
    if (lane == 0) {
      st_agt((unsigned*)(my + OFF_WFLAG) + (snb * 8 + rank) * 32 + sks, seq);
      st_agt(ctr, seq);
    }
  }
  seq = __builtin_amdgcn_readfirstlane(seq);
  const unsigned par = seq & 1;

  if (role == 0) {
    const unsigned sub = idx >> 3;
    const unsigned nb = (sub >> 1) * 8 + (idx & 7);
    const unsigned mb0 = (sub & 1) * 4;
    if (wid == 4) {
      const unsigned psrc = (rank + ord_dist(lane >> 3, nb)) & 7;
      const unsigned* const wf = (const unsigned*)(my + OFF_WFLAG) + (nb * 8 + psrc) * 32 + (lane & 7);
      const __amdgpu_buffer_rsrc_t rxf = mk_rsrc(my + OFF_XFLAG, 64 * 128);
      const unsigned xfo = (((mb0 + (lane & 3)) * 8 + psrc) * 32 + ((lane >> 2) & 1) * 4) * 4;
      bool wok = false, xok = false;
      unsigned long long last = 0;
      for (unsigned spin = 0;; ++spin) {
        const unsigned front = (unsigned)__builtin_ctzll(~last) >> 3;
        const bool near = (lane >> 3) < front + POLLW;
        if (!wok && near) wok = ld_agt(wf) >= seq;
        if (!xok && near) {
          const v4u f = __builtin_amdgcn_raw_buffer_load_b128(rxf, xfo, 0, AUX_SYS);
          xok = f[0] >= seq && f[1] >= seq && f[2] >= seq && f[3] >= seq;
        }
        asm volatile("" ::: "memory");
        const unsigned long long xm = __ballot(xok);
        const unsigned long long m = __ballot(wok && ((xm >> (lane & 56)) & 0xff) == 0xff);
        if (m != last) {
          last = m;
          if (lane < 2) st_lds(&lds[lane], lane ? (unsigned)(m >> 32) : (unsigned)m);
          if (m == ~0ull) break;
        }
        if ((spin & 63) == 63 &&
            (ld_agt(tmo) == seq || __builtin_amdgcn_s_memrealtime() - t_start > DEADLINE)) {
          st_agt(tmo, seq);
          if (lane == 0) st_lds(&lds[2], 1);
          asm volatile("s_waitcnt lgkmcnt(0)" ::: "memory");
          if (lane < 2) st_lds(&lds[lane], 0xffffffffu);
          break;
        }
        if (QSLEEP) __builtin_amdgcn_s_sleep(QSLEEP);
      }
    } else if (wid < 4) {
      const unsigned mb = mb0 + wid;
      const float scale0 = p.sx[0] * p.sw[0];
      v4i acc[4][4];
#pragma unroll
      for (int i = 0; i < 4; ++i)
#pragma unroll
        for (int j = 0; j < 4; ++j) acc[i][j] = v4i{0, 0, 0, 0};
      v4i ra[4][4], rb[4][4];
#pragma unroll
      for (int s = 0; s < 4; ++s)
#pragma unroll
        for (int i = 0; i < 4; ++i) { ra[s][i] = v4i{0, 0, 0, 0}; rb[s][i] = v4i{0, 0, 0, 0}; }

      unsigned tmo_val = 0;
      const __amdgpu_buffer_rsrc_t rws = mk_rsrc(my, OFF_FLG);
      for (unsigned ord = 0; ord < 8; ++ord) {
        const unsigned src = (rank + ord_dist(ord, nb)) & 7;
        const unsigned aoff = OFF_X + par * (NR * XBLK) + src * XBLK + mb * (8 * UNIT);
        const unsigned boff = (src == rank ? OFF_STG + par * STG : OFF_WG + par * WGB + src * WGS) + slot_of(nb, 0) * UNIT;
        const unsigned* const mword = &lds[ord >> 2];
        const unsigned sh = (ord & 3) * 8;
        unsigned ready = 0;
#pragma unroll
        for (int ks = 0; ks < 8; ++ks) {
          while (!((ready >> ks) & 1)) {
            ready = __builtin_amdgcn_readfirstlane(ld_lds(mword)) >> sh;
            if ((ready >> ks) & 1) break;
            if (GSLEEP) __builtin_amdgcn_s_sleep(GSLEEP);
          }
          asm volatile("" ::: "memory");
          if (ks == 7) STAMP(3 + ord);
          const int s = ks & 3;
#pragma unroll
          for (int i = 0; i < 4; ++i)
            rb[s][i] = (v4i)__builtin_amdgcn_raw_buffer_load_b128(rws, lane * 16 + i * 1024, boff + ks * (GCELLS * UNIT), 0);
#pragma unroll
          for (int i = 0; i < 4; ++i)
            ra[s][i] = (v4i)__builtin_amdgcn_raw_buffer_load_b128(rws, lane * 16 + i * 1024, aoff + ks * UNIT, 0);
          if (ks == 7) tmo_val = ld_agt(tmo);
          const int s3 = (ks + 1) & 3;
#pragma unroll
          for (int i = 0; i < 4; ++i)
#pragma unroll
            for (int j = 0; j < 4; ++j)
              acc[i][j] = __builtin_amdgcn_mfma_i32_16x16x64_i8(ra[s3][i], rb[s3][j], acc[i][j], 0, 0, 0);
        }
      }
#pragma unroll
      for (int s = 1; s < 4; ++s)
#pragma unroll
        for (int i = 0; i < 4; ++i)
#pragma unroll
          for (int j = 0; j < 4; ++j)
            acc[i][j] = __builtin_amdgcn_mfma_i32_16x16x64_i8(ra[s][i], rb[s][j], acc[i][j], 0, 0, 0);

      STAMP(11);
      const bool failed = ld_lds(&lds[2]) != 0;
      const float scale = (failed || tmo_val == seq) ? __builtin_nanf("") : scale0;
      const __amdgpu_buffer_rsrc_t ro = mk_rsrc(p.out, (unsigned)MS * NN * 4);
#pragma unroll
      for (int i = 0; i < 4; ++i)
#pragma unroll
        for (int r = 0; r < 4; ++r) {
          v4f v;
#pragma unroll
          for (int j = 0; j < 4; ++j) v[j] = (float)acc[i][j][r] * scale;
          const unsigned row = mb * 64 + i * 16 + q * 4 + r;
          const unsigned off = (row * NN + nb * 64 + c * 4) * 4;
          __builtin_amdgcn_raw_buffer_store_b128((v4u)v, ro, off, 0, OUT_AUX);
        }
    }
  } else if (wid >= 4) {
    const unsigned xcc = __builtin_amdgcn_s_getreg((3 << 11) | 20) & 7;
    const unsigned row = (wid - 4) * 16 + (b >> 4);
    const unsigned CLQ = 0xB2u;
    const unsigned same = (((CLQ >> rank) & 1) ? CLQ : (~CLQ & 0xffu)) & ~(1u << rank);
    const unsigned other = ~same & ~(1u << rank) & 0xffu;
    unsigned cand = AFFINE == 1 ? ((xcc & 4) ? other : same) : ((xcc & 4) ? same : other);
    unsigned li = (row + xcc) & 3;
    if (AFFINE == 0) { cand = 0xffu & ~(1u << rank); li = (row + xcc) & 7; }
    unsigned src0 = 8;
    {
      unsigned m = cand;
      for (unsigned i = 0; i < li && m; ++i) m &= m - 1;
      if (m) src0 = __builtin_ctz(m);
    }
    if (row < PL && src0 < 8) {
      unsigned* const qh0 = (unsigned*)(my + OFF_QCTR) + par * (8 * 32);
      const __amdgpu_buffer_rsrc_t rdst = mk_rsrc(my + OFF_X, OFF_FLG - OFF_X);
      const __amdgpu_buffer_rsrc_t rflag = mk_rsrc(my + OFF_FLG, OFF_SFLAG - OFF_FLG);
      unsigned vzero = 0, vone = 1;
      asm volatile("" : "+v"(vzero), "+v"(vone));
      bool failed = false;
      const unsigned lane16 = lane * 16;
      char* const qws = p.ws[src0];
      unsigned tk = 0;
      unsigned resp = 1u << src0;
      unsigned availv = 0;
      unsigned tried = 0;
      unsigned ksrc = 8, known = 0;
      bool planned = !PLAN;
      unsigned p_own = NT, p_ext = 0, p_xs = 0, p_es = 0;
      const unsigned* const snap = (const unsigned*)(my + OFF_SNAP) + (lane & 7);
      auto make_plan = [&](unsigned sv) {
        const unsigned s8 = lane & 7;
        if (__ballot(s8 == rank || sv >= seq * 1024) != ~0ull) return;
        const unsigned h = s8 == rank ? 0 : sv - seq * 1024;
        unsigned sum = h;
        sum += __shfl_xor((int)sum, 1); sum += __shfl_xor((int)sum, 2); sum += __shfl_xor((int)sum, 4);
        const unsigned a = sum / 7;
        const unsigned e = (s8 != rank && h + PLAN_DEAD < a) ? min(a - h - PLAN_DEAD, 56u) : 0;
        const unsigned sp = (s8 != rank && h > a) ? h - a : 0;
        unsigned pe = e, ps = sp;
#pragma unroll
        for (int o = 1; o < 8; o <<= 1) {
          const unsigned ue = __shfl_up((int)pe, o, 8), us = __shfl_up((int)ps, o, 8);
          if (s8 >= (unsigned)o) { pe += ue; ps += us; }
        }
        const unsigned etot = __shfl((int)pe, 7, 8);
        p_own = NT - e;
        p_es = pe - e;
        p_xs = ps - sp;
        p_ext = etot > p_xs ? min(etot - p_xs, sp) : 0;
        planned = true;
      };
      auto take_snapshot = [&]() {
        unsigned old = seq;
        if (lane == 0) old = __hip_atomic_fetch_max((unsigned*)(my + OFF_SOWN), seq, __ATOMIC_RELAXED, __HIP_MEMORY_SCOPE_AGENT);
        if (__builtin_amdgcn_readfirstlane(old) < seq) {
          const unsigned hv = ld_agt(qh0 + (lane & 7) * 32);
          __builtin_amdgcn_raw_buffer_store_b32(seq * 1024 + min(hv, NT), rflag,
                                                lane < 8 ? (OFF_SNAP - OFF_FLG) + lane * 4 : OOB, 0, AUX_AGT);
        }
        asm volatile("s_waitcnt vmcnt(0)" ::: "memory");
      };
      auto map = [&](unsigned qs, unsigned tt, unsigned& us, unsigned& ut) -> bool {
        us = qs; ut = tt;
        if (tt + 64 < NT) return true;
        for (unsigned spin = 0; !planned; ++spin) {
          make_plan(ld_agt(snap));
          if (planned) break;
          if (spin == 0) take_snapshot();
          if ((spin & 63) == 63 &&
              (ld_agt(tmo) == seq || __builtin_amdgcn_s_memrealtime() - t_start > DEADLINE)) {
            st_agt(tmo, seq);
            failed = true;
            return false;
          }
          __builtin_amdgcn_s_sleep(2);
        }
        if (tt < (unsigned)__builtin_amdgcn_readlane(p_own, qs)) return true;
        if (tt < NT || tt - NT >= (unsigned)__builtin_amdgcn_readlane(p_ext, qs)) return false;
        const unsigned g = __builtin_amdgcn_readlane(p_xs, qs) + (tt - NT);
        const unsigned m = (unsigned)__ballot(g >= p_es && g - p_es < NT - p_own) & 0xffu;
        if (!m) return false;
        us = __builtin_ctz(m);
        ut = __builtin_amdgcn_readlane(p_own, us) + (g - __builtin_amdgcn_readlane(p_es, us));
        return true;
      };

      auto draw = [&](unsigned qs) {
        unsigned* const qh = qh0 + qs * 32;
        if (lane == 0)
          asm volatile("global_atomic_add %0, %1, %2, %3 sc0" : "=v"(tk) : "v"(vzero), "v"(vone), "s"(qh) : "memory");
      };
      auto dense_ptr = [&](unsigned s) {
        return (const unsigned*)(qws + OFF_DENSE) + s * 512 + ((448 + 64 * ((s - rank) & 7) + lane) & 511);
      };
      auto prep = [&](unsigned s, unsigned tt) {
        if (s != src0 && tt >= NT - 64 && !((tried >> s) & 1)) {
          availv |= (ld_sys(dense_ptr(s)) >= seq ? 1u : 0u) << s;
          tried |= 1u << s;
        }
        const bool rel = s != src0 && tt >= NT - 64 &&
                         ((__builtin_amdgcn_readlane(availv, (tt - (NT - 64)) & 63) >> s) & 1);
        if (!rel && (s != ksrc || tt >= known)) {
          const unsigned sd = (s - rank) & 7;
          const unsigned* const sf = (const unsigned*)(my + OFF_SFLAG) + s * (128 * 32);
          const unsigned g0 = ((lane + 16 * sd) & 127) * 32, g1 = ((lane + 64 + 16 * sd) & 127) * 32;
          const unsigned* const sxf = (const unsigned*)(my + OFF_SXFLAG) + (s * 16 + (lane & 15)) * 32;
          ksrc = s;
          for (unsigned spin = 0;; ++spin) {
            const unsigned f0 = ld_sys(sf + g0), f1 = ld_sys(sf + g1), fx = ld_sys(sxf);
            const unsigned long long m0 = __ballot(f0 >= seq), m1 = __ballot(f1 >= seq);
            const bool xall = (__ballot(fx >= seq || (lane & 15) < NXG) & 0xffffull) == 0xffffull;
            const unsigned lead = ~m0 ? (unsigned)__builtin_ctzll(~m0) : 64 + (~m1 ? (unsigned)__builtin_ctzll(~m1) : 64);
            constexpr unsigned GG = GCELLS < 2 ? 2 : GCELLS;
            const unsigned kw = lead / (2 * GG) * (8 * GG);
            known = kw <= XPOS ? kw : (xall ? kw + NXU : XPOS);
            if (tt < known) break;
            if ((spin & 63) == 63 &&
                (ld_agt(tmo) == seq || __builtin_amdgcn_s_memrealtime() - t_start > DEADLINE)) {
              st_agt(tmo, seq);
              failed = true;
              break;
            }
            if (PSLEEP) __builtin_amdgcn_s_sleep(PSLEEP);
          }
          asm volatile("" ::: "memory");
        }
        char* const base = rel ? qws + OFF_WG + par * WGB + s * WGS : p.ws[s] + OFF_STG + par * STG;
        return mk_rsrc(base, rel ? WGS : STG);
      };

      unsigned ts = src0;
      for (unsigned hop = 0; !failed; ++hop) {
        if (hop > 0) {
          const unsigned s8 = lane & 7;
          const unsigned hv = ld_agt(qh0 + s8 * 32);
          const unsigned lim = p_ext ? NT + p_ext : p_own;
          const bool open = s8 != rank && (hv + HELP_MIN < lim || (((resp >> s8) & 1) && hv < lim));
          unsigned key = (open ? 0xfffu - (lim - hv) : 0xffffu) * 8 + s8;
          key = min(key, (unsigned)__shfl_xor((int)key, 1));
          key = min(key, (unsigned)__shfl_xor((int)key, 2));
          key = min(key, (unsigned)__shfl_xor((int)key, 4));
          key = __builtin_amdgcn_readfirstlane(key);
          if ((key >> 3) >= 0xfffu) break;
          ts = key & 7;
          resp |= 1u << ts;
        }
        draw(ts);
        asm volatile("s_waitcnt vmcnt(0)" : "+v"(tk) :: "memory");
        unsigned t = __builtin_amdgcn_readfirstlane(tk);
        unsigned us, ut;
        if (!map(ts, t, us, ut)) continue;
        Move mv = decode(ut, us, rank, par);
        v4u cur[4];
        {
          const __amdgpu_buffer_rsrc_t rsrc = prep(us, ut);
          if (failed) break;
          if (hop == 0) STAMP(2);
          draw(ts);
#pragma unroll
          for (int j = 0; j < 4; ++j)
            cur[j] = __builtin_amdgcn_raw_buffer_load_b128(rsrc, lane16 + j * 1024, mv.soff, AUX_SYS);
        }
        asm volatile("s_waitcnt vmcnt(4)" : "+v"(tk) :: "memory");
        __builtin_amdgcn_raw_buffer_store_b32(seq, rflag, OOB, 0, AUX_SYS);
        t = __builtin_amdgcn_readfirstlane(tk);
        if (ksrc == ts) {
          const __amdgpu_buffer_rsrc_t rsrc = mk_rsrc(p.ws[ts] + OFF_STG + par * STG, STG);
          while (t + SNAP_AT < NT && t < known) {
            const Move nx = decode(t, ts, rank, par);
#pragma unroll
            for (int j = 0; j < 4; ++j) __builtin_amdgcn_raw_buffer_store_b128(cur[j], rdst, mv.doff + lane16 + j * 1024, 0, AUX_SYS);
            draw(ts);
#pragma unroll
            for (int j = 0; j < 4; ++j)
              cur[j] = __builtin_amdgcn_raw_buffer_load_b128(rsrc, lane16 + j * 1024, nx.soff, AUX_SYS);
            asm volatile("s_waitcnt vmcnt(4)" : "+v"(tk) :: "memory");
            __builtin_amdgcn_raw_buffer_store_b32(seq, rflag, lane == 0 ? mv.foff : (lane == 1 ? mv.goff : OOB), 0, AUX_SYS);
            t = __builtin_amdgcn_readfirstlane(tk);
            mv = nx;
          }
        }
        while (map(ts, t, us, ut)) {
          const Move nx = decode(ut, us, rank, par);
          const __amdgpu_buffer_rsrc_t rsrc = prep(us, ut);
          if (failed) break;
          if (PLAN && t + SNAP_AT == NT) take_snapshot();
#pragma unroll
          for (int j = 0; j < 4; ++j) __builtin_amdgcn_raw_buffer_store_b128(cur[j], rdst, mv.doff + lane16 + j * 1024, 0, AUX_SYS);
          draw(ts);
          unsigned sv = 0, vv = 0, wantv = 8;
          const bool want_plan = !planned && t + 112 >= NT;
          if (want_plan) sv = ld_agt(snap);
          if (planned && ts == src0 && t + 2 * PL >= NT && __builtin_amdgcn_readlane(p_ext, src0) != 0) {
            const unsigned g = __builtin_amdgcn_readlane(p_xs, src0);
            const unsigned m = (unsigned)__ballot(g >= p_es && g - p_es < NT - p_own) & 0xffu & ~tried;
            if (m) { wantv = __builtin_ctz(m); vv = ld_sys(dense_ptr(wantv)); }
          }
#pragma unroll
          for (int j = 0; j < 4; ++j)
            cur[j] = __builtin_amdgcn_raw_buffer_load_b128(rsrc, lane16 + j * 1024, nx.soff, AUX_SYS);
          asm volatile("s_waitcnt vmcnt(4)" : "+v"(tk) :: "memory");
          __builtin_amdgcn_raw_buffer_store_b32(seq, rflag, lane == 0 ? mv.foff : (lane == 1 ? mv.goff : OOB), 0, AUX_SYS);
          t = __builtin_amdgcn_readfirstlane(tk);
          mv = nx;
          if (want_plan) make_plan(sv);
          if (wantv < 8) {
            availv |= (vv >= seq ? 1u : 0u) << wantv;
            tried |= 1u << wantv;
          }
        }
        if (failed) break;
#pragma unroll
        for (int j = 0; j < 4; ++j) __builtin_amdgcn_raw_buffer_store_b128(cur[j], rdst, mv.doff + lane16 + j * 1024, 0, AUX_SYS);
        asm volatile("s_waitcnt vmcnt(0)" ::: "memory");
        __builtin_amdgcn_raw_buffer_store_b32(seq, rflag, lane == 0 ? mv.foff : (lane == 1 ? mv.goff : OOB), 0, AUX_SYS);
        if (hop == 0) STAMP(4);
        STAMP(3);
      }
    }
  }
  if (wid >= 4) {
    if (lane == 0) st_agt(ctr, seq);
    if (b == 8 && wid == 5 && lane < 8) st_agt((unsigned*)(my + OFF_QCTR) + ((par ^ 1) * 8 + lane) * 32, 0);
  }
}

}

extern "C" void kernel_launch(int rank, int nranks, ncclComm_t comm,
                              hipStream_t stream,
                              void* const* d_in, const int* in_sizes, int n_in,
                              void* d_out, int out_size,
                              const int* in_dtypes, int out_dtype,
                              void* const* d_out_all,
                              void* const* d_ws_all, size_t ws_bytes) {
  (void)comm; (void)in_dtypes; (void)out_dtype; (void)d_out_all;
  if (nranks != NR || (unsigned)rank >= (unsigned)NR || n_in != 4 || ws_bytes < WS_USED ||
      in_sizes[0] != NR * MS * KS || in_sizes[1] != KS * NN || out_size != MS * NN) {
    fprintf(stderr, "unexpected problem geometry\n");
    return;
  }
  Params p;
  p.x = (const int8_t*)d_in[0];
  p.w = (const int8_t*)d_in[1];
  p.sx = (const float*)d_in[2];
  p.sw = (const float*)d_in[3];
  p.out = (float*)d_out;
  for (int i = 0; i < NR; ++i) p.ws[i] = (char*)d_ws_all[i];
  p.rank = rank;
  fused_kernel<<<GRID, BLOCK, 0, stream>>>(p);
  HIP_CHECK(hipGetLastError());
}
```

```cpp
#include <hip/hip_runtime.h>
#include <rccl/rccl.h>
#include <cstdio>
#include <cstdint>

#define HIP_CHECK(expr) do { hipError_t _e = (expr); if (_e != hipSuccess) { \
    fprintf(stderr, "HIP error %s at %s:%d\n", hipGetErrorString(_e), __FILE__, __LINE__); \
    return; } } while (0)

namespace {

typedef int v4i __attribute__((ext_vector_type(4)));
typedef unsigned v4u __attribute__((ext_vector_type(4)));
typedef float v4f __attribute__((ext_vector_type(4)));

constexpr int NR = 8;
constexpr int NN = 4096;
constexpr int MS = 512;
constexpr int KS = 512;
constexpr unsigned UNIT = 4096;
constexpr unsigned SHARD = 2u << 20;
constexpr unsigned XBLK = 256u << 10;

#define PL 14
#define NXG 8
#define XPOS 32
#define GCELLS 8
#define HELP_MIN 64
#define POLLW 2
#define AFFINE 1
#define GSLEEP 1
#define QSLEEP 1
#define PSLEEP 1
#define OUT_AUX 16
#define PLAN 1
#define SNAP_AT 128
#define PLAN_DEAD 4
#define LATE 0
#define CODEPF 1
#define LIGHT 1
#define SWAPROLE 0

constexpr unsigned STG = 4u << 20;
constexpr unsigned OFF_STG = 0;
constexpr unsigned OFF_X = 8u << 20;
constexpr unsigned OFF_WG = 12u << 20;
constexpr unsigned WGS = SHARD;
constexpr unsigned WGB = NR * WGS;
constexpr unsigned OFF_FLG = 44u << 20;
constexpr unsigned OFF_WFLAG = OFF_FLG;
constexpr unsigned OFF_XFLAG = OFF_WFLAG + 512 * 128;
constexpr unsigned OFF_DENSE = OFF_XFLAG + 64 * 128;
constexpr unsigned OFF_SNAP = OFF_DENSE + 8 * 512 * 4;
constexpr unsigned OFF_SOWN = OFF_SNAP + 128;
constexpr unsigned OFF_SFLAG = OFF_SOWN + 128;
constexpr unsigned OFF_SXFLAG = OFF_SFLAG + 1024 * 128;
constexpr unsigned OFF_CTR = OFF_SXFLAG + 128 * 128;
constexpr unsigned OFF_QCTR = OFF_CTR + 256 * 128;
constexpr unsigned OFF_TMO = OFF_QCTR + 16 * 128;
constexpr unsigned WS_USED = OFF_TMO + 128;
static_assert(OFF_WG + 2 * WGB <= OFF_FLG, "layout");

constexpr unsigned NXU = (16 - NXG) * 4;
constexpr unsigned NT = 512 + NXU;

constexpr int GRID = 256;
constexpr int BLOCK = 384;
constexpr unsigned long long DEADLINE = 100000000ull;

constexpr int AUX_SYS = 17;
constexpr int AUX_AGT = 16;
constexpr unsigned OOB = 0xfffffff0u;

#define STAMP(i) do {} while (0)

struct Params {
  const int8_t* x;
  const int8_t* w;
  char* my;
  int rank;
  int pad;
  const float* sx;
  const float* sw;
  float* out;
  void* pad2;
  char* ws[NR];
};

__device__ __forceinline__ __amdgpu_buffer_rsrc_t mk_rsrc(const void* p, unsigned bytes) {
  return __builtin_amdgcn_make_buffer_rsrc(const_cast<void*>(p), 0, bytes, 0x00020000);
}
__device__ __forceinline__ unsigned ld_sys(const unsigned* p) {
  return __hip_atomic_load(p, __ATOMIC_RELAXED, __HIP_MEMORY_SCOPE_SYSTEM);
}
__device__ __forceinline__ void st_sys(unsigned* p, unsigned v) {
  __hip_atomic_store(p, v, __ATOMIC_RELAXED, __HIP_MEMORY_SCOPE_SYSTEM);
}
__device__ __forceinline__ unsigned ld_agt(const unsigned* p) {
  return __hip_atomic_load(p, __ATOMIC_RELAXED, __HIP_MEMORY_SCOPE_AGENT);
}
__device__ __forceinline__ void st_agt(unsigned* p, unsigned v) {
  __hip_atomic_store(p, v, __ATOMIC_RELAXED, __HIP_MEMORY_SCOPE_AGENT);
}
__device__ __forceinline__ unsigned ld_lds(const unsigned* p) {
  return __hip_atomic_load(p, __ATOMIC_RELAXED, __HIP_MEMORY_SCOPE_WORKGROUP);
}
__device__ __forceinline__ void st_lds(unsigned* p, unsigned v) {
  __hip_atomic_store(p, v, __ATOMIC_RELAXED, __HIP_MEMORY_SCOPE_WORKGROUP);
}

__device__ __forceinline__ unsigned slot_of(unsigned nb, unsigned ks) {
  return (nb / GCELLS) * (8 * GCELLS) + ks * GCELLS + nb % GCELLS;
}
__device__ __forceinline__ unsigned ord_dist(unsigned ord, unsigned nb) {
  const unsigned qn = nb >> 3;
  unsigned dd = (qn + 9 - ord) & 7;
  if (ord - 1 >= qn) dd = (dd + 7) & 7;
  return ord == 0 ? 0 : dd;
}

struct Move { unsigned soff, doff, foff, goff; };
__device__ __forceinline__ Move decode(unsigned t, unsigned src, unsigned rank, unsigned par) {
  const unsigned sd = (src - rank) & 7;
  const bool isx = t - XPOS < NXU;
  const unsigned xu = NXG * 4 + (t - XPOS);
  const unsigned tw = t < XPOS ? t : t - NXU;
  const unsigned slot = (tw + 64 * sd) & 511;
  const unsigned r = slot % (8 * GCELLS);
  const unsigned nb = (slot / (8 * GCELLS)) * GCELLS + r % GCELLS, ks = r / GCELLS;
  Move m;
  m.soff = (isx ? 512 + rank * 64 + xu : slot) * UNIT;
  m.doff = isx ? par * (NR * XBLK) + (src * 64 + xu) * UNIT : (OFF_WG - OFF_X) + par * WGB + src * WGS + slot * UNIT;
  m.foff = isx ? (OFF_XFLAG - OFF_FLG) + (((xu >> 3) * 8 + src) * 32 + (xu & 7)) * 4
               : (OFF_WFLAG - OFF_FLG) + ((nb * 8 + src) * 32 + ks) * 4;
  m.goff = isx ? OOB : (OFF_DENSE - OFF_FLG) + (src * 512 + slot) * 4;
  return m;
}

__global__ void __launch_bounds__(BLOCK, 2) fused_kernel(Params p) {
  __shared__ unsigned lds[21 * 1024];
  const unsigned b = blockIdx.x;
  const unsigned lane = threadIdx.x & 63;
  const unsigned wid = __builtin_amdgcn_readfirstlane(threadIdx.x >> 6);
  asm volatile("" :: "s"(p.x), "s"(p.w), "s"(p.my), "s"(p.rank), "s"(p.ws[0]));
  const unsigned rank = p.rank;
  char* const my = p.my;
  const unsigned sidx = (b >> 3) * 4 + (b & 3);
  const unsigned role = (b >> 3) & 1;
  const unsigned idx = ((b >> 4) << 3) | (b & 7);
  const unsigned q = lane >> 4, c = lane & 15;

  unsigned* const ctr = (unsigned*)(my + OFF_CTR) + b * 32 + wid;
  unsigned* const tmo = (unsigned*)(my + OFF_TMO);
  const unsigned long long t_start = __builtin_amdgcn_s_memrealtime();
  unsigned seq;

  STAMP(0);
  if (wid >= 4) {
    if (wid == 4 && lane < 4) st_lds(&lds[lane], 0);
    asm volatile("s_waitcnt lgkmcnt(0)" ::: "memory");
    __builtin_amdgcn_s_barrier();
    asm volatile("" ::: "memory");
    if (CODEPF && wid == 5 && b < 8) {
      unsigned long long pc;
      int la, lb;
      asm volatile("s_getpc_b64 %0\n.Lpf_here:\n\ts_mov_b32 %1, .Lpf_gemm_end-.Lpf_here\n\ts_mov_b32 %2, .Lpf_pull_end-.Lpf_here"
                   : "=s"(pc), "=s"(la), "=s"(lb));
      const unsigned len = (unsigned)max(max(la, lb), 0);
      if (len < 32768u) {
        const __amdgpu_buffer_rsrc_t rc = mk_rsrc((const void*)pc, len & ~15u);
        v4u acc = {0, 0, 0, 0};
#pragma unroll
        for (int i = 0; i < 4; ++i) acc |= __builtin_amdgcn_raw_buffer_load_b128(rc, (lane + i * 64) * 128, 0, 0);
        asm volatile("" :: "v"(acc));
      }
    }
    seq = __builtin_amdgcn_readfirstlane(ld_agt(ctr)) + 1;
  } else if (((b >> 2) & 1) == SWAPROLE) {
    const unsigned dd = (((sidx >> 2) & 7) + 1) & 7;
    const unsigned dst = (rank + dd) & 7;
    const unsigned xg = (sidx >> 5) * 4 + (sidx & 3);
    if (LATE > 0 && (xg >= NXG || dd == 0)) __builtin_amdgcn_s_sleep(LATE);
    v4u v[4];
    const int8_t* src = p.x + (size_t)(dst * MS + (xg >> 1) * 64 + c) * KS + (xg & 1) * 256 + wid * 64 + q * 16;
#pragma unroll
    for (int i = 0; i < 4; ++i) v[i] = *(const v4u*)(src + i * 16 * KS);
    const unsigned cv = ld_agt(ctr);
    char* const dws = p.ws[dst];
    __builtin_amdgcn_s_barrier();
    seq = __builtin_amdgcn_readfirstlane(cv) + 1;
    const bool push = xg < NXG || dd == 0;
    char* const base = push ? dws + OFF_X + (seq & 1) * (NR * XBLK) + rank * XBLK
                            : my + OFF_STG + (seq & 1) * STG + SHARD + dst * XBLK;
    const __amdgpu_buffer_rsrc_t rs = mk_rsrc(base, XBLK);
    const unsigned off = (xg * 4 + wid) * UNIT + lane * 16;
#pragma unroll
    for (int i = 0; i < 4; ++i) __builtin_amdgcn_raw_buffer_store_b128(v[i], rs, off + i * 1024, 0, AUX_SYS);
    asm volatile("s_waitcnt vmcnt(0)" ::: "memory");
    STAMP(1);
    unsigned old = 0;
    if (lane == 0) old = __hip_atomic_fetch_add(&lds[3], 1u, __ATOMIC_RELAXED, __HIP_MEMORY_SCOPE_WORKGROUP);
    old = __builtin_amdgcn_readfirstlane(old);
    if (old == 3) {
      const unsigned fo = push ? OFF_XFLAG + (((xg >> 1) * 8 + rank) * 32 + (xg & 1) * 4) * 4
                               : OFF_SXFLAG + (rank * 16 + xg) * 128;
      if (lane < 4) st_sys((unsigned*)(dws + fo) + lane, seq);
    }
    if (lane == 0) st_agt(ctr, seq);
  } else {
    const unsigned sslot = sidx * 4 + wid;
    const unsigned snb = (sslot / (8 * GCELLS)) * GCELLS + sslot % GCELLS, sks = (sslot % (8 * GCELLS)) / GCELLS;
    if (LATE > 0 && (sidx & 15) >= 8) __builtin_amdgcn_s_sleep(LATE);
    char* const peer_ws = p.ws[(rank + lane) & (NR - 1)];
    unsigned d[16];
    {
      const unsigned* src = (const unsigned*)(p.w + (size_t)(sks * 64 + q * 16) * NN + snb * 64 + c * 4);
#pragma unroll
      for (int r = 0; r < 16; ++r) d[r] = src[r * (NN / 4)];
    }
    const unsigned cv = ld_agt(ctr);
    __builtin_amdgcn_s_barrier();
    seq = __builtin_amdgcn_readfirstlane(cv) + 1;
    v4u o[4];
#pragma unroll
    for (int g = 0; g < 4; ++g) {
      const unsigned t0 = __builtin_amdgcn_perm(d[4 * g + 1], d[4 * g + 0], 0x05010400u);
      const unsigned t1 = __builtin_amdgcn_perm(d[4 * g + 1], d[4 * g + 0], 0x07030602u);
      const unsigned t2 = __builtin_amdgcn_perm(d[4 * g + 3], d[4 * g + 2], 0x05010400u);
      const unsigned t3 = __builtin_amdgcn_perm(d[4 * g + 3], d[4 * g + 2], 0x07030602u);
      o[0][g] = __builtin_amdgcn_perm(t2, t0, 0x05040100u);
      o[1][g] = __builtin_amdgcn_perm(t2, t0, 0x07060302u);
      o[2][g] = __builtin_amdgcn_perm(t3, t1, 0x05040100u);
      o[3][g] = __builtin_amdgcn_perm(t3, t1, 0x07060302u);
    }
    const __amdgpu_buffer_rsrc_t rs = mk_rsrc(my + OFF_STG + (seq & 1) * STG, SHARD);
    const unsigned off = sslot * UNIT + lane * 16;
#pragma unroll
    for (int j = 0; j < 4; ++j) __builtin_amdgcn_raw_buffer_store_b128(o[j], rs, off + j * 1024, 0, AUX_SYS);
    asm volatile("s_waitcnt vmcnt(0)" ::: "memory");
    STAMP(1);
    unsigned old = 0;
    if (lane == 0) old = __hip_atomic_fetch_add(&lds[3], 1u, __ATOMIC_RELAXED, __HIP_MEMORY_SCOPE_WORKGROUP);
    old = __builtin_amdgcn_readfirstlane(old);
    if (old == 3 && lane >= 1 && lane < NR)
      st_sys((unsigned*)(peer_ws + OFF_SFLAG) + (rank * 128 + sidx) * 32, seq);
    if (lane == 0) {
      st_agt((unsigned*)(my + OFF_WFLAG) + (snb * 8 + rank) * 32 + sks, seq);
      st_agt(ctr, seq);
    }
  }
  seq = __builtin_amdgcn_readfirstlane(seq);
  const unsigned par = seq & 1;

  if (role == 0) {
    const unsigned sub = idx >> 3;
    const unsigned nb = (sub >> 1) * 8 + (idx & 7);
    const unsigned mb0 = (sub & 1) * 4;
    if (wid == 4) {
      const unsigned psrc = (rank + ord_dist(lane >> 3, nb)) & 7;
      const unsigned* const wf = (const unsigned*)(my + OFF_WFLAG) + (nb * 8 + psrc) * 32 + (lane & 7);
      const __amdgpu_buffer_rsrc_t rxf = mk_rsrc(my + OFF_XFLAG, 64 * 128);
      const unsigned xfo = (((mb0 + (lane & 3)) * 8 + psrc) * 32 + ((lane >> 2) & 1) * 4) * 4;
      bool wok = false, xok = false;
      unsigned long long last = 0;
      for (unsigned spin = 0;; ++spin) {
        const unsigned front = (unsigned)__builtin_ctzll(~last) >> 3;
        const bool near = (lane >> 3) < front + POLLW;
        if (!wok && near) wok = ld_agt(wf) >= seq;
        if (!xok && near) {
          const v4u f = __builtin_amdgcn_raw_buffer_load_b128(rxf, xfo, 0, AUX_SYS);
          xok = f[0] >= seq && f[1] >= seq && f[2] >= seq && f[3] >= seq;
        }
        asm volatile("" ::: "memory");
        const unsigned long long xm = __ballot(xok);
        const unsigned long long m = __ballot(wok && ((xm >> (lane & 56)) & 0xff) == 0xff);
        if (m != last) {
          last = m;
          if (lane < 2) st_lds(&lds[lane], lane ? (unsigned)(m >> 32) : (unsigned)m);
          if (m == ~0ull) break;
        }
        if ((spin & 63) == 63 &&
            (ld_agt(tmo) == seq || __builtin_amdgcn_s_memrealtime() - t_start > DEADLINE)) {
          st_agt(tmo, seq);
          if (lane == 0) st_lds(&lds[2], 1);
          asm volatile("s_waitcnt lgkmcnt(0)" ::: "memory");
          if (lane < 2) st_lds(&lds[lane], 0xffffffffu);
          break;
        }
        if (QSLEEP) __builtin_amdgcn_s_sleep(QSLEEP);
      }
    } else if (wid < 4) {
      const unsigned mb = mb0 + wid;
      const float scale0 = p.sx[0] * p.sw[0];
      v4i acc[4][4];
#pragma unroll
      for (int i = 0; i < 4; ++i)
#pragma unroll
        for (int j = 0; j < 4; ++j) acc[i][j] = v4i{0, 0, 0, 0};
      v4i ra[4][4], rb[4][4];
#pragma unroll
      for (int s = 0; s < 4; ++s)
#pragma unroll
        for (int i = 0; i < 4; ++i) { ra[s][i] = v4i{0, 0, 0, 0}; rb[s][i] = v4i{0, 0, 0, 0}; }

      unsigned tmo_val = 0;
      const __amdgpu_buffer_rsrc_t rws = mk_rsrc(my, OFF_FLG);
      for (unsigned ord = 0; ord < 8; ++ord) {
        const unsigned src = (rank + ord_dist(ord, nb)) & 7;
        const unsigned aoff = OFF_X + par * (NR * XBLK) + src * XBLK + mb * (8 * UNIT);
        const unsigned boff = (src == rank ? OFF_STG + par * STG : OFF_WG + par * WGB + src * WGS) + slot_of(nb, 0) * UNIT;
        const unsigned* const mword = &lds[ord >> 2];
        const unsigned sh = (ord & 3) * 8;
        unsigned ready = 0;
#pragma unroll
        for (int ks = 0; ks < 8; ++ks) {
          while (!((ready >> ks) & 1)) {
            ready = __builtin_amdgcn_readfirstlane(ld_lds(mword)) >> sh;
            if ((ready >> ks) & 1) break;
            if (GSLEEP) __builtin_amdgcn_s_sleep(GSLEEP);
          }
          asm volatile("" ::: "memory");
          if (ks == 7) STAMP(3 + ord);
          const int s = ks & 3;
#pragma unroll
          for (int i = 0; i < 4; ++i)
            rb[s][i] = (v4i)__builtin_amdgcn_raw_buffer_load_b128(rws, lane * 16 + i * 1024, boff + ks * (GCELLS * UNIT), 0);
#pragma unroll
          for (int i = 0; i < 4; ++i)
            ra[s][i] = (v4i)__builtin_amdgcn_raw_buffer_load_b128(rws, lane * 16 + i * 1024, aoff + ks * UNIT, 0);
          if (ks == 7) tmo_val = ld_agt(tmo);
          const int s3 = (ks + 1) & 3;
#pragma unroll
          for (int i = 0; i < 4; ++i)
#pragma unroll
            for (int j = 0; j < 4; ++j)
              acc[i][j] = __builtin_amdgcn_mfma_i32_16x16x64_i8(ra[s3][i], rb[s3][j], acc[i][j], 0, 0, 0);
        }
      }
#pragma unroll
      for (int s = 1; s < 4; ++s)
#pragma unroll
        for (int i = 0; i < 4; ++i)
#pragma unroll
          for (int j = 0; j < 4; ++j)
            acc[i][j] = __builtin_amdgcn_mfma_i32_16x16x64_i8(ra[s][i], rb[s][j], acc[i][j], 0, 0, 0);

      STAMP(11);
      const bool failed = ld_lds(&lds[2]) != 0;
      const float scale = (failed || tmo_val == seq) ? __builtin_nanf("") : scale0;
      const __amdgpu_buffer_rsrc_t ro = mk_rsrc(p.out, (unsigned)MS * NN * 4);
#pragma unroll
      for (int i = 0; i < 4; ++i)
#pragma unroll
        for (int r = 0; r < 4; ++r) {
          v4f v;
#pragma unroll
          for (int j = 0; j < 4; ++j) v[j] = (float)acc[i][j][r] * scale;
          const unsigned row = mb * 64 + i * 16 + q * 4 + r;
          const unsigned off = (row * NN + nb * 64 + c * 4) * 4;
          __builtin_amdgcn_raw_buffer_store_b128((v4u)v, ro, off, 0, OUT_AUX);
        }
      asm volatile(".Lpf_gemm_end:");
    }
  } else if (wid >= 4) {
    const unsigned xcc = __builtin_amdgcn_s_getreg((3 << 11) | 20) & 7;
    const unsigned row = (wid - 4) * 16 + (b >> 4);
    const unsigned CLQ = 0xB2u;
    const unsigned same = (((CLQ >> rank) & 1) ? CLQ : (~CLQ & 0xffu)) & ~(1u << rank);
    const unsigned other = ~same & ~(1u << rank) & 0xffu;
    unsigned cand = AFFINE == 1 ? ((xcc & 4) ? other : same) : ((xcc & 4) ? same : other);
    unsigned li = (row + xcc) & 3;
    if (AFFINE == 0) { cand = 0xffu & ~(1u << rank); li = (row + xcc) & 7; }
    unsigned src0 = 8;
    {
      unsigned m = cand;
      for (unsigned i = 0; i < li && m; ++i) m &= m - 1;
      if (m) src0 = __builtin_ctz(m);
    }
    if (row < PL && src0 < 8) {
      unsigned* const qh0 = (unsigned*)(my + OFF_QCTR) + par * (8 * 32);
      const __amdgpu_buffer_rsrc_t rdst = mk_rsrc(my + OFF_X, OFF_FLG - OFF_X);
      const __amdgpu_buffer_rsrc_t rflag = mk_rsrc(my + OFF_FLG, OFF_SFLAG - OFF_FLG);
      unsigned vzero = 0, vone = 1;
      asm volatile("" : "+v"(vzero), "+v"(vone));
      bool failed = false;
      const unsigned lane16 = lane * 16;
      char* const qws = p.ws[src0];
      unsigned tk = 0;
      unsigned resp = 1u << src0;
      unsigned availv = 0;
      unsigned tried = 0;
      unsigned ksrc = 8, known = 0;
      bool planned = !PLAN;
      unsigned p_own = NT, p_ext = 0, p_xs = 0, p_es = 0;
      const unsigned* const snap = (const unsigned*)(my + OFF_SNAP) + (lane & 7);
      auto make_plan = [&](unsigned sv) {
        const unsigned s8 = lane & 7;
        if (__ballot(s8 == rank || sv >= seq * 1024) != ~0ull) return;
        const unsigned h = s8 == rank ? 0 : sv - seq * 1024;
        unsigned sum = h;
        sum += __shfl_xor((int)sum, 1); sum += __shfl_xor((int)sum, 2); sum += __shfl_xor((int)sum, 4);
        const unsigned a = sum / 7;
        const unsigned e = (s8 != rank && h + PLAN_DEAD < a) ? min(a - h - PLAN_DEAD, 56u) : 0;
        const unsigned sp = (s8 != rank && h > a) ? h - a : 0;
        unsigned pe = e, ps = sp;
#pragma unroll
        for (int o = 1; o < 8; o <<= 1) {
          const unsigned ue = __shfl_up((int)pe, o, 8), us = __shfl_up((int)ps, o, 8);
          if (s8 >= (unsigned)o) { pe += ue; ps += us; }
        }
        const unsigned etot = __shfl((int)pe, 7, 8);
        p_own = NT - e;
        p_es = pe - e;
        p_xs = ps - sp;
        p_ext = etot > p_xs ? min(etot - p_xs, sp) : 0;
        planned = true;
      };
      auto take_snapshot = [&]() {
        unsigned old = seq;
        if (lane == 0) old = __hip_atomic_fetch_max((unsigned*)(my + OFF_SOWN), seq, __ATOMIC_RELAXED, __HIP_MEMORY_SCOPE_AGENT);
        if (__builtin_amdgcn_readfirstlane(old) < seq) {
          const unsigned hv = ld_agt(qh0 + (lane & 7) * 32);
          __builtin_amdgcn_raw_buffer_store_b32(seq * 1024 + min(hv, NT), rflag,
                                                lane < 8 ? (OFF_SNAP - OFF_FLG) + lane * 4 : OOB, 0, AUX_AGT);
        }
        asm volatile("s_waitcnt vmcnt(0)" ::: "memory");
      };
      auto map = [&](unsigned qs, unsigned tt, unsigned& us, unsigned& ut) -> bool {
        us = qs; ut = tt;
        if (tt + 64 < NT) return true;
        for (unsigned spin = 0; !planned; ++spin) {
          make_plan(ld_agt(snap));
          if (planned) break;
          if (spin == 0) take_snapshot();
          if ((spin & 63) == 63 &&
              (ld_agt(tmo) == seq || __builtin_amdgcn_s_memrealtime() - t_start > DEADLINE)) {
            st_agt(tmo, seq);
            failed = true;
            return false;
          }
          __builtin_amdgcn_s_sleep(2);
        }
        if (tt < (unsigned)__builtin_amdgcn_readlane(p_own, qs)) return true;
        if (tt < NT || tt - NT >= (unsigned)__builtin_amdgcn_readlane(p_ext, qs)) return false;
        const unsigned g = __builtin_amdgcn_readlane(p_xs, qs) + (tt - NT);
        const unsigned m = (unsigned)__ballot(g >= p_es && g - p_es < NT - p_own) & 0xffu;
        if (!m) return false;
        us = __builtin_ctz(m);
        ut = __builtin_amdgcn_readlane(p_own, us) + (g - __builtin_amdgcn_readlane(p_es, us));
        return true;
      };

      auto draw = [&](unsigned qs) {
        unsigned* const qh = qh0 + qs * 32;
        if (lane == 0)
          asm volatile("global_atomic_add %0, %1, %2, %3 sc0" : "=v"(tk) : "v"(vzero), "v"(vone), "s"(qh) : "memory");
      };
      auto dense_ptr = [&](unsigned s) {
        return (const unsigned*)(qws + OFF_DENSE) + s * 512 + ((448 + 64 * ((s - rank) & 7) + lane) & 511);
      };
      auto prep = [&](unsigned s, unsigned tt) {
        if (s != src0 && tt >= NT - 64 && !((tried >> s) & 1)) {
          availv |= (ld_sys(dense_ptr(s)) >= seq ? 1u : 0u) << s;
          tried |= 1u << s;
        }
        const bool rel = s != src0 && tt >= NT - 64 &&
                         ((__builtin_amdgcn_readlane(availv, (tt - (NT - 64)) & 63) >> s) & 1);
        if (!rel && (s != ksrc || tt >= known)) {
          const unsigned sd = (s - rank) & 7;
          const unsigned* const sf = (const unsigned*)(my + OFF_SFLAG) + s * (128 * 32);
          const unsigned g0 = ((lane + 16 * sd) & 127) * 32, g1 = ((lane + 64 + 16 * sd) & 127) * 32;
          const unsigned* const sxf = (const unsigned*)(my + OFF_SXFLAG) + (s * 16 + (lane & 15)) * 32;
          ksrc = s;
          for (unsigned spin = 0;; ++spin) {
            const unsigned f0 = ld_sys(sf + g0), f1 = ld_sys(sf + g1), fx = ld_sys(sxf);
            const unsigned long long m0 = __ballot(f0 >= seq), m1 = __ballot(f1 >= seq);
            const bool xall = (__ballot(fx >= seq || (lane & 15) < NXG) & 0xffffull) == 0xffffull;
            const unsigned lead = ~m0 ? (unsigned)__builtin_ctzll(~m0) : 64 + (~m1 ? (unsigned)__builtin_ctzll(~m1) : 64);
            const unsigned kw = lead * 4;
            known = kw <= XPOS ? kw : (xall ? kw + NXU : XPOS);
            if (tt < known) break;
            if ((spin & 63) == 63 &&
                (ld_agt(tmo) == seq || __builtin_amdgcn_s_memrealtime() - t_start > DEADLINE)) {
              st_agt(tmo, seq);
              failed = true;
              break;
            }
            if (PSLEEP) __builtin_amdgcn_s_sleep(PSLEEP);
          }
          asm volatile("" ::: "memory");
        }
        char* const base = rel ? qws + OFF_WG + par * WGB + s * WGS : p.ws[s] + OFF_STG + par * STG;
        return mk_rsrc(base, rel ? WGS : STG);
      };

      unsigned ts = src0;
      for (unsigned hop = 0; !failed; ++hop) {
        if (hop > 0) {
          const unsigned s8 = lane & 7;
          const unsigned hv = ld_agt(qh0 + s8 * 32);
          const unsigned lim = p_ext ? NT + p_ext : p_own;
          const bool open = s8 != rank && (hv + HELP_MIN < lim || (((resp >> s8) & 1) && hv < lim));
          unsigned key = (open ? 0xfffu - (lim - hv) : 0xffffu) * 8 + s8;
          key = min(key, (unsigned)__shfl_xor((int)key, 1));
          key = min(key, (unsigned)__shfl_xor((int)key, 2));
          key = min(key, (unsigned)__shfl_xor((int)key, 4));
          key = __builtin_amdgcn_readfirstlane(key);
          if ((key >> 3) >= 0xfffu) break;
          ts = key & 7;
          resp |= 1u << ts;
        }
        draw(ts);
        asm volatile("s_waitcnt vmcnt(0)" : "+v"(tk) :: "memory");
        unsigned t = __builtin_amdgcn_readfirstlane(tk);
        unsigned us, ut;
        if (!map(ts, t, us, ut)) continue;
        Move mv = decode(ut, us, rank, par);
        v4u cur[4];
        if (LIGHT && hop == 0 && ut < XPOS) {
          const unsigned* const f1 = (const unsigned*)(my + OFF_SFLAG) + (us * 128 + (((ut >> 2) + 16 * ((us - rank) & 7)) & 127)) * 32;
          for (unsigned spin = 0; spin < 4096; ++spin)
            if (__builtin_amdgcn_readfirstlane(ld_sys(f1)) >= seq) { ksrc = us; known = ut + 1; break; }
          asm volatile("" ::: "memory");
        }
        {
          const __amdgpu_buffer_rsrc_t rsrc = prep(us, ut);
          if (failed) break;
          if (hop == 0) STAMP(2);
          draw(ts);
#pragma unroll
          for (int j = 0; j < 4; ++j)
            cur[j] = __builtin_amdgcn_raw_buffer_load_b128(rsrc, lane16 + j * 1024, mv.soff, AUX_SYS);
        }
        asm volatile("s_waitcnt vmcnt(4)" : "+v"(tk) :: "memory");
        __builtin_amdgcn_raw_buffer_store_b32(seq, rflag, OOB, 0, AUX_SYS);
        t = __builtin_amdgcn_readfirstlane(tk);
        while (t + SNAP_AT < NT) {
          const __amdgpu_buffer_rsrc_t rsrc = prep(ts, t);
          if (failed) break;
          while (t + SNAP_AT < NT && t < known) {
            const Move nx = decode(t, ts, rank, par);
#pragma unroll
            for (int j = 0; j < 4; ++j) __builtin_amdgcn_raw_buffer_store_b128(cur[j], rdst, mv.doff + lane16 + j * 1024, 0, AUX_SYS);
            draw(ts);
#pragma unroll
            for (int j = 0; j < 4; ++j)
              cur[j] = __builtin_amdgcn_raw_buffer_load_b128(rsrc, lane16 + j * 1024, nx.soff, AUX_SYS);
            asm volatile("s_waitcnt vmcnt(4)" : "+v"(tk) :: "memory");
            __builtin_amdgcn_raw_buffer_store_b32(seq, rflag, lane == 0 ? mv.foff : (lane == 1 ? mv.goff : OOB), 0, AUX_SYS);
            t = __builtin_amdgcn_readfirstlane(tk);
            mv = nx;
          }
        }
        if (failed) break;
        while (map(ts, t, us, ut)) {
          const Move nx = decode(ut, us, rank, par);
          const __amdgpu_buffer_rsrc_t rsrc = prep(us, ut);
          if (failed) break;
          if (PLAN && t + SNAP_AT == NT) take_snapshot();
#pragma unroll
          for (int j = 0; j < 4; ++j) __builtin_amdgcn_raw_buffer_store_b128(cur[j], rdst, mv.doff + lane16 + j * 1024, 0, AUX_SYS);
          draw(ts);
          unsigned sv = 0, vv = 0, wantv = 8;
          const bool want_plan = !planned && t + 112 >= NT;
          if (want_plan) sv = ld_agt(snap);
          if (planned && ts == src0 && t + 2 * PL >= NT && __builtin_amdgcn_readlane(p_ext, src0) != 0) {
            const unsigned g = __builtin_amdgcn_readlane(p_xs, src0);
            const unsigned m = (unsigned)__ballot(g >= p_es && g - p_es < NT - p_own) & 0xffu & ~tried;
            if (m) { wantv = __builtin_ctz(m); vv = ld_sys(dense_ptr(wantv)); }
          }
#pragma unroll
          for (int j = 0; j < 4; ++j)
            cur[j] = __builtin_amdgcn_raw_buffer_load_b128(rsrc, lane16 + j * 1024, nx.soff, AUX_SYS);
          asm volatile("s_waitcnt vmcnt(4)" : "+v"(tk) :: "memory");
          __builtin_amdgcn_raw_buffer_store_b32(seq, rflag, lane == 0 ? mv.foff : (lane == 1 ? mv.goff : OOB), 0, AUX_SYS);
          t = __builtin_amdgcn_readfirstlane(tk);
          mv = nx;
          if (want_plan) make_plan(sv);
          if (wantv < 8) {
            availv |= (vv >= seq ? 1u : 0u) << wantv;
            tried |= 1u << wantv;
          }
        }
        if (failed) break;
#pragma unroll
        for (int j = 0; j < 4; ++j) __builtin_amdgcn_raw_buffer_store_b128(cur[j], rdst, mv.doff + lane16 + j * 1024, 0, AUX_SYS);
        asm volatile("s_waitcnt vmcnt(0)" ::: "memory");
        __builtin_amdgcn_raw_buffer_store_b32(seq, rflag, lane == 0 ? mv.foff : (lane == 1 ? mv.goff : OOB), 0, AUX_SYS);
        asm volatile(".Lpf_pull_end:");
        if (hop == 0) STAMP(4);
        STAMP(3);
      }
    }
  }
  if (wid >= 4) {
    if (lane == 0) st_agt(ctr, seq);
    if (b == 8 && wid == 5 && lane < 8) st_agt((unsigned*)(my + OFF_QCTR) + ((par ^ 1) * 8 + lane) * 32, 0);
  }
}

}

extern "C" void kernel_launch(int rank, int nranks, ncclComm_t comm,
                              hipStream_t stream,
                              void* const* d_in, const int* in_sizes, int n_in,
                              void* d_out, int out_size,
                              const int* in_dtypes, int out_dtype,
                              void* const* d_out_all,
                              void* const* d_ws_all, size_t ws_bytes) {
  (void)comm; (void)in_dtypes; (void)out_dtype; (void)d_out_all;
  if (nranks != NR || (unsigned)rank >= (unsigned)NR || n_in != 4 || ws_bytes < WS_USED ||
      in_sizes[0] != NR * MS * KS || in_sizes[1] != KS * NN || out_size != MS * NN) {
    fprintf(stderr, "unexpected problem geometry\n");
    return;
  }
  Params p;
  p.x = (const int8_t*)d_in[0];
  p.w = (const int8_t*)d_in[1];
  p.sx = (const float*)d_in[2];
  p.sw = (const float*)d_in[3];
  p.out = (float*)d_out;
  for (int i = 0; i < NR; ++i) p.ws[i] = (char*)d_ws_all[i];
  p.rank = rank;
  p.my = p.ws[rank];
  p.pad = 0;
  p.pad2 = nullptr;
  fused_kernel<<<GRID, BLOCK, 0, stream>>>(p);
  HIP_CHECK(hipGetLastError());
}
```

```cpp
#include <hip/hip_runtime.h>
#include <rccl/rccl.h>
#include <cstdio>
#include <cstdint>

#define HIP_CHECK(expr) do { hipError_t _e = (expr); if (_e != hipSuccess) { \
    fprintf(stderr, "HIP error %s at %s:%d\n", hipGetErrorString(_e), __FILE__, __LINE__); \
    return; } } while (0)

namespace {

typedef int v4i __attribute__((ext_vector_type(4)));
typedef unsigned v4u __attribute__((ext_vector_type(4)));
typedef float v4f __attribute__((ext_vector_type(4)));

constexpr int NR = 8;
constexpr int NN = 4096;
constexpr int MS = 512;
constexpr int KS = 512;
constexpr unsigned UNIT = 4096;
constexpr unsigned SHARD = 2u << 20;
constexpr unsigned XBLK = 256u << 10;

#define PL 14
#define NXG 16
#define XPOS 32
#define GCELLS 8
#define HELP_MIN 64
#define POLLW 2
#define AFFINE 1
#define GSLEEP 1
#define QSLEEP 1
#define PSLEEP 1
#define OUT_AUX 16
#define PLAN 1
#define SNAP_AT 128
#define PLAN_DEAD 4
#define LATE 0
#define CODEPF 1
#define LIGHT 1
#define LD_AUX 17
#define ST_AUX 17
#define APRE 0
#define ROLEBIT 2
#define STAG 0
#define SPLIT 0
#define PCLEAN 0
#define THROT 1
#define PWIN 3
#define WDELAY 48
#define XSPLIT 6
#define SWAPROLE 0

constexpr unsigned STG = 4u << 20;
constexpr unsigned OFF_STG = 0;
constexpr unsigned OFF_X = 8u << 20;
constexpr unsigned OFF_WG = 12u << 20;
constexpr unsigned WGS = SHARD;
constexpr unsigned WGB = NR * WGS;
constexpr unsigned OFF_FLG = 44u << 20;
constexpr unsigned OFF_WFLAG = OFF_FLG;
constexpr unsigned OFF_XFLAG = OFF_WFLAG + 512 * 128;
constexpr unsigned OFF_DENSE = OFF_XFLAG + 64 * 128;
constexpr unsigned OFF_SNAP = OFF_DENSE + 8 * 512 * 4;
constexpr unsigned OFF_SOWN = OFF_SNAP + 128;
constexpr unsigned OFF_SFLAG = OFF_SOWN + 128;
constexpr unsigned OFF_SXFLAG = OFF_SFLAG + 1024 * 128;
constexpr unsigned OFF_CTR = OFF_SXFLAG + 128 * 128;
constexpr unsigned OFF_QCTR = OFF_CTR + 256 * 128;
constexpr unsigned OFF_TMO = OFF_QCTR + 16 * 128;
constexpr unsigned WS_USED = OFF_TMO + 128;
static_assert(OFF_WG + 2 * WGB <= OFF_FLG, "layout");

constexpr unsigned NXU = (16 - NXG) * 4;
constexpr unsigned NT = 512 + NXU;

constexpr int GRID = 256;
constexpr int NWAVE = 6;
constexpr int BLOCK = 384;
constexpr unsigned long long DEADLINE = 100000000ull;

constexpr int AUX_SYS = 17;
constexpr int AUX_AGT = 16;
constexpr unsigned OOB = 0xfffffff0u;

#define STAMP(i) do {} while (0)
#define TSTAMP() do {} while (0)

struct Params {
  const int8_t* x;
  const int8_t* w;
  char* my;
  int rank;
  int pad;
  const float* sx;
  const float* sw;
  float* out;
  void* pad2;
  char* ws[NR];
};

__device__ __forceinline__ __amdgpu_buffer_rsrc_t mk_rsrc(const void* p, unsigned bytes) {
  return __builtin_amdgcn_make_buffer_rsrc(const_cast<void*>(p), 0, bytes, 0x00020000);
}
__device__ __forceinline__ unsigned ld_sys(const unsigned* p) {
  return __hip_atomic_load(p, __ATOMIC_RELAXED, __HIP_MEMORY_SCOPE_SYSTEM);
}
__device__ __forceinline__ void st_sys(unsigned* p, unsigned v) {
  __hip_atomic_store(p, v, __ATOMIC_RELAXED, __HIP_MEMORY_SCOPE_SYSTEM);
}
__device__ __forceinline__ unsigned ld_agt(const unsigned* p) {
  return __hip_atomic_load(p, __ATOMIC_RELAXED, __HIP_MEMORY_SCOPE_AGENT);
}
__device__ __forceinline__ void st_agt(unsigned* p, unsigned v) {
  __hip_atomic_store(p, v, __ATOMIC_RELAXED, __HIP_MEMORY_SCOPE_AGENT);
}
__device__ __forceinline__ unsigned ld_lds(const unsigned* p) {
  return __hip_atomic_load(p, __ATOMIC_RELAXED, __HIP_MEMORY_SCOPE_WORKGROUP);
}
__device__ __forceinline__ void st_lds(unsigned* p, unsigned v) {
  __hip_atomic_store(p, v, __ATOMIC_RELAXED, __HIP_MEMORY_SCOPE_WORKGROUP);
}

__device__ __forceinline__ unsigned slot_of(unsigned nb, unsigned ks) {
  return (nb / GCELLS) * (8 * GCELLS) + ks * GCELLS + nb % GCELLS;
}
__device__ __forceinline__ unsigned ord_dist(unsigned ord, unsigned nb) {
  const unsigned qn = nb >> 3;
  unsigned dd = (qn + 9 - ord) & 7;
  if (ord - 1 >= qn) dd = (dd + 7) & 7;
  return ord == 0 ? 0 : dd;
}

struct Move { unsigned soff, doff, foff, goff; };
__device__ __forceinline__ Move decode(unsigned t, unsigned src, unsigned rank, unsigned par) {
  const unsigned sd = (src - rank) & 7;
  const bool isx = t - XPOS < NXU;
  const unsigned xu = NXG * 4 + (t - XPOS);
  const unsigned tw = t < XPOS ? t : t - NXU;
  const unsigned slot = (tw + 64 * sd) & 511;
  const unsigned r = slot % (8 * GCELLS);
  const unsigned nb = (slot / (8 * GCELLS)) * GCELLS + r % GCELLS, ks = r / GCELLS;
  Move m;
  m.soff = (isx ? 512 + rank * 64 + xu : slot) * UNIT;
  m.doff = isx ? par * (NR * XBLK) + (src * 64 + xu) * UNIT : (OFF_WG - OFF_X) + par * WGB + src * WGS + slot * UNIT;
  m.foff = isx ? (OFF_XFLAG - OFF_FLG) + (((xu >> 3) * 8 + src) * 32 + (xu & 7)) * 4
               : (OFF_WFLAG - OFF_FLG) + ((nb * 8 + src) * 32 + ks) * 4;
  m.goff = isx ? OOB : (OFF_DENSE - OFF_FLG) + (src * 512 + slot) * 4;
  return m;
}

__global__ void __launch_bounds__(BLOCK, 2) fused_kernel(Params p) {
  __shared__ unsigned lds[21 * 1024];
  const unsigned b = blockIdx.x;
  const unsigned lane = threadIdx.x & 63;
  const unsigned wid = __builtin_amdgcn_readfirstlane(threadIdx.x >> 6);
  asm volatile("" :: "s"(p.x), "s"(p.w), "s"(p.my), "s"(p.rank), "s"(p.ws[0]));
  const unsigned rank = p.rank;
  char* const my = p.my;
  const unsigned srole = (b >> 3) & 1;
  const unsigned sidx = ((b >> 4) << 3) | (b & 7);
  const unsigned role = (b >> 3) & 1;
  const unsigned idx = ((b >> 4) << 3) | (b & 7);
  const unsigned q = lane >> 4, c = lane & 15;

  unsigned* const ctr = (unsigned*)(my + OFF_CTR) + b * 32 + wid;
  unsigned* const tmo = (unsigned*)(my + OFF_TMO);
  const unsigned long long t_start = __builtin_amdgcn_s_memrealtime();
  unsigned seq;

  STAMP(0);
  if (wid >= 4) {
    if (wid == 4 && lane < 4) st_lds(&lds[lane], 0);
    asm volatile("s_waitcnt lgkmcnt(0)" ::: "memory");
    __builtin_amdgcn_s_barrier();
    asm volatile("" ::: "memory");
    if (CODEPF && wid == 5 && b < 8) {
      unsigned long long pc;
      int la, lb;
      asm volatile("s_getpc_b64 %0\n.Lpf_here:\n\ts_mov_b32 %1, .Lpf_gemm_end-.Lpf_here\n\ts_mov_b32 %2, .Lpf_pull_end-.Lpf_here"
                   : "=s"(pc), "=s"(la), "=s"(lb));
      const unsigned len = (unsigned)max(max(la, lb), 0);
      if (len < 32768u) {
        const __amdgpu_buffer_rsrc_t rc = mk_rsrc((const void*)pc, len & ~15u);
        v4u acc = {0, 0, 0, 0};
#pragma unroll
        for (int i = 0; i < 4; ++i) acc |= __builtin_amdgcn_raw_buffer_load_b128(rc, (lane + i * 64) * 128, 0, 0);
        asm volatile("" :: "v"(acc));
      }
    }
    seq = __builtin_amdgcn_readfirstlane(ld_agt(ctr)) + 1;
  } else if (srole == SWAPROLE && (sidx >> 3) < NXG && (((sidx & 7) + (sidx >> 3)) & 7) != 0) {
    if (wid == 0) {
      const unsigned xg = sidx >> 3;
      const unsigned dst = (rank + (sidx & 7) + xg) & 7;
      v4u v[4][4];
      const int8_t* src = p.x + (size_t)(dst * MS + (xg >> 1) * 64 + c) * KS + (xg & 1) * 256 + q * 16;
      const unsigned cv = ld_agt(ctr);
      asm volatile("" ::: "memory");
#pragma unroll
      for (int u = 0; u < 4; ++u) {
#pragma unroll
        for (int i = 0; i < 4; ++i) v[u][i] = *(const v4u*)(src + i * 16 * KS + u * 64);
        if (XSPLIT > 0 && u == 1) { asm volatile("" ::: "memory"); __builtin_amdgcn_s_sleep(XSPLIT); asm volatile("" ::: "memory"); }
      }
      char* const dws = p.ws[dst];
      __builtin_amdgcn_s_barrier();
      seq = __builtin_amdgcn_readfirstlane(cv) + 1;
      const __amdgpu_buffer_rsrc_t rs = mk_rsrc(dws + OFF_X + (seq & 1) * (NR * XBLK) + rank * XBLK, XBLK);
      const unsigned off = xg * 4 * UNIT + lane * 16;
#pragma unroll
      for (int k = 0; k < 16; ++k) {
        if (k >= PWIN) asm volatile("s_waitcnt vmcnt(%0)" :: "n"(PWIN - 1) : "memory");
        __builtin_amdgcn_raw_buffer_store_b128(v[k >> 2][k & 3], rs, off + (k & 3) * 1024, (k >> 2) * UNIT, AUX_SYS);
      }
      asm volatile("s_waitcnt vmcnt(0)" ::: "memory");
      STAMP(1);
      if (lane < 4) st_sys((unsigned*)(dws + OFF_XFLAG + (((xg >> 1) * 8 + rank) * 32 + (xg & 1) * 4) * 4) + lane, seq);
    } else {
      __builtin_amdgcn_s_barrier();
      asm volatile("" ::: "memory");
      seq = __builtin_amdgcn_readfirstlane(ld_agt(ctr)) + 1;
    }
    if (lane == 0) st_agt(ctr, seq);
  } else if (srole == SWAPROLE) {
    const unsigned xg = sidx >> 3;
    const unsigned dd = ((sidx & 7) + xg) & 7;
    const unsigned dst = (rank + dd) & 7;
    if (LATE > 0 && (xg >= NXG || dd == 0)) __builtin_amdgcn_s_sleep(LATE);
    if (WDELAY > 0) __builtin_amdgcn_s_sleep(WDELAY);
    v4u v[4];
    const int8_t* src = p.x + (size_t)(dst * MS + (xg >> 1) * 64 + c) * KS + (xg & 1) * 256 + wid * 64 + q * 16;
#pragma unroll
    for (int i = 0; i < 4; ++i) v[i] = *(const v4u*)(src + i * 16 * KS);
    const unsigned cv = ld_agt(ctr);
    char* const dws = p.ws[dst];
    __builtin_amdgcn_s_barrier();
    seq = __builtin_amdgcn_readfirstlane(cv) + 1;
    const bool push = xg < NXG || dd == 0;
    char* const base = push ? dws + OFF_X + (seq & 1) * (NR * XBLK) + rank * XBLK
                            : my + OFF_STG + (seq & 1) * STG + SHARD + dst * XBLK;
    const __amdgpu_buffer_rsrc_t rs = mk_rsrc(base, XBLK);
    const unsigned off = (xg * 4 + wid) * UNIT + lane * 16;
#pragma unroll
    for (int i = 0; i < 4; ++i) __builtin_amdgcn_raw_buffer_store_b128(v[i], rs, off + i * 1024, 0, AUX_SYS);
    asm volatile("s_waitcnt vmcnt(0)" ::: "memory");
    STAMP(1);
    unsigned old = 0;
    if (lane == 0) old = __hip_atomic_fetch_add(&lds[3], 1u, __ATOMIC_RELAXED, __HIP_MEMORY_SCOPE_WORKGROUP);
    old = __builtin_amdgcn_readfirstlane(old);
    if (old == 3) {
      const unsigned fo = push ? OFF_XFLAG + (((xg >> 1) * 8 + rank) * 32 + (xg & 1) * 4) * 4
                               : OFF_SXFLAG + (rank * 16 + xg) * 128;
      if (lane < 4) st_sys((unsigned*)(dws + fo) + lane, seq);
    }
    if (lane == 0) st_agt(ctr, seq);
  } else {
    const unsigned sslot = sidx * 4 + wid;
    const unsigned snb = (sslot / (8 * GCELLS)) * GCELLS + sslot % GCELLS, sks = (sslot % (8 * GCELLS)) / GCELLS;
    if (LATE > 0 && (sidx & 15) >= 8) __builtin_amdgcn_s_sleep(LATE);
    if (WDELAY > 0) __builtin_amdgcn_s_sleep(WDELAY);
    char* const peer_ws = p.ws[(rank + lane) & (NR - 1)];
    unsigned d[16];
    {
      const unsigned* src = (const unsigned*)(p.w + (size_t)(sks * 64 + q * 16) * NN + snb * 64 + c * 4);
#pragma unroll
      for (int r = 0; r < 16; ++r) d[r] = src[r * (NN / 4)];
    }
    const unsigned cv = ld_agt(ctr);
    __builtin_amdgcn_s_barrier();
    seq = __builtin_amdgcn_readfirstlane(cv) + 1;
    v4u o[4];
#pragma unroll
    for (int g = 0; g < 4; ++g) {
      const unsigned t0 = __builtin_amdgcn_perm(d[4 * g + 1], d[4 * g + 0], 0x05010400u);
      const unsigned t1 = __builtin_amdgcn_perm(d[4 * g + 1], d[4 * g + 0], 0x07030602u);
      const unsigned t2 = __builtin_amdgcn_perm(d[4 * g + 3], d[4 * g + 2], 0x05010400u);
      const unsigned t3 = __builtin_amdgcn_perm(d[4 * g + 3], d[4 * g + 2], 0x07030602u);
      o[0][g] = __builtin_amdgcn_perm(t2, t0, 0x05040100u);
      o[1][g] = __builtin_amdgcn_perm(t2, t0, 0x07060302u);
      o[2][g] = __builtin_amdgcn_perm(t3, t1, 0x05040100u);
      o[3][g] = __builtin_amdgcn_perm(t3, t1, 0x07060302u);
    }
    const __amdgpu_buffer_rsrc_t rs = mk_rsrc(my + OFF_STG + (seq & 1) * STG, SHARD);
    const unsigned off = sslot * UNIT + lane * 16;
#pragma unroll
    for (int j = 0; j < 4; ++j) __builtin_amdgcn_raw_buffer_store_b128(o[j], rs, off + j * 1024, 0, AUX_SYS);
    asm volatile("s_waitcnt vmcnt(0)" ::: "memory");
    STAMP(1);
    unsigned old = 0;
    if (lane == 0) old = __hip_atomic_fetch_add(&lds[3], 1u, __ATOMIC_RELAXED, __HIP_MEMORY_SCOPE_WORKGROUP);
    old = __builtin_amdgcn_readfirstlane(old);
    if (old == 3 && lane >= 1 && lane < NR)
      st_sys((unsigned*)(peer_ws + OFF_SFLAG) + (rank * 128 + sidx) * 32, seq);
    if (lane == 0) {
      st_agt((unsigned*)(my + OFF_WFLAG) + (snb * 8 + rank) * 32 + sks, seq);
      st_agt(ctr, seq);
    }
  }
  seq = __builtin_amdgcn_readfirstlane(seq);
  const unsigned par = seq & 1;
  bool ctr_done = false, ctr_mark = false;

  if (role == 0) {
    const unsigned sub = idx >> 3;
    const unsigned nb = (sub >> 1) * 8 + (idx & 7);
    const unsigned mb0 = (sub & 1) * 4;
    if (wid == 4) {
      const unsigned psrc = (rank + ord_dist(lane >> 3, nb)) & 7;
      const unsigned* const wf = (const unsigned*)(my + OFF_WFLAG) + (nb * 8 + psrc) * 32 + (lane & 7);
      const __amdgpu_buffer_rsrc_t rxf = mk_rsrc(my + OFF_XFLAG, 64 * 128);
      const unsigned xfo = (((mb0 + (lane & 3)) * 8 + psrc) * 32 + ((lane >> 2) & 1) * 4) * 4;
      bool wok = false, xok = false;
      unsigned long long last = 0;
      for (unsigned spin = 0;; ++spin) {
        const unsigned front = (unsigned)__builtin_ctzll(~last) >> 3;
        const bool near = (lane >> 3) < front + POLLW;
        if (!wok && near) wok = ld_agt(wf) >= seq;
        if (!xok && near) {
          const v4u f = __builtin_amdgcn_raw_buffer_load_b128(rxf, xfo, 0, AUX_SYS);
          xok = f[0] >= seq && f[1] >= seq && f[2] >= seq && f[3] >= seq;
        }
        asm volatile("" ::: "memory");
        const unsigned long long xm = __ballot(xok);
        const unsigned long long m = __ballot(wok && ((xm >> (lane & 56)) & 0xff) == 0xff);
        if (m != last) {
          last = m;
          if (lane < 2) st_lds(&lds[lane], lane ? (unsigned)(m >> 32) : (unsigned)m);
          if (m == ~0ull) break;
        }
        if ((spin & 63) == 63 &&
            (ld_agt(tmo) == seq || __builtin_amdgcn_s_memrealtime() - t_start > DEADLINE)) {
          st_agt(tmo, seq);
          if (lane == 0) st_lds(&lds[2], 1);
          asm volatile("s_waitcnt lgkmcnt(0)" ::: "memory");
          if (lane < 2) st_lds(&lds[lane], 0xffffffffu);
          break;
        }
        if (QSLEEP) __builtin_amdgcn_s_sleep(QSLEEP);
      }
    } else if (wid < 4) {
      const unsigned mb = mb0 + wid;
      const float scale0 = p.sx[0] * p.sw[0];
      v4i acc[4][4];
#pragma unroll
      for (int i = 0; i < 4; ++i)
#pragma unroll
        for (int j = 0; j < 4; ++j) acc[i][j] = v4i{0, 0, 0, 0};
      v4i ra[4][4], rb[4][4];
#pragma unroll
      for (int s = 0; s < 4; ++s)
#pragma unroll
        for (int i = 0; i < 4; ++i) { ra[s][i] = v4i{0, 0, 0, 0}; rb[s][i] = v4i{0, 0, 0, 0}; }

      unsigned tmo_val = 0;
      const __amdgpu_buffer_rsrc_t rws = mk_rsrc(my, OFF_FLG);
      for (unsigned ord = 0; ord < 8; ++ord) {
        const unsigned src = (rank + ord_dist(ord, nb)) & 7;
        const unsigned aoff = OFF_X + par * (NR * XBLK) + src * XBLK + mb * (8 * UNIT);
        const unsigned boff = (src == rank ? OFF_STG + par * STG : OFF_WG + par * WGB + src * WGS) + slot_of(nb, 0) * UNIT;
        const unsigned* const mword = &lds[ord >> 2];
        const unsigned sh = (ord & 3) * 8;
        unsigned ready = 0;
#pragma unroll
        for (int ks = 0; ks < 8; ++ks) {
          if (APRE && ks > 0) {
#pragma unroll
            for (int i = 0; i < 4; ++i)
              ra[ks & 3][i] = (v4i)__builtin_amdgcn_raw_buffer_load_b128(rws, lane * 16 + i * 1024, aoff + ks * UNIT, 0);
            asm volatile("" ::: "memory");
          }
          while (!((ready >> ks) & 1)) {
            ready = __builtin_amdgcn_readfirstlane(ld_lds(mword)) >> sh;
            if ((ready >> ks) & 1) break;
            if (GSLEEP) __builtin_amdgcn_s_sleep(GSLEEP);
          }
          asm volatile("" ::: "memory");
          if (ks == 7) STAMP(3 + ord);
          const int s = ks & 3;
#pragma unroll
          for (int i = 0; i < 4; ++i)
            rb[s][i] = (v4i)__builtin_amdgcn_raw_buffer_load_b128(rws, lane * 16 + i * 1024, boff + ks * (GCELLS * UNIT), 0);
          if (!APRE || ks == 0) {
#pragma unroll
            for (int i = 0; i < 4; ++i)
              ra[s][i] = (v4i)__builtin_amdgcn_raw_buffer_load_b128(rws, lane * 16 + i * 1024, aoff + ks * UNIT, 0);
          }
          if (ks == 7) tmo_val = ld_agt(tmo);
          const int s3 = (ks + 1) & 3;
#pragma unroll
          for (int i = 0; i < 4; ++i)
#pragma unroll
            for (int j = 0; j < 4; ++j)
              acc[i][j] = __builtin_amdgcn_mfma_i32_16x16x64_i8(ra[s3][i], rb[s3][j], acc[i][j], 0, 0, 0);
        }
      }
#pragma unroll
      for (int s = 1; s < 4; ++s)
#pragma unroll
        for (int i = 0; i < 4; ++i)
#pragma unroll
          for (int j = 0; j < 4; ++j)
            acc[i][j] = __builtin_amdgcn_mfma_i32_16x16x64_i8(ra[s][i], rb[s][j], acc[i][j], 0, 0, 0);

      STAMP(11);
      const bool failed = ld_lds(&lds[2]) != 0;
      const float scale = (failed || tmo_val == seq) ? __builtin_nanf("") : scale0;
      const __amdgpu_buffer_rsrc_t ro = mk_rsrc(p.out, (unsigned)MS * NN * 4);
#pragma unroll
      for (int i = 0; i < 4; ++i)
#pragma unroll
        for (int r = 0; r < 4; ++r) {
          v4f v;
#pragma unroll
          for (int j = 0; j < 4; ++j) v[j] = (float)acc[i][j][r] * scale;
          const unsigned row = mb * 64 + i * 16 + q * 4 + r;
          const unsigned off = (row * NN + nb * 64 + c * 4) * 4;
          __builtin_amdgcn_raw_buffer_store_b128((v4u)v, ro, off, 0, OUT_AUX);
        }
      asm volatile(".Lpf_gemm_end:");
    }
  } else if (wid >= 4) {
    const unsigned xcc = __builtin_amdgcn_s_getreg((3 << 11) | 20) & 7;
    const unsigned row = (wid - 4) * 16 + (b >> 4);
    const unsigned CLQ = 0xB2u;
    const unsigned same = (((CLQ >> rank) & 1) ? CLQ : (~CLQ & 0xffu)) & ~(1u << rank);
    const unsigned other = ~same & ~(1u << rank) & 0xffu;
    unsigned cand = AFFINE == 1 ? ((xcc & 4) ? other : same) : ((xcc & 4) ? same : other);
    unsigned li = (row + xcc) & 3;
    if (AFFINE == 0) { cand = 0xffu & ~(1u << rank); li = (row + xcc) & 7; }
    unsigned src0 = 8;
    {
      unsigned m = cand;
      for (unsigned i = 0; i < li && m; ++i) m &= m - 1;
      if (m) src0 = __builtin_ctz(m);
    }
    if (row < PL && src0 < 8) {
      unsigned* const qh0 = (unsigned*)(my + OFF_QCTR) + par * (8 * 32);
      const __amdgpu_buffer_rsrc_t rdst = mk_rsrc(my + OFF_X, OFF_FLG - OFF_X);
      const __amdgpu_buffer_rsrc_t rflag = mk_rsrc(my + OFF_FLG, OFF_SFLAG - OFF_FLG);
      unsigned vzero = 0, vone = 1;
      asm volatile("" : "+v"(vzero), "+v"(vone));
      bool failed = false;
      const unsigned lane16 = lane * 16;
      char* const qws = p.ws[src0];
      unsigned tk = 0;
      unsigned resp = 1u << src0;
      unsigned availv = 0;
      unsigned tried = 0;
      unsigned ksrc = 8, known = 0;
      bool planned = !PLAN;
      bool need_scan = !PLAN;
      unsigned p_own = NT, p_ext = 0, p_xs = 0, p_es = 0;
      const unsigned* const snap = (const unsigned*)(my + OFF_SNAP) + (lane & 7);
      auto make_plan = [&](unsigned sv) {
        const unsigned s8 = lane & 7;
        if (__ballot(s8 == rank || sv >= seq * 1024) != ~0ull) return;
        const unsigned h = s8 == rank ? 0 : sv - seq * 1024;
        unsigned sum = h;
        sum += __shfl_xor((int)sum, 1); sum += __shfl_xor((int)sum, 2); sum += __shfl_xor((int)sum, 4);
        const unsigned a = sum / 7;
        const unsigned e = (s8 != rank && h + PLAN_DEAD < a) ? min(a - h - PLAN_DEAD, 56u) : 0;
        const unsigned sp = (s8 != rank && h > a) ? h - a : 0;
        need_scan = __ballot(s8 != rank && h + PLAN_DEAD + 56u < a) != 0;
        unsigned pe = e, ps = sp;
#pragma unroll
        for (int o = 1; o < 8; o <<= 1) {
          const unsigned ue = __shfl_up((int)pe, o, 8), us = __shfl_up((int)ps, o, 8);
          if (s8 >= (unsigned)o) { pe += ue; ps += us; }
        }
        const unsigned etot = __shfl((int)pe, 7, 8);
        p_own = NT - e;
        p_es = pe - e;
        p_xs = ps - sp;
        p_ext = etot > p_xs ? min(etot - p_xs, sp) : 0;
        planned = true;
      };
      auto take_snapshot = [&]() {
        unsigned old = seq;
        if (lane == 0) old = __hip_atomic_fetch_max((unsigned*)(my + OFF_SOWN), seq, __ATOMIC_RELAXED, __HIP_MEMORY_SCOPE_AGENT);
        if (__builtin_amdgcn_readfirstlane(old) < seq) {
          const unsigned hv = ld_agt(qh0 + (lane & 7) * 32);
          __builtin_amdgcn_raw_buffer_store_b32(seq * 1024 + min(hv, NT), rflag,
                                                lane < 8 ? (OFF_SNAP - OFF_FLG) + lane * 4 : OOB, 0, AUX_AGT);
        }
        asm volatile("s_waitcnt vmcnt(0)" ::: "memory");
      };
      auto map = [&](unsigned qs, unsigned tt, unsigned& us, unsigned& ut) -> bool {
        us = qs; ut = tt;
        if (tt + 64 < NT) return true;
        for (unsigned spin = 0; !planned; ++spin) {
          make_plan(ld_agt(snap));
          if (planned) break;
          if (spin == 0) take_snapshot();
          if ((spin & 63) == 63 &&
              (ld_agt(tmo) == seq || __builtin_amdgcn_s_memrealtime() - t_start > DEADLINE)) {
            st_agt(tmo, seq);
            failed = true;
            return false;
          }
          __builtin_amdgcn_s_sleep(2);
        }
        if (tt < (unsigned)__builtin_amdgcn_readlane(p_own, qs)) return true;
        if (tt < NT || tt - NT >= (unsigned)__builtin_amdgcn_readlane(p_ext, qs)) return false;
        const unsigned g = __builtin_amdgcn_readlane(p_xs, qs) + (tt - NT);
        const unsigned m = (unsigned)__ballot(g >= p_es && g - p_es < NT - p_own) & 0xffu;
        if (!m) return false;
        us = __builtin_ctz(m);
        ut = __builtin_amdgcn_readlane(p_own, us) + (g - __builtin_amdgcn_readlane(p_es, us));
        return true;
      };

      auto draw = [&](unsigned qs) {
        unsigned* const qh = qh0 + qs * 32;
        if (lane == 0)
          asm volatile("global_atomic_add %0, %1, %2, %3 sc0" : "=v"(tk) : "v"(vzero), "v"(vone), "s"(qh) : "memory");
      };
      auto dense_ptr = [&](unsigned s) {
        return (const unsigned*)(qws + OFF_DENSE) + s * 512 + ((448 + 64 * ((s - rank) & 7) + lane) & 511);
      };
      auto prep = [&](unsigned s, unsigned tt) {
        if (s != src0 && tt >= NT - 64 && !((tried >> s) & 1)) {
          availv |= (ld_sys(dense_ptr(s)) >= seq ? 1u : 0u) << s;
          tried |= 1u << s;
        }
        const bool rel = s != src0 && tt >= NT - 64 &&
                         ((__builtin_amdgcn_readlane(availv, (tt - (NT - 64)) & 63) >> s) & 1);
        if (!rel && (s != ksrc || tt >= known)) {
          const unsigned sd = (s - rank) & 7;
          const unsigned* const sf = (const unsigned*)(my + OFF_SFLAG) + s * (128 * 32);
          const unsigned g0 = ((lane + 16 * sd) & 127) * 32, g1 = ((lane + 64 + 16 * sd) & 127) * 32;
          const unsigned* const sxf = (const unsigned*)(my + OFF_SXFLAG) + (s * 16 + (lane & 15)) * 32;
          ksrc = s;
          for (unsigned spin = 0;; ++spin) {
            const unsigned f0 = ld_sys(sf + g0), f1 = ld_sys(sf + g1), fx = ld_sys(sxf);
            const unsigned long long m0 = __ballot(f0 >= seq), m1 = __ballot(f1 >= seq);
            const bool xall = (__ballot(fx >= seq || (lane & 15) < NXG) & 0xffffull) == 0xffffull;
            const unsigned lead = ~m0 ? (unsigned)__builtin_ctzll(~m0) : 64 + (~m1 ? (unsigned)__builtin_ctzll(~m1) : 64);
            const unsigned kw = lead * 4;
            known = kw <= XPOS ? kw : (xall ? kw + NXU : XPOS);
            if (tt < known) break;
            if ((spin & 63) == 63 &&
                (ld_agt(tmo) == seq || __builtin_amdgcn_s_memrealtime() - t_start > DEADLINE)) {
              st_agt(tmo, seq);
              failed = true;
              break;
            }
            if (PSLEEP) __builtin_amdgcn_s_sleep(PSLEEP);
          }
          asm volatile("" ::: "memory");
        }
        char* const base = rel ? qws + OFF_WG + par * WGB + s * WGS : p.ws[s] + OFF_STG + par * STG;
        return mk_rsrc(base, rel ? WGS : STG);
      };

      unsigned ts = src0;
      for (unsigned hop = 0; !failed; ++hop) {
        if (hop > 0) {
          if (!need_scan) break;
          const unsigned s8 = lane & 7;
          const unsigned hv = ld_agt(qh0 + s8 * 32);
          const unsigned lim = p_ext ? NT + p_ext : p_own;
          const bool open = s8 != rank && (hv + HELP_MIN < lim || (((resp >> s8) & 1) && hv < lim));
          unsigned key = (open ? 0xfffu - (lim - hv) : 0xffffu) * 8 + s8;
          key = min(key, (unsigned)__shfl_xor((int)key, 1));
          key = min(key, (unsigned)__shfl_xor((int)key, 2));
          key = min(key, (unsigned)__shfl_xor((int)key, 4));
          key = __builtin_amdgcn_readfirstlane(key);
          if ((key >> 3) >= 0xfffu) break;
          ts = key & 7;
          resp |= 1u << ts;
        }
        draw(ts);
        asm volatile("s_waitcnt vmcnt(0)" : "+v"(tk) :: "memory");
        unsigned t = __builtin_amdgcn_readfirstlane(tk);
        unsigned us, ut;
        if (!map(ts, t, us, ut)) continue;
        Move mv = decode(ut, us, rank, par);
        v4u cur[4];
        if (LIGHT && hop == 0 && ut < XPOS) {
          const unsigned* const f1 = (const unsigned*)(my + OFF_SFLAG) + (us * 128 + (((ut >> 2) + 16 * ((us - rank) & 7)) & 127)) * 32;
          for (unsigned spin = 0; spin < 4096; ++spin)
            if (__builtin_amdgcn_readfirstlane(ld_sys(f1)) >= seq) { ksrc = us; known = ut + 1; break; }
          asm volatile("" ::: "memory");
        }
        {
          const __amdgpu_buffer_rsrc_t rsrc = prep(us, ut);
          if (failed) break;
          if (hop == 0) STAMP(2);
          draw(ts);
#pragma unroll
          for (int j = 0; j < 4; ++j)
            cur[j] = __builtin_amdgcn_raw_buffer_load_b128(rsrc, lane16 + j * 1024, mv.soff, AUX_SYS);
        }
        asm volatile("s_waitcnt vmcnt(4)" : "+v"(tk) :: "memory");
        __builtin_amdgcn_raw_buffer_store_b32(seq, rflag, OOB, 0, AUX_SYS);
        t = __builtin_amdgcn_readfirstlane(tk);
        const unsigned fast_end = NT - SNAP_AT - row * STAG;
        while (t < fast_end) {
          const __amdgpu_buffer_rsrc_t rsrc = prep(ts, t);
          if (failed) break;
          while (t < fast_end && t < known) {
            const Move nx = decode(t, ts, rank, par);
            TSTAMP();
#pragma unroll
            for (int j = 0; j < 4; ++j) __builtin_amdgcn_raw_buffer_store_b128(cur[j], rdst, mv.doff + lane16 + j * 1024, 0, ST_AUX);
            draw(ts);
#pragma unroll
            for (int j = 0; j < 4; ++j)
              cur[j] = __builtin_amdgcn_raw_buffer_load_b128(rsrc, lane16 + j * 1024, nx.soff, LD_AUX);
            asm volatile("s_waitcnt vmcnt(4)" : "+v"(tk) :: "memory");
            __builtin_amdgcn_raw_buffer_store_b32(seq, rflag, lane == 0 ? mv.foff : (lane == 1 ? mv.goff : OOB), 0, AUX_SYS);
            t = __builtin_amdgcn_readfirstlane(tk);
            mv = nx;
          }
        }
        if (failed) break;
        if (hop == 0) { if (lane == 0) st_agt(ctr, seq); ctr_mark = true; }
        while (map(ts, t, us, ut)) {
          const Move nx = decode(ut, us, rank, par);
          TSTAMP();
          const __amdgpu_buffer_rsrc_t rsrc = prep(us, ut);
          if (failed) break;
          if (PLAN && t + SNAP_AT == NT) take_snapshot();
#pragma unroll
          for (int j = 0; j < 4; ++j) __builtin_amdgcn_raw_buffer_store_b128(cur[j], rdst, mv.doff + lane16 + j * 1024, 0, ST_AUX);
          draw(ts);
          unsigned sv = 0, vv = 0, wantv = 8;
          const bool want_plan = !planned && t + 112 >= NT;
          if (want_plan) sv = ld_agt(snap);
          if (planned && ts == src0 && t + 2 * PL >= NT && __builtin_amdgcn_readlane(p_ext, src0) != 0) {
            const unsigned g = __builtin_amdgcn_readlane(p_xs, src0);
            const unsigned m = (unsigned)__ballot(g >= p_es && g - p_es < NT - p_own) & 0xffu & ~tried;
            if (m) { wantv = __builtin_ctz(m); vv = ld_sys(dense_ptr(wantv)); }
          }
#pragma unroll
          for (int j = 0; j < 4; ++j)
            cur[j] = __builtin_amdgcn_raw_buffer_load_b128(rsrc, lane16 + j * 1024, nx.soff, LD_AUX);
          asm volatile("s_waitcnt vmcnt(4)" : "+v"(tk) :: "memory");
          __builtin_amdgcn_raw_buffer_store_b32(seq, rflag, lane == 0 ? mv.foff : (lane == 1 ? mv.goff : OOB), 0, AUX_SYS);
          t = __builtin_amdgcn_readfirstlane(tk);
          mv = nx;
          if (want_plan) make_plan(sv);
          if (wantv < 8) {
            availv |= (vv >= seq ? 1u : 0u) << wantv;
            tried |= 1u << wantv;
          }
        }
        if (failed) break;
#pragma unroll
        for (int j = 0; j < 4; ++j) __builtin_amdgcn_raw_buffer_store_b128(cur[j], rdst, mv.doff + lane16 + j * 1024, 0, ST_AUX);
        asm volatile("s_waitcnt vmcnt(0)" ::: "memory");
        __builtin_amdgcn_raw_buffer_store_b32(seq, rflag, lane == 0 ? mv.foff : (lane == 1 ? mv.goff : OOB), 0, AUX_SYS);
        asm volatile(".Lpf_pull_end:");
        if (hop == 0) STAMP(4);
        STAMP(3);
      }
      ctr_done = ctr_mark;
    }
  }
  if (wid >= 4) {
    if (lane == 0 && !ctr_done) st_agt(ctr, seq);
    if (b == 9 && wid == 5 && lane < 8) st_agt((unsigned*)(my + OFF_QCTR) + ((par ^ 1) * 8 + lane) * 32, 0);
  }
}

}

extern "C" void kernel_launch(int rank, int nranks, ncclComm_t comm,
                              hipStream_t stream,
                              void* const* d_in, const int* in_sizes, int n_in,
                              void* d_out, int out_size,
                              const int* in_dtypes, int out_dtype,
                              void* const* d_out_all,
                              void* const* d_ws_all, size_t ws_bytes) {
  (void)comm; (void)in_dtypes; (void)out_dtype; (void)d_out_all;
  if (nranks != NR || (unsigned)rank >= (unsigned)NR || n_in != 4 || ws_bytes < WS_USED ||
      in_sizes[0] != NR * MS * KS || in_sizes[1] != KS * NN || out_size != MS * NN) {
    fprintf(stderr, "unexpected problem geometry\n");
    return;
  }
  Params p;
  p.x = (const int8_t*)d_in[0];
  p.w = (const int8_t*)d_in[1];
  p.sx = (const float*)d_in[2];
  p.sw = (const float*)d_in[3];
  p.out = (float*)d_out;
  for (int i = 0; i < NR; ++i) p.ws[i] = (char*)d_ws_all[i];
  p.rank = rank;
  p.my = p.ws[rank];
  p.pad = 0;
  p.pad2 = nullptr;
  fused_kernel<<<GRID, BLOCK, 0, stream>>>(p);
  HIP_CHECK(hipGetLastError());
}
```

```cpp
#include <hip/hip_runtime.h>
#include <rccl/rccl.h>
#include <cstdio>
#include <cstdint>

#define HIP_CHECK(expr) do { hipError_t _e = (expr); if (_e != hipSuccess) { \
    fprintf(stderr, "HIP error %s at %s:%d\n", hipGetErrorString(_e), __FILE__, __LINE__); \
    return; } } while (0)

namespace {

typedef int v4i __attribute__((ext_vector_type(4)));
typedef unsigned v4u __attribute__((ext_vector_type(4)));
typedef float v4f __attribute__((ext_vector_type(4)));

constexpr int NR = 8;
constexpr int NN = 4096;
constexpr int MS = 512;
constexpr int KS = 512;
constexpr unsigned UNIT = 4096;
constexpr unsigned SHARD = 2u << 20;
constexpr unsigned XBLK = 256u << 10;

#define TM 48
#define PL 14
#define PWIN 3
#define WDELAY 48
#define XSPLIT 6
#define SNAP_AT 128
#define PLAN_DEAD 4
#define HELP_MIN 64
#define POLLW 2
#define AFFINE 1

constexpr unsigned OFF_STG = 0;
constexpr unsigned OFF_X = 4u << 20;
constexpr unsigned OFF_WG = 8u << 20;
constexpr unsigned WGB = NR * SHARD;
constexpr unsigned OFF_FLG = 40u << 20;
constexpr unsigned OFF_WFLAG = OFF_FLG;
constexpr unsigned OFF_DENSE = OFF_WFLAG + 512 * 128;
constexpr unsigned OFF_SNAP = OFF_DENSE + 8 * 512 * 4;
constexpr unsigned OFF_SOWN = OFF_SNAP + 128;
constexpr unsigned OFF_XFLAG = OFF_SOWN + 128;
constexpr unsigned OFF_SFLAG = OFF_XFLAG + 128 * 128;
constexpr unsigned OFF_CTR = OFF_SFLAG + 1024 * 128;
constexpr unsigned OFF_QCTR = OFF_CTR + 256 * 128;
constexpr unsigned OFF_TMO = OFF_QCTR + 16 * 128;
constexpr unsigned WS_USED = OFF_TMO + 128;
static_assert(OFF_WG + 2 * WGB <= OFF_FLG, "layout");

constexpr unsigned NT = 512;
constexpr int GRID = 256;
constexpr int NWAVE = 6;
constexpr int BLOCK = NWAVE * 64;
constexpr int MF = TM / 16;
constexpr unsigned long long DEADLINE = 100000000ull;

constexpr int AUX_SYS = 17;
constexpr int AUX_AGT = 16;
constexpr unsigned OOB = 0xfffffff0u;

#define STAMP(i) do {} while (0)

struct Params {
  const int8_t* x;
  const int8_t* w;
  char* my;
  int rank;
  int pad;
  const float* sx;
  const float* sw;
  float* out;
  void* pad2;
  char* ws[NR];
};

__device__ __forceinline__ __amdgpu_buffer_rsrc_t mk_rsrc(const void* p, unsigned bytes) {
  return __builtin_amdgcn_make_buffer_rsrc(const_cast<void*>(p), 0, bytes, 0x00020000);
}
__device__ __forceinline__ unsigned ld_sys(const unsigned* p) {
  return __hip_atomic_load(p, __ATOMIC_RELAXED, __HIP_MEMORY_SCOPE_SYSTEM);
}
__device__ __forceinline__ void st_sys(unsigned* p, unsigned v) {
  __hip_atomic_store(p, v, __ATOMIC_RELAXED, __HIP_MEMORY_SCOPE_SYSTEM);
}
__device__ __forceinline__ unsigned ld_agt(const unsigned* p) {
  return __hip_atomic_load(p, __ATOMIC_RELAXED, __HIP_MEMORY_SCOPE_AGENT);
}
__device__ __forceinline__ void st_agt(unsigned* p, unsigned v) {
  __hip_atomic_store(p, v, __ATOMIC_RELAXED, __HIP_MEMORY_SCOPE_AGENT);
}
__device__ __forceinline__ unsigned ld_lds(const unsigned* p) {
  return __hip_atomic_load(p, __ATOMIC_RELAXED, __HIP_MEMORY_SCOPE_WORKGROUP);
}
__device__ __forceinline__ void st_lds(unsigned* p, unsigned v) {
  __hip_atomic_store(p, v, __ATOMIC_RELAXED, __HIP_MEMORY_SCOPE_WORKGROUP);
}

__device__ __forceinline__ unsigned slot_of(unsigned nb, unsigned ks) { return (nb >> 3) * 64 + ks * 8 + (nb & 7); }
__device__ __forceinline__ unsigned ord_dist(unsigned ord, unsigned nb) {
  const unsigned qn = nb >> 3;
  unsigned dd = (qn + 9 - ord) & 7;
  if (ord - 1 >= qn) dd = (dd + 7) & 7;
  return ord == 0 ? 0 : dd;
}

struct Move { unsigned soff, doff, foff, goff; };
__device__ __forceinline__ Move decode(unsigned t, unsigned src, unsigned rank) {
  const unsigned slot = (t + 64 * ((src - rank) & 7)) & 511;
  const unsigned nb = (slot >> 6) * 8 + (slot & 7), ks = (slot >> 3) & 7;
  Move m;
  m.soff = slot * UNIT;
  m.doff = src * SHARD + slot * UNIT;
  m.foff = (OFF_WFLAG - OFF_FLG) + ((nb * 8 + src) * 32 + ks) * 4;
  m.goff = (OFF_DENSE - OFF_FLG) + (src * 512 + slot) * 4;
  return m;
}

__global__ void __launch_bounds__(BLOCK, 2) fused_kernel(Params p) {
  __shared__ unsigned lds[21 * 1024];
  asm volatile("" :: "s"(p.x), "s"(p.w), "s"(p.my), "s"(p.rank), "s"(p.ws[0]));
  const unsigned b = blockIdx.x;
  const unsigned lane = threadIdx.x & 63;
  const unsigned wid = __builtin_amdgcn_readfirstlane(threadIdx.x >> 6);
  const unsigned rank = p.rank;
  char* const my = p.my;
  const unsigned role = (b >> 3) & 1;
  const unsigned idx = ((b >> 4) << 3) | (b & 7);
  const unsigned q = lane >> 4, c = lane & 15;

  unsigned* const ctr = (unsigned*)(my + OFF_CTR) + b * 32 + wid;
  unsigned* const tmo = (unsigned*)(my + OFF_TMO);
  const unsigned long long t_start = __builtin_amdgcn_s_memrealtime();
  unsigned seq;

  STAMP(0);
  if (role == 0 && wid == 0) {
    const unsigned xg = idx >> 3;
    const unsigned dst = (rank + (idx & 7) + xg) & 7;
    v4u v[4][4];
    const int8_t* src = p.x + (size_t)(dst * MS + (xg >> 1) * 64 + c) * KS + (xg & 1) * 256 + q * 16;
    const unsigned cv = ld_agt(ctr);
    asm volatile("" ::: "memory");
#pragma unroll
    for (int u = 0; u < 4; ++u) {
#pragma unroll
      for (int i = 0; i < 4; ++i) v[u][i] = *(const v4u*)(src + i * 16 * KS + u * 64);
      if (XSPLIT > 0 && u == 1) { asm volatile("" ::: "memory"); __builtin_amdgcn_s_sleep(XSPLIT); asm volatile("" ::: "memory"); }
    }
    char* const dws = p.ws[dst];
    __builtin_amdgcn_s_barrier();
    seq = __builtin_amdgcn_readfirstlane(cv) + 1;
    const __amdgpu_buffer_rsrc_t rs = mk_rsrc(dws + OFF_X + (seq & 1) * (NR * XBLK) + rank * XBLK, XBLK);
    const unsigned off = xg * 4 * UNIT + lane * 16;
#pragma unroll
    for (int k = 0; k < 16; ++k) {
      if (k >= PWIN) asm volatile("s_waitcnt vmcnt(%0)" :: "n"(PWIN - 1) : "memory");
      __builtin_amdgcn_raw_buffer_store_b128(v[k >> 2][k & 3], rs, off + (k & 3) * 1024, (k >> 2) * UNIT, AUX_SYS);
    }
    asm volatile("s_waitcnt vmcnt(0)" ::: "memory");
    STAMP(1);
    if (lane == 0) {
      st_sys((unsigned*)(dws + OFF_XFLAG) + (((xg >> 1) * 8 + rank) * 2 + (xg & 1)) * 32, seq);
      st_agt(ctr, seq);
    }
  } else if (role == 1 && wid < 4) {
    const unsigned sslot = idx * 4 + wid;
    const unsigned snb = (sslot >> 6) * 8 + (sslot & 7), sks = (sslot >> 3) & 7;
    if (WDELAY > 0) __builtin_amdgcn_s_sleep(WDELAY);
    char* const peer_ws = p.ws[(rank + lane) & (NR - 1)];
    unsigned d[16];
    {
      const unsigned* src = (const unsigned*)(p.w + (size_t)(sks * 64 + q * 16) * NN + snb * 64 + c * 4);
#pragma unroll
      for (int r = 0; r < 16; ++r) d[r] = src[r * (NN / 4)];
    }
    const unsigned cv = ld_agt(ctr);
    __builtin_amdgcn_s_barrier();
    seq = __builtin_amdgcn_readfirstlane(cv) + 1;
    v4u o[4];
#pragma unroll
    for (int g = 0; g < 4; ++g) {
      const unsigned t0 = __builtin_amdgcn_perm(d[4 * g + 1], d[4 * g + 0], 0x05010400u);
      const unsigned t1 = __builtin_amdgcn_perm(d[4 * g + 1], d[4 * g + 0], 0x07030602u);
      const unsigned t2 = __builtin_amdgcn_perm(d[4 * g + 3], d[4 * g + 2], 0x05010400u);
      const unsigned t3 = __builtin_amdgcn_perm(d[4 * g + 3], d[4 * g + 2], 0x07030602u);
      o[0][g] = __builtin_amdgcn_perm(t2, t0, 0x05040100u);
      o[1][g] = __builtin_amdgcn_perm(t2, t0, 0x07060302u);
      o[2][g] = __builtin_amdgcn_perm(t3, t1, 0x05040100u);
      o[3][g] = __builtin_amdgcn_perm(t3, t1, 0x07060302u);
    }
    const __amdgpu_buffer_rsrc_t rs = mk_rsrc(my + OFF_STG + (seq & 1) * SHARD, SHARD);
    const unsigned off = sslot * UNIT + lane * 16;
#pragma unroll
    for (int j = 0; j < 4; ++j) __builtin_amdgcn_raw_buffer_store_b128(o[j], rs, off + j * 1024, 0, AUX_SYS);
    asm volatile("s_waitcnt vmcnt(0)" ::: "memory");
    STAMP(1);
    unsigned old = 0;
    if (lane == 0) old = __hip_atomic_fetch_add(&lds[3], 1u, __ATOMIC_RELAXED, __HIP_MEMORY_SCOPE_WORKGROUP);
    old = __builtin_amdgcn_readfirstlane(old);
    if (old == 3 && lane >= 1 && lane < NR)
      st_sys((unsigned*)(peer_ws + OFF_SFLAG) + (rank * 128 + idx) * 32, seq);
    if (lane == 0) {
      st_agt((unsigned*)(my + OFF_WFLAG) + (snb * 8 + rank) * 32 + sks, seq);
      st_agt(ctr, seq);
    }
  } else {
    if (wid == 4 && lane < 4) st_lds(&lds[lane], 0);
    asm volatile("s_waitcnt lgkmcnt(0)" ::: "memory");
    __builtin_amdgcn_s_barrier();
    asm volatile("" ::: "memory");
    if (wid == 5 && b < 8) {
      unsigned long long pc;
      int la, lb;
      asm volatile("s_getpc_b64 %0\n.Lpf_here:\n\ts_mov_b32 %1, .Lpf_gemm_end-.Lpf_here\n\ts_mov_b32 %2, .Lpf_pull_end-.Lpf_here"
                   : "=s"(pc), "=s"(la), "=s"(lb));
      const unsigned len = (unsigned)max(max(la, lb), 0);
      if (len < 32768u) {
        const __amdgpu_buffer_rsrc_t rc = mk_rsrc((const void*)pc, len & ~15u);
        v4u acc = {0, 0, 0, 0};
#pragma unroll
        for (int i = 0; i < 4; ++i) acc |= __builtin_amdgcn_raw_buffer_load_b128(rc, (lane + i * 64) * 128, 0, 0);
        asm volatile("" :: "v"(acc));
      }
    }
    seq = __builtin_amdgcn_readfirstlane(ld_agt(ctr)) + 1;
    if (lane == 0) st_agt(ctr, seq);
    if (b == 16 && wid == 5 && lane < 8) st_agt((unsigned*)(my + OFF_QCTR) + (((seq & 1) ^ 1) * 8 + lane) * 32, 0);
  }
  seq = __builtin_amdgcn_readfirstlane(seq);
  const unsigned par = seq & 1;

  const unsigned part = TM == 64 ? ((b >> 3) & 3) == 0 ? 0 : ((b >> 3) & 3) == 2 ? 1 : 3 : (b >> 3) & 3;
  const bool gemm_wg = part < 3;
  const unsigned nb = (b >> 5) * 8 + (b & 7);
  const unsigned f0 = part * (TM == 64 ? 16 : 11);
  const unsigned f1 = min(f0 + (TM == 64 ? 16 : 11), 32u);
  const bool mover_wave = TM == 64 ? (role == 1 && wid == 5) : (part == 3 && wid >= 4);

  if (wid == 4 && gemm_wg) {
    {
      const unsigned psrc = (rank + ord_dist(lane >> 3, nb)) & 7;
      const unsigned* const wf = (const unsigned*)(my + OFF_WFLAG) + (nb * 8 + psrc) * 32 + (lane & 7);
      const unsigned xmb = min((f0 >> 2) + ((lane >> 1) & 3), (f1 - 1) >> 2);
      const unsigned* const xf = (const unsigned*)(my + OFF_XFLAG) + ((xmb * 8 + psrc) * 2 + (lane & 1)) * 32;
      bool wok = false, xok = false;
      unsigned long long last = 0;
      for (unsigned spin = 0;; ++spin) {
        const unsigned front = (unsigned)__builtin_ctzll(~last) >> 3;
        const bool near = (lane >> 3) < front + POLLW;
        if (!wok && near) wok = ld_agt(wf) >= seq;
        if (!xok && near) xok = ld_sys(xf) >= seq;
        asm volatile("" ::: "memory");
        const unsigned long long xm = __ballot(xok);
        const unsigned long long m = __ballot(wok && ((xm >> (lane & 56)) & 0xff) == 0xff);
        if (m != last) {
          last = m;
          if (lane < 2) st_lds(&lds[lane], lane ? (unsigned)(m >> 32) : (unsigned)m);
          if (m == ~0ull) break;
        }
        if ((spin & 63) == 63 &&
            (ld_agt(tmo) == seq || __builtin_amdgcn_s_memrealtime() - t_start > DEADLINE)) {
          st_agt(tmo, seq);
          if (lane == 0) st_lds(&lds[2], 1);
          asm volatile("s_waitcnt lgkmcnt(0)" ::: "memory");
          if (lane < 2) st_lds(&lds[lane], 0xffffffffu);
          break;
        }
        __builtin_amdgcn_s_sleep(1);
      }
    }
  } else if (wid < 4) {
    if (gemm_wg) {
      const unsigned wf0 = f0 + wid * MF;
      unsigned avo[MF];
#pragma unroll
      for (int i = 0; i < MF; ++i)
        avo[i] = wf0 + i < f1 ? ((wf0 + i) >> 2) * (8 * UNIT) + ((wf0 + i) & 3) * 1024 + lane * 16 : 0x80000000u;
      const float scale0 = p.sx[0] * p.sw[0];
      v4i acc[MF][4];
#pragma unroll
      for (int i = 0; i < MF; ++i)
#pragma unroll
        for (int j = 0; j < 4; ++j) acc[i][j] = v4i{0, 0, 0, 0};
      v4i ra[4][MF], rb[4][4];
#pragma unroll
      for (int s = 0; s < 4; ++s) {
#pragma unroll
        for (int i = 0; i < MF; ++i) ra[s][i] = v4i{0, 0, 0, 0};
#pragma unroll
        for (int j = 0; j < 4; ++j) rb[s][j] = v4i{0, 0, 0, 0};
      }

      unsigned tmo_val = 0;
      const __amdgpu_buffer_rsrc_t rws = mk_rsrc(my, OFF_FLG);
      for (unsigned ord = 0; ord < 8; ++ord) {
        const unsigned src = (rank + ord_dist(ord, nb)) & 7;
        const unsigned aoff = OFF_X + par * (NR * XBLK) + src * XBLK;
        const unsigned boff = (src == rank ? OFF_STG + par * SHARD : OFF_WG + par * WGB + src * SHARD) + slot_of(nb, 0) * UNIT;
        const unsigned* const mword = &lds[ord >> 2];
        const unsigned sh = (ord & 3) * 8;
        unsigned ready = 0;
#pragma unroll
        for (int ks = 0; ks < 8; ++ks) {
          while (!((ready >> ks) & 1)) {
            ready = __builtin_amdgcn_readfirstlane(ld_lds(mword)) >> sh;
            if ((ready >> ks) & 1) break;
            __builtin_amdgcn_s_sleep(1);
          }
          asm volatile("" ::: "memory");
          if (ks == 7) STAMP(3 + ord);
          const int s = ks & 3;
#pragma unroll
          for (int j = 0; j < 4; ++j)
            rb[s][j] = (v4i)__builtin_amdgcn_raw_buffer_load_b128(rws, lane * 16 + j * 1024, boff + ks * (8 * UNIT), 0);
#pragma unroll
          for (int i = 0; i < MF; ++i)
            ra[s][i] = (v4i)__builtin_amdgcn_raw_buffer_load_b128(rws, avo[i], aoff + ks * UNIT, 0);
          if (ks == 7) tmo_val = ld_agt(tmo);
          const int s3 = (ks + 1) & 3;
#pragma unroll
          for (int i = 0; i < MF; ++i)
#pragma unroll
            for (int j = 0; j < 4; ++j)
              acc[i][j] = __builtin_amdgcn_mfma_i32_16x16x64_i8(ra[s3][i], rb[s3][j], acc[i][j], 0, 0, 0);
        }
      }
#pragma unroll
      for (int s = 1; s < 4; ++s)
#pragma unroll
        for (int i = 0; i < MF; ++i)
#pragma unroll
          for (int j = 0; j < 4; ++j)
            acc[i][j] = __builtin_amdgcn_mfma_i32_16x16x64_i8(ra[s][i], rb[s][j], acc[i][j], 0, 0, 0);

      STAMP(11);
      const bool failed = ld_lds(&lds[2]) != 0;
      const float scale = (failed || tmo_val == seq) ? __builtin_nanf("") : scale0;
      const __amdgpu_buffer_rsrc_t ro = mk_rsrc(p.out, (unsigned)MS * NN * 4);
#pragma unroll
      for (int i = 0; i < MF; ++i)
#pragma unroll
        for (int r = 0; r < 4; ++r) {
          v4f v;
#pragma unroll
          for (int j = 0; j < 4; ++j) v[j] = (float)acc[i][j][r] * scale;
          const unsigned row = (wf0 + i) * 16 + q * 4 + r;
          const unsigned off = wf0 + i < f1 ? (row * NN + nb * 64 + c * 4) * 4 : OOB;
          __builtin_amdgcn_raw_buffer_store_b128((v4u)v, ro, off, 0, AUX_AGT);
        }
      asm volatile(".Lpf_gemm_end:");
    }
  } else if (mover_wave) {
    const unsigned xcc = __builtin_amdgcn_s_getreg((3 << 11) | 20) & 7;
    const unsigned row = TM == 64 ? b >> 4 : (wid - 4) * 8 + (b >> 5);
    const unsigned CLQ = 0xB2u;
    const unsigned same = (((CLQ >> rank) & 1) ? CLQ : (~CLQ & 0xffu)) & ~(1u << rank);
    const unsigned other = ~same & ~(1u << rank) & 0xffu;
    unsigned cand = AFFINE == 1 ? ((xcc & 4) ? other : same) : ((xcc & 4) ? same : other);
    unsigned li = (row + xcc) & 3;
    if (AFFINE == 0) { cand = 0xffu & ~(1u << rank); li = (row + xcc) & 7; }
    unsigned src0 = 8;
    {
      unsigned m = cand;
      for (unsigned i = 0; i < li && m; ++i) m &= m - 1;
      if (m) src0 = __builtin_ctz(m);
    }
    if (row < PL && src0 < 8) {
      unsigned* const qh0 = (unsigned*)(my + OFF_QCTR) + par * (8 * 32);
      const __amdgpu_buffer_rsrc_t rdst = mk_rsrc(my + OFF_WG + par * WGB, WGB);
      const __amdgpu_buffer_rsrc_t rflag = mk_rsrc(my + OFF_FLG, OFF_XFLAG - OFF_FLG);
      unsigned vzero = 0, vone = 1;
      asm volatile("" : "+v"(vzero), "+v"(vone));
      bool failed = false;
      const unsigned lane16 = lane * 16;
      char* const qws = p.ws[src0];
      unsigned tk = 0;
      unsigned resp = 1u << src0;
      unsigned availv = 0;
      unsigned tried = 0;
      unsigned ksrc = 8, known = 0;
      bool planned = false;
      bool need_scan = false;
      unsigned p_own = NT, p_ext = 0, p_xs = 0, p_es = 0;
      const unsigned* const snap = (const unsigned*)(my + OFF_SNAP) + (lane & 7);

      auto timed_out = [&]() {
        if (ld_agt(tmo) == seq || __builtin_amdgcn_s_memrealtime() - t_start > DEADLINE) {
          st_agt(tmo, seq);
          failed = true;
        }
        return failed;
      };
      auto make_plan = [&](unsigned sv) {
        const unsigned s8 = lane & 7;
        if (__ballot(s8 == rank || sv >= seq * 1024) != ~0ull) return;
        const unsigned h = s8 == rank ? 0 : sv - seq * 1024;
        unsigned sum = h;
        sum += __shfl_xor((int)sum, 1); sum += __shfl_xor((int)sum, 2); sum += __shfl_xor((int)sum, 4);
        const unsigned a = sum / 7;
        const unsigned e = (s8 != rank && h + PLAN_DEAD < a) ? min(a - h - PLAN_DEAD, 56u) : 0;
        const unsigned sp = (s8 != rank && h > a) ? h - a : 0;
        need_scan = __ballot(s8 != rank && (h == 0 || h + PLAN_DEAD + 56u < a)) != 0;
        unsigned pe = e, ps = sp;
#pragma unroll
        for (int o = 1; o < 8; o <<= 1) {
          const unsigned ue = __shfl_up((int)pe, o, 8), us = __shfl_up((int)ps, o, 8);
          if (s8 >= (unsigned)o) { pe += ue; ps += us; }
        }
        const unsigned etot = __shfl((int)pe, 7, 8);
        p_own = NT - e;
        p_es = pe - e;
        p_xs = ps - sp;
        p_ext = etot > p_xs ? min(etot - p_xs, sp) : 0;
        planned = true;
      };
      auto take_snapshot = [&]() {
        unsigned old = seq;
        if (lane == 0) old = __hip_atomic_fetch_max((unsigned*)(my + OFF_SOWN), seq, __ATOMIC_RELAXED, __HIP_MEMORY_SCOPE_AGENT);
        if (__builtin_amdgcn_readfirstlane(old) < seq) {
          const unsigned hv = ld_agt(qh0 + (lane & 7) * 32);
          __builtin_amdgcn_raw_buffer_store_b32(seq * 1024 + min(hv, NT), rflag,
                                                lane < 8 ? (OFF_SNAP - OFF_FLG) + lane * 4 : OOB, 0, AUX_AGT);
        }
        asm volatile("s_waitcnt vmcnt(0)" ::: "memory");
      };
      auto map = [&](unsigned qs, unsigned tt, unsigned& us, unsigned& ut) -> bool {
        us = qs; ut = tt;
        if (tt + 64 < NT) return true;
        for (unsigned spin = 0; !planned; ++spin) {
          make_plan(ld_agt(snap));
          if (planned) break;
          if (spin == 0) take_snapshot();
          if ((spin & 63) == 63 && timed_out()) return false;
          __builtin_amdgcn_s_sleep(2);
        }
        if (tt < (unsigned)__builtin_amdgcn_readlane(p_own, qs)) return true;
        if (tt < NT || tt - NT >= (unsigned)__builtin_amdgcn_readlane(p_ext, qs)) return false;
        const unsigned g = __builtin_amdgcn_readlane(p_xs, qs) + (tt - NT);
        const unsigned m = (unsigned)__ballot(g >= p_es && g - p_es < NT - p_own) & 0xffu;
        if (!m) return false;
        us = __builtin_ctz(m);
        ut = __builtin_amdgcn_readlane(p_own, us) + (g - __builtin_amdgcn_readlane(p_es, us));
        return true;
      };
      auto draw = [&](unsigned qs) {
        unsigned* const qh = qh0 + qs * 32;
        if (lane == 0)
          asm volatile("global_atomic_add %0, %1, %2, %3 sc0" : "=v"(tk) : "v"(vzero), "v"(vone), "s"(qh) : "memory");
      };
      auto dense_ptr = [&](unsigned s) {
        return (const unsigned*)(qws + OFF_DENSE) + s * 512 + ((448 + 64 * ((s - rank) & 7) + lane) & 511);
      };
      auto prep = [&](unsigned s, unsigned tt) {
        if (s != src0 && tt >= NT - 64 && !((tried >> s) & 1)) {
          availv |= (ld_sys(dense_ptr(s)) >= seq ? 1u : 0u) << s;
          tried |= 1u << s;
        }
        const bool rel = s != src0 && tt >= NT - 64 &&
                         ((__builtin_amdgcn_readlane(availv, (tt - (NT - 64)) & 63) >> s) & 1);
        if (!rel && (s != ksrc || tt >= known)) {
          const unsigned sd = (s - rank) & 7;
          const unsigned* const sf = (const unsigned*)(my + OFF_SFLAG) + s * (128 * 32);
          const unsigned g0 = ((lane + 16 * sd) & 127) * 32, g1 = ((lane + 64 + 16 * sd) & 127) * 32;
          ksrc = s;
          for (unsigned spin = 0;; ++spin) {
            const unsigned f0 = ld_sys(sf + g0), f1 = ld_sys(sf + g1);
            const unsigned long long m0 = __ballot(f0 >= seq), m1 = __ballot(f1 >= seq);
            known = 4 * (~m0 ? (unsigned)__builtin_ctzll(~m0) : 64 + (~m1 ? (unsigned)__builtin_ctzll(~m1) : 64));
            if (tt < known) break;
            if ((spin & 63) == 63 && timed_out()) break;
            __builtin_amdgcn_s_sleep(1);
          }
          asm volatile("" ::: "memory");
        }
        return mk_rsrc(rel ? qws + OFF_WG + par * WGB + s * SHARD : p.ws[s] + OFF_STG + par * SHARD, SHARD);
      };

      unsigned ts = src0;
      for (unsigned hop = 0; !failed; ++hop) {
        if (hop > 0) {
          if (!need_scan) break;
          const unsigned s8 = lane & 7;
          const unsigned hv = ld_agt(qh0 + s8 * 32);
          const unsigned lim = p_ext ? NT + p_ext : p_own;
          const bool open = s8 != rank && (hv + HELP_MIN < lim || (((resp >> s8) & 1) && hv < lim));
          unsigned key = (open ? 0xfffu - (lim - hv) : 0xffffu) * 8 + s8;
          key = min(key, (unsigned)__shfl_xor((int)key, 1));
          key = min(key, (unsigned)__shfl_xor((int)key, 2));
          key = min(key, (unsigned)__shfl_xor((int)key, 4));
          key = __builtin_amdgcn_readfirstlane(key);
          if ((key >> 3) >= 0xfffu) break;
          ts = key & 7;
          resp |= 1u << ts;
        }
        draw(ts);
        asm volatile("s_waitcnt vmcnt(0)" : "+v"(tk) :: "memory");
        unsigned t = __builtin_amdgcn_readfirstlane(tk);
        unsigned us, ut;
        if (!map(ts, t, us, ut)) continue;
        Move mv = decode(ut, us, rank);
        v4u cur[4];
        if (hop == 0 && ut < 64) {
          const unsigned* const f1 = (const unsigned*)(my + OFF_SFLAG) + (us * 128 + (mv.soff >> 14)) * 32;
          for (unsigned spin = 0; spin < 4096; ++spin)
            if (__builtin_amdgcn_readfirstlane(ld_sys(f1)) >= seq) { ksrc = us; known = ut + 1; break; }
          asm volatile("" ::: "memory");
        }
        {
          const __amdgpu_buffer_rsrc_t rsrc = prep(us, ut);
          if (failed) break;
          if (hop == 0) STAMP(2);
          draw(ts);
#pragma unroll
          for (int j = 0; j < 4; ++j)
            cur[j] = __builtin_amdgcn_raw_buffer_load_b128(rsrc, lane16 + j * 1024, mv.soff, AUX_SYS);
        }
        asm volatile("s_waitcnt vmcnt(4)" : "+v"(tk) :: "memory");
        __builtin_amdgcn_raw_buffer_store_b32(seq, rflag, OOB, 0, AUX_SYS);
        t = __builtin_amdgcn_readfirstlane(tk);
        while (t + SNAP_AT < NT) {
          const __amdgpu_buffer_rsrc_t rsrc = prep(ts, t);
          if (failed) break;
          while (t + SNAP_AT < NT && t < known) {
            const Move nx = decode(t, ts, rank);
#pragma unroll
            for (int j = 0; j < 4; ++j) __builtin_amdgcn_raw_buffer_store_b128(cur[j], rdst, mv.doff + lane16 + j * 1024, 0, AUX_SYS);
            draw(ts);
#pragma unroll
            for (int j = 0; j < 4; ++j)
              cur[j] = __builtin_amdgcn_raw_buffer_load_b128(rsrc, lane16 + j * 1024, nx.soff, AUX_SYS);
            asm volatile("s_waitcnt vmcnt(4)" : "+v"(tk) :: "memory");
            __builtin_amdgcn_raw_buffer_store_b32(seq, rflag, lane == 0 ? mv.foff : (lane == 1 ? mv.goff : OOB), 0, AUX_SYS);
            t = __builtin_amdgcn_readfirstlane(tk);
            mv = nx;
          }
        }
        if (failed) break;
        while (map(ts, t, us, ut)) {
          const Move nx = decode(ut, us, rank);
          const __amdgpu_buffer_rsrc_t rsrc = prep(us, ut);
          if (failed) break;
          if (t + SNAP_AT == NT) take_snapshot();
#pragma unroll
          for (int j = 0; j < 4; ++j) __builtin_amdgcn_raw_buffer_store_b128(cur[j], rdst, mv.doff + lane16 + j * 1024, 0, AUX_SYS);
          draw(ts);
          unsigned sv = 0, vv = 0, wantv = 8;
          const bool want_plan = !planned && t + 112 >= NT;
          if (want_plan) sv = ld_agt(snap);
          if (planned && ts == src0 && t + 2 * PL >= NT && __builtin_amdgcn_readlane(p_ext, src0) != 0) {
            const unsigned g = __builtin_amdgcn_readlane(p_xs, src0);
            const unsigned m = (unsigned)__ballot(g >= p_es && g - p_es < NT - p_own) & 0xffu & ~tried;
            if (m) { wantv = __builtin_ctz(m); vv = ld_sys(dense_ptr(wantv)); }
          }
#pragma unroll
          for (int j = 0; j < 4; ++j)
            cur[j] = __builtin_amdgcn_raw_buffer_load_b128(rsrc, lane16 + j * 1024, nx.soff, AUX_SYS);
          asm volatile("s_waitcnt vmcnt(4)" : "+v"(tk) :: "memory");
          __builtin_amdgcn_raw_buffer_store_b32(seq, rflag, lane == 0 ? mv.foff : (lane == 1 ? mv.goff : OOB), 0, AUX_SYS);
          t = __builtin_amdgcn_readfirstlane(tk);
          mv = nx;
          if (want_plan) make_plan(sv);
          if (wantv < 8) {
            availv |= (vv >= seq ? 1u : 0u) << wantv;
            tried |= 1u << wantv;
          }
        }
        if (failed) break;
#pragma unroll
        for (int j = 0; j < 4; ++j) __builtin_amdgcn_raw_buffer_store_b128(cur[j], rdst, mv.doff + lane16 + j * 1024, 0, AUX_SYS);
        asm volatile("s_waitcnt vmcnt(0)" ::: "memory");
        __builtin_amdgcn_raw_buffer_store_b32(seq, rflag, lane == 0 ? mv.foff : (lane == 1 ? mv.goff : OOB), 0, AUX_SYS);
        asm volatile(".Lpf_pull_end:");
        if (hop == 0) STAMP(4);
        STAMP(3);
      }
    }
  }
}

}

extern "C" void kernel_launch(int rank, int nranks, ncclComm_t comm,
                              hipStream_t stream,
                              void* const* d_in, const int* in_sizes, int n_in,
                              void* d_out, int out_size,
                              const int* in_dtypes, int out_dtype,
                              void* const* d_out_all,
                              void* const* d_ws_all, size_t ws_bytes) {
  (void)comm; (void)in_dtypes; (void)out_dtype; (void)d_out_all;
  if (nranks != NR || (unsigned)rank >= (unsigned)NR || n_in != 4 || ws_bytes < WS_USED ||
      in_sizes[0] != NR * MS * KS || in_sizes[1] != KS * NN || out_size != MS * NN) {
    fprintf(stderr, "unexpected problem geometry\n");
    return;
  }
  Params p;
  p.x = (const int8_t*)d_in[0];
  p.w = (const int8_t*)d_in[1];
  p.sx = (const float*)d_in[2];
  p.sw = (const float*)d_in[3];
  p.out = (float*)d_out;
  for (int i = 0; i < NR; ++i) p.ws[i] = (char*)d_ws_all[i];
  p.rank = rank;
  p.my = p.ws[rank];
  p.pad = 0;
  p.pad2 = nullptr;
  fused_kernel<<<GRID, BLOCK, 0, stream>>>(p);
  HIP_CHECK(hipGetLastError());
}
```

```cpp
#include <hip/hip_runtime.h>
#include <rccl/rccl.h>
#include <cstdio>
#include <cstdint>

#define HIP_CHECK(expr) do { hipError_t _e = (expr); if (_e != hipSuccess) { \
    fprintf(stderr, "HIP error %s at %s:%d\n", hipGetErrorString(_e), __FILE__, __LINE__); \
    return; } } while (0)

namespace {

typedef int v4i __attribute__((ext_vector_type(4)));
typedef unsigned v4u __attribute__((ext_vector_type(4)));
typedef float v4f __attribute__((ext_vector_type(4)));

constexpr int NR = 8;
constexpr int NN = 4096;
constexpr int MS = 512;
constexpr int KS = 512;
constexpr unsigned UNIT = 4096;
constexpr unsigned SHARD = 2u << 20;
constexpr unsigned XBLK = 256u << 10;

#define TM 48
#define PL 14
#define PWIN 3
#define WDELAY 48
#define XSPLIT 6
#define RW 128
#define XAT 80
#define SNAP_AT (RW + 64)
#define PLAN_DEAD 2
#define HELP_MIN 64
#define POLLW 2
#define AFFINE 1

constexpr unsigned OFF_STG = 0;
constexpr unsigned OFF_X = 4u << 20;
constexpr unsigned OFF_WG = 8u << 20;
constexpr unsigned WGB = NR * SHARD;
constexpr unsigned OFF_FLG = 40u << 20;
constexpr unsigned OFF_WFLAG = OFF_FLG;
constexpr unsigned OFF_DENSE = OFF_WFLAG + 512 * 128;
constexpr unsigned OFF_SNAP = OFF_DENSE + 8 * 512 * 4;
constexpr unsigned OFF_SOWN = OFF_SNAP + 128;
constexpr unsigned OFF_XFLAG = OFF_SOWN + 128;
constexpr unsigned OFF_SFLAG = OFF_XFLAG + 128 * 128;
constexpr unsigned OFF_CTR = OFF_SFLAG + 1024 * 128;
constexpr unsigned OFF_QCTR = OFF_CTR + 256 * 128;
constexpr unsigned OFF_TMO = OFF_QCTR + 16 * 128;
constexpr unsigned WS_USED = OFF_TMO + 128;
static_assert(OFF_WG + 2 * WGB <= OFF_FLG, "layout");

constexpr unsigned NT = 512;
constexpr int GRID = 256;
constexpr int NWAVE = 6;
constexpr int BLOCK = NWAVE * 64;
constexpr int MF = TM / 16;
constexpr unsigned long long DEADLINE = 100000000ull;

constexpr int AUX_SYS = 17;
constexpr int AUX_AGT = 16;
constexpr unsigned OOB = 0xfffffff0u;

#define STAMP(i) do {} while (0)

struct Params {
  const int8_t* x;
  const int8_t* w;
  char* my;
  int rank;
  int pad;
  const float* sx;
  const float* sw;
  float* out;
  void* pad2;
  char* ws[NR];
};

__device__ __forceinline__ __amdgpu_buffer_rsrc_t mk_rsrc(const void* p, unsigned bytes) {
  return __builtin_amdgcn_make_buffer_rsrc(const_cast<void*>(p), 0, bytes, 0x00020000);
}
__device__ __forceinline__ unsigned ld_sys(const unsigned* p) {
  return __hip_atomic_load(p, __ATOMIC_RELAXED, __HIP_MEMORY_SCOPE_SYSTEM);
}
__device__ __forceinline__ void st_sys(unsigned* p, unsigned v) {
  __hip_atomic_store(p, v, __ATOMIC_RELAXED, __HIP_MEMORY_SCOPE_SYSTEM);
}
__device__ __forceinline__ unsigned ld_agt(const unsigned* p) {
  return __hip_atomic_load(p, __ATOMIC_RELAXED, __HIP_MEMORY_SCOPE_AGENT);
}
__device__ __forceinline__ void st_agt(unsigned* p, unsigned v) {
  __hip_atomic_store(p, v, __ATOMIC_RELAXED, __HIP_MEMORY_SCOPE_AGENT);
}
__device__ __forceinline__ unsigned ld_lds(const unsigned* p) {
  return __hip_atomic_load(p, __ATOMIC_RELAXED, __HIP_MEMORY_SCOPE_WORKGROUP);
}
__device__ __forceinline__ void st_lds(unsigned* p, unsigned v) {
  __hip_atomic_store(p, v, __ATOMIC_RELAXED, __HIP_MEMORY_SCOPE_WORKGROUP);
}

__device__ __forceinline__ unsigned slot_of(unsigned nb, unsigned ks) { return (nb >> 3) * 64 + ks * 8 + (nb & 7); }
__device__ __forceinline__ unsigned ord_dist(unsigned ord, unsigned nb) {
  const unsigned qn = nb >> 3;
  unsigned dd = (qn + 9 - ord) & 7;
  if (ord - 1 >= qn) dd = (dd + 7) & 7;
  return ord == 0 ? 0 : dd;
}

struct Move { unsigned soff, doff, foff, goff; };
__device__ __forceinline__ Move decode(unsigned t, unsigned src, unsigned rank) {
  const unsigned slot = (t + 64 * ((src - rank) & 7)) & 511;
  const unsigned nb = (slot >> 6) * 8 + (slot & 7), ks = (slot >> 3) & 7;
  Move m;
  m.soff = slot * UNIT;
  m.doff = src * SHARD + slot * UNIT;
  m.foff = (OFF_WFLAG - OFF_FLG) + ((nb * 8 + src) * 32 + ks) * 4;
  m.goff = (OFF_DENSE - OFF_FLG) + (src * 512 + slot) * 4;
  return m;
}

__global__ void __launch_bounds__(BLOCK, 2) fused_kernel(Params p) {
  __shared__ unsigned lds[21 * 1024];
  asm volatile("" :: "s"(p.x), "s"(p.w), "s"(p.my), "s"(p.rank), "s"(p.ws[0]));
  const unsigned b = blockIdx.x;
  const unsigned lane = threadIdx.x & 63;
  const unsigned wid = __builtin_amdgcn_readfirstlane(threadIdx.x >> 6);
  const unsigned rank = p.rank;
  char* const my = p.my;
  const unsigned role = (b >> 3) & 1;
  const unsigned idx = ((b >> 4) << 3) | (b & 7);
  const unsigned q = lane >> 4, c = lane & 15;

  unsigned* const ctr = (unsigned*)(my + OFF_CTR) + b * 32 + wid;
  unsigned* const tmo = (unsigned*)(my + OFF_TMO);
  const unsigned long long t_start = __builtin_amdgcn_s_memrealtime();
  unsigned seq;

  STAMP(0);
  if (role == 0 && wid == 0) {
    const unsigned xg = idx >> 3;
    const unsigned dst = (rank + (idx & 7) + xg) & 7;
    v4u v[4][4];
    const int8_t* src = p.x + (size_t)(dst * MS + (xg >> 1) * 64 + c) * KS + (xg & 1) * 256 + q * 16;
    const unsigned cv = ld_agt(ctr);
    asm volatile("" ::: "memory");
#pragma unroll
    for (int u = 0; u < 4; ++u) {
#pragma unroll
      for (int i = 0; i < 4; ++i) v[u][i] = *(const v4u*)(src + i * 16 * KS + u * 64);
      if (XSPLIT > 0 && u == 1) { asm volatile("" ::: "memory"); __builtin_amdgcn_s_sleep(XSPLIT); asm volatile("" ::: "memory"); }
    }
    char* const dws = p.ws[dst];
    __builtin_amdgcn_s_barrier();
    seq = __builtin_amdgcn_readfirstlane(cv) + 1;
    const __amdgpu_buffer_rsrc_t rs = mk_rsrc(dws + OFF_X + (seq & 1) * (NR * XBLK) + rank * XBLK, XBLK);
    const unsigned off = xg * 4 * UNIT + lane * 16;
#pragma unroll
    for (int k = 0; k < 16; ++k) {
      if (k >= PWIN) asm volatile("s_waitcnt vmcnt(%0)" :: "n"(PWIN - 1) : "memory");
      __builtin_amdgcn_raw_buffer_store_b128(v[k >> 2][k & 3], rs, off + (k & 3) * 1024, (k >> 2) * UNIT, AUX_SYS);
    }
    asm volatile("s_waitcnt vmcnt(0)" ::: "memory");
    STAMP(1);
    if (lane == 0) {
      st_sys((unsigned*)(dws + OFF_XFLAG) + (((xg >> 1) * 8 + rank) * 2 + (xg & 1)) * 32, seq);
      st_agt(ctr, seq);
    }
  } else if (role == 1 && wid < 4) {
    const unsigned sslot = idx * 4 + wid;
    const unsigned snb = (sslot >> 6) * 8 + (sslot & 7), sks = (sslot >> 3) & 7;
    if (WDELAY > 0) __builtin_amdgcn_s_sleep(WDELAY);
    char* const peer_ws = p.ws[(rank + lane) & (NR - 1)];
    unsigned d[16];
    {
      const unsigned* src = (const unsigned*)(p.w + (size_t)(sks * 64 + q * 16) * NN + snb * 64 + c * 4);
#pragma unroll
      for (int r = 0; r < 16; ++r) d[r] = src[r * (NN / 4)];
    }
    const unsigned cv = ld_agt(ctr);
    __builtin_amdgcn_s_barrier();
    seq = __builtin_amdgcn_readfirstlane(cv) + 1;
    v4u o[4];
#pragma unroll
    for (int g = 0; g < 4; ++g) {
      const unsigned t0 = __builtin_amdgcn_perm(d[4 * g + 1], d[4 * g + 0], 0x05010400u);
      const unsigned t1 = __builtin_amdgcn_perm(d[4 * g + 1], d[4 * g + 0], 0x07030602u);
      const unsigned t2 = __builtin_amdgcn_perm(d[4 * g + 3], d[4 * g + 2], 0x05010400u);
      const unsigned t3 = __builtin_amdgcn_perm(d[4 * g + 3], d[4 * g + 2], 0x07030602u);
      o[0][g] = __builtin_amdgcn_perm(t2, t0, 0x05040100u);
      o[1][g] = __builtin_amdgcn_perm(t2, t0, 0x07060302u);
      o[2][g] = __builtin_amdgcn_perm(t3, t1, 0x05040100u);
      o[3][g] = __builtin_amdgcn_perm(t3, t1, 0x07060302u);
    }
    const __amdgpu_buffer_rsrc_t rs = mk_rsrc(my + OFF_STG + (seq & 1) * SHARD, SHARD);
    const unsigned off = sslot * UNIT + lane * 16;
#pragma unroll
    for (int j = 0; j < 4; ++j) __builtin_amdgcn_raw_buffer_store_b128(o[j], rs, off + j * 1024, 0, AUX_SYS);
    asm volatile("s_waitcnt vmcnt(0)" ::: "memory");
    STAMP(1);
    unsigned old = 0;
    if (lane == 0) old = __hip_atomic_fetch_add(&lds[3], 1u, __ATOMIC_RELAXED, __HIP_MEMORY_SCOPE_WORKGROUP);
    old = __builtin_amdgcn_readfirstlane(old);
    if (old == 3 && lane >= 1 && lane < NR)
      st_sys((unsigned*)(peer_ws + OFF_SFLAG) + (rank * 128 + idx) * 32, seq);
    if (lane == 0) {
      st_agt((unsigned*)(my + OFF_WFLAG) + (snb * 8 + rank) * 32 + sks, seq);
      st_agt(ctr, seq);
    }
  } else {
    if (wid == 4 && lane < 4) st_lds(&lds[lane], 0);
    asm volatile("s_waitcnt lgkmcnt(0)" ::: "memory");
    __builtin_amdgcn_s_barrier();
    asm volatile("" ::: "memory");
    if (wid == 5 && b < 8) {
      unsigned long long pc;
      int la, lb;
      asm volatile("s_getpc_b64 %0\n.Lpf_here:\n\ts_mov_b32 %1, .Lpf_gemm_end-.Lpf_here\n\ts_mov_b32 %2, .Lpf_pull_end-.Lpf_here"
                   : "=s"(pc), "=s"(la), "=s"(lb));
      const unsigned len = (unsigned)max(max(la, lb), 0);
      if (len < 32768u) {
        const __amdgpu_buffer_rsrc_t rc = mk_rsrc((const void*)pc, len & ~15u);
        v4u acc = {0, 0, 0, 0};
#pragma unroll
        for (int i = 0; i < 4; ++i) acc |= __builtin_amdgcn_raw_buffer_load_b128(rc, (lane + i * 64) * 128, 0, 0);
        asm volatile("" :: "v"(acc));
      }
    }
    seq = __builtin_amdgcn_readfirstlane(ld_agt(ctr)) + 1;
    if (lane == 0) st_agt(ctr, seq);
    if (b == 16 && wid == 5 && lane < 8) st_agt((unsigned*)(my + OFF_QCTR) + (((seq & 1) ^ 1) * 8 + lane) * 32, 0);
  }
  seq = __builtin_amdgcn_readfirstlane(seq);
  const unsigned par = seq & 1;

  const unsigned part = TM == 64 ? ((b >> 3) & 3) == 0 ? 0 : ((b >> 3) & 3) == 2 ? 1 : 3 : (b >> 3) & 3;
  const bool gemm_wg = part < 3;
  const unsigned nb = (b >> 5) * 8 + (b & 7);
  const unsigned f0 = part * (TM == 64 ? 16 : 11);
  const unsigned f1 = min(f0 + (TM == 64 ? 16 : 11), 32u);
  const bool mover_wave = TM == 64 ? (role == 1 && wid == 5) : (part == 3 && wid >= 4);

  if (wid == 4 && gemm_wg) {
    {
      const unsigned psrc = (rank + ord_dist(lane >> 3, nb)) & 7;
      const unsigned* const wf = (const unsigned*)(my + OFF_WFLAG) + (nb * 8 + psrc) * 32 + (lane & 7);
      const unsigned xmb = min((f0 >> 2) + ((lane >> 1) & 3), (f1 - 1) >> 2);
      const unsigned* const xf = (const unsigned*)(my + OFF_XFLAG) + ((xmb * 8 + psrc) * 2 + (lane & 1)) * 32;
      bool wok = false, xok = false;
      unsigned long long last = 0;
      for (unsigned spin = 0;; ++spin) {
        const unsigned front = (unsigned)__builtin_ctzll(~last) >> 3;
        const bool near = (lane >> 3) < front + POLLW;
        if (!wok && near) wok = ld_agt(wf) >= seq;
        if (!xok && near) xok = ld_sys(xf) >= seq;
        asm volatile("" ::: "memory");
        const unsigned long long xm = __ballot(xok);
        const unsigned long long m = __ballot(wok && ((xm >> (lane & 56)) & 0xff) == 0xff);
        if (m != last) {
          last = m;
          if (lane < 2) st_lds(&lds[lane], lane ? (unsigned)(m >> 32) : (unsigned)m);
          if (m == ~0ull) break;
        }
        if ((spin & 63) == 63 &&
            (ld_agt(tmo) == seq || __builtin_amdgcn_s_memrealtime() - t_start > DEADLINE)) {
          st_agt(tmo, seq);
          if (lane == 0) st_lds(&lds[2], 1);
          asm volatile("s_waitcnt lgkmcnt(0)" ::: "memory");
          if (lane < 2) st_lds(&lds[lane], 0xffffffffu);
          break;
        }
        __builtin_amdgcn_s_sleep(1);
      }
    }
  } else if (wid < 4) {
    if (gemm_wg) {
      const unsigned wf0 = f0 + wid * MF;
      unsigned avo[MF];
#pragma unroll
      for (int i = 0; i < MF; ++i)
        avo[i] = wf0 + i < f1 ? ((wf0 + i) >> 2) * (8 * UNIT) + ((wf0 + i) & 3) * 1024 + lane * 16 : 0x80000000u;
      const float scale0 = p.sx[0] * p.sw[0];
      v4i acc[MF][4];
#pragma unroll
      for (int i = 0; i < MF; ++i)
#pragma unroll
        for (int j = 0; j < 4; ++j) acc[i][j] = v4i{0, 0, 0, 0};
      v4i ra[4][MF], rb[4][4];
#pragma unroll
      for (int s = 0; s < 4; ++s) {
#pragma unroll
        for (int i = 0; i < MF; ++i) ra[s][i] = v4i{0, 0, 0, 0};
#pragma unroll
        for (int j = 0; j < 4; ++j) rb[s][j] = v4i{0, 0, 0, 0};
      }

      unsigned tmo_val = 0;
      const __amdgpu_buffer_rsrc_t rws = mk_rsrc(my, OFF_FLG);
      for (unsigned ord = 0; ord < 8; ++ord) {
        const unsigned src = (rank + ord_dist(ord, nb)) & 7;
        const unsigned aoff = OFF_X + par * (NR * XBLK) + src * XBLK;
        const unsigned boff = (src == rank ? OFF_STG + par * SHARD : OFF_WG + par * WGB + src * SHARD) + slot_of(nb, 0) * UNIT;
        const unsigned* const mword = &lds[ord >> 2];
        const unsigned sh = (ord & 3) * 8;
        unsigned ready = 0;
#pragma unroll
        for (int ks = 0; ks < 8; ++ks) {
          while (!((ready >> ks) & 1)) {
            ready = __builtin_amdgcn_readfirstlane(ld_lds(mword)) >> sh;
            if ((ready >> ks) & 1) break;
            __builtin_amdgcn_s_sleep(1);
          }
          asm volatile("" ::: "memory");
          if (ks == 7) STAMP(3 + ord);
          const int s = ks & 3;
#pragma unroll
          for (int j = 0; j < 4; ++j)
            rb[s][j] = (v4i)__builtin_amdgcn_raw_buffer_load_b128(rws, lane * 16 + j * 1024, boff + ks * (8 * UNIT), 0);
#pragma unroll
          for (int i = 0; i < MF; ++i)
            ra[s][i] = (v4i)__builtin_amdgcn_raw_buffer_load_b128(rws, avo[i], aoff + ks * UNIT, 0);
          if (ks == 7) tmo_val = ld_agt(tmo);
          const int s3 = (ks + 1) & 3;
#pragma unroll
          for (int i = 0; i < MF; ++i)
#pragma unroll
            for (int j = 0; j < 4; ++j)
              acc[i][j] = __builtin_amdgcn_mfma_i32_16x16x64_i8(ra[s3][i], rb[s3][j], acc[i][j], 0, 0, 0);
        }
      }
#pragma unroll
      for (int s = 1; s < 4; ++s)
#pragma unroll
        for (int i = 0; i < MF; ++i)
#pragma unroll
          for (int j = 0; j < 4; ++j)
            acc[i][j] = __builtin_amdgcn_mfma_i32_16x16x64_i8(ra[s][i], rb[s][j], acc[i][j], 0, 0, 0);

      STAMP(11);
      const bool failed = ld_lds(&lds[2]) != 0;
      const float scale = (failed || tmo_val == seq) ? __builtin_nanf("") : scale0;
      const __amdgpu_buffer_rsrc_t ro = mk_rsrc(p.out, (unsigned)MS * NN * 4);
#pragma unroll
      for (int i = 0; i < MF; ++i)
#pragma unroll
        for (int r = 0; r < 4; ++r) {
          v4f v;
#pragma unroll
          for (int j = 0; j < 4; ++j) v[j] = (float)acc[i][j][r] * scale;
          const unsigned row = (wf0 + i) * 16 + q * 4 + r;
          const unsigned off = wf0 + i < f1 ? (row * NN + nb * 64 + c * 4) * 4 : OOB;
          __builtin_amdgcn_raw_buffer_store_b128((v4u)v, ro, off, 0, AUX_AGT);
        }
      asm volatile(".Lpf_gemm_end:");
    }
  } else if (mover_wave) {
    const unsigned xcc = __builtin_amdgcn_s_getreg((3 << 11) | 20) & 7;
    const unsigned row = TM == 64 ? b >> 4 : (wid - 4) * 8 + (b >> 5);
    const unsigned CLQ = 0xB2u;
    const unsigned same = (((CLQ >> rank) & 1) ? CLQ : (~CLQ & 0xffu)) & ~(1u << rank);
    const unsigned other = ~same & ~(1u << rank) & 0xffu;
    unsigned cand = AFFINE == 1 ? ((xcc & 4) ? other : same) : ((xcc & 4) ? same : other);
    unsigned li = (row + xcc) & 3;
    if (AFFINE == 0) { cand = 0xffu & ~(1u << rank); li = (row + xcc) & 7; }
    unsigned src0 = 8;
    {
      unsigned m = cand;
      for (unsigned i = 0; i < li && m; ++i) m &= m - 1;
      if (m) src0 = __builtin_ctz(m);
    }
    if (row < PL && src0 < 8) {
      unsigned* const qh0 = (unsigned*)(my + OFF_QCTR) + par * (8 * 32);
      const __amdgpu_buffer_rsrc_t rdst = mk_rsrc(my + OFF_WG + par * WGB, WGB);
      const __amdgpu_buffer_rsrc_t rflag = mk_rsrc(my + OFF_FLG, OFF_XFLAG - OFF_FLG);
      unsigned vzero = 0, vone = 1;
      asm volatile("" : "+v"(vzero), "+v"(vone));
      bool failed = false;
      const unsigned lane16 = lane * 16;
      char* const qws = p.ws[src0];
      unsigned tk = 0;
      unsigned resp = 1u << src0;
      unsigned availv = 0;
      unsigned tried = 0;
      unsigned ksrc = 8, known = 0;
      bool planned = false;
      bool need_scan = false;
      unsigned p_e = 0, p_ext = 0, p_xs = 0, p_es = 0;
      const unsigned* const snap = (const unsigned*)(my + OFF_SNAP) + (lane & 7);

      auto timed_out = [&]() {
        if (ld_agt(tmo) == seq || __builtin_amdgcn_s_memrealtime() - t_start > DEADLINE) {
          st_agt(tmo, seq);
          failed = true;
        }
        return failed;
      };
      auto make_plan = [&](unsigned sv) {
        const unsigned s8 = lane & 7;
        if (__ballot(s8 == rank || sv >= seq * 1024) != ~0ull) return;
        const unsigned h = s8 == rank ? 0 : sv - seq * 1024;
        unsigned sum = h;
        sum += __shfl_xor((int)sum, 1); sum += __shfl_xor((int)sum, 2); sum += __shfl_xor((int)sum, 4);
        const unsigned a = sum / 7;
        const unsigned e = (s8 != rank && h + PLAN_DEAD < a) ? min(a - h - PLAN_DEAD, RW - 8u) : 0;
        const unsigned sp = (s8 != rank && s8 != ((rank + 1) & 7) && h > a) ? h - a : 0;
        need_scan = __ballot(s8 != rank && (h == 0 || h + PLAN_DEAD + RW - 8u < a)) != 0;
        unsigned pe = e, ps = sp;
#pragma unroll
        for (int o = 1; o < 8; o <<= 1) {
          const unsigned ue = __shfl_up((int)pe, o, 8), us = __shfl_up((int)ps, o, 8);
          if (s8 >= (unsigned)o) { pe += ue; ps += us; }
        }
        const unsigned tot = min((unsigned)__shfl((int)pe, 7, 8), (unsigned)__shfl((int)ps, 7, 8));
        p_es = pe - e;
        p_xs = ps - sp;
        p_e = tot > p_es ? min(tot - p_es, e) : 0;
        p_ext = tot > p_xs ? min(tot - p_xs, sp) : 0;
        planned = true;
      };
      auto take_snapshot = [&]() {
        unsigned old = seq;
        if (lane == 0) old = __hip_atomic_fetch_max((unsigned*)(my + OFF_SOWN), seq, __ATOMIC_RELAXED, __HIP_MEMORY_SCOPE_AGENT);
        if (__builtin_amdgcn_readfirstlane(old) < seq) {
          const unsigned hv = ld_agt(qh0 + (lane & 7) * 32);
          __builtin_amdgcn_raw_buffer_store_b32(seq * 1024 + min(hv, NT), rflag,
                                                lane < 8 ? (OFF_SNAP - OFF_FLG) + lane * 4 : OOB, 0, AUX_AGT);
        }
        asm volatile("s_waitcnt vmcnt(0)" ::: "memory");
      };
      auto map = [&](unsigned qs, unsigned tt, unsigned& us, unsigned& ut) -> bool {
        us = qs; ut = tt;
        if (tt + RW < NT) return true;
        for (unsigned spin = 0; !planned; ++spin) {
          make_plan(ld_agt(snap));
          if (planned) break;
          if (spin == 0) take_snapshot();
          if ((spin & 63) == 63 && timed_out()) return false;
          __builtin_amdgcn_s_sleep(2);
        }
        const unsigned ex = __builtin_amdgcn_readlane(p_ext, qs);
        if (tt + XAT >= NT && tt + XAT - NT < ex) {
          const unsigned g = __builtin_amdgcn_readlane(p_xs, qs) + (tt + XAT - NT);
          const unsigned m = (unsigned)__ballot(g >= p_es && g - p_es < p_e) & 0xffu;
          us = __builtin_ctz(m | 0x80u);
          ut = NT - RW + (g - __builtin_amdgcn_readlane(p_es, us));
          return true;
        }
        ut = tt + __builtin_amdgcn_readlane(p_e, qs) - (tt + XAT >= NT ? ex : 0);
        return ut < NT;
      };
      auto draw = [&](unsigned qs) {
        unsigned* const qh = qh0 + qs * 32;
        if (lane == 0)
          asm volatile("global_atomic_add %0, %1, %2, %3 sc0" : "=v"(tk) : "v"(vzero), "v"(vone), "s"(qh) : "memory");
      };
      auto dense_ptr = [&](unsigned s, unsigned back) {
        return (const unsigned*)(qws + OFF_DENSE) + s * 512 + ((512 - back + 64 * ((s - rank) & 7) + lane) & 511);
      };
      auto prep = [&](unsigned s, unsigned tt) {
        const unsigned hb = tt + 64 >= NT ? s : s + 8;
        if (s != src0 && tt + RW >= NT && !((tried >> hb) & 1)) {
          availv |= (ld_sys(dense_ptr(s, hb < 8 ? 64 : 128)) >= seq ? 1u : 0u) << hb;
          tried |= 1u << hb;
        }
        const bool rel = s != src0 && tt + RW >= NT && ((__builtin_amdgcn_readlane(availv, tt & 63) >> hb) & 1);
        if (!rel && (s != ksrc || tt >= known)) {
          const unsigned sd = (s - rank) & 7;
          const unsigned* const sf = (const unsigned*)(my + OFF_SFLAG) + s * (128 * 32);
          const unsigned g0 = ((lane + 16 * sd) & 127) * 32, g1 = ((lane + 64 + 16 * sd) & 127) * 32;
          ksrc = s;
          for (unsigned spin = 0;; ++spin) {
            const unsigned f0 = ld_sys(sf + g0), f1 = ld_sys(sf + g1);
            const unsigned long long m0 = __ballot(f0 >= seq), m1 = __ballot(f1 >= seq);
            known = 4 * (~m0 ? (unsigned)__builtin_ctzll(~m0) : 64 + (~m1 ? (unsigned)__builtin_ctzll(~m1) : 64));
            if (tt < known) break;
            if ((spin & 63) == 63 && timed_out()) break;
            __builtin_amdgcn_s_sleep(1);
          }
          asm volatile("" ::: "memory");
        }
        return mk_rsrc(rel ? qws + OFF_WG + par * WGB + s * SHARD : p.ws[s] + OFF_STG + par * SHARD, SHARD);
      };

      unsigned ts = src0;
      for (unsigned hop = 0; !failed; ++hop) {
        if (hop > 0) {
          if (!need_scan) break;
          const unsigned s8 = lane & 7;
          const unsigned hv = ld_agt(qh0 + s8 * 32);
          const unsigned lim = NT + p_ext - p_e;
          const bool open = s8 != rank && (hv + HELP_MIN < lim || (((resp >> s8) & 1) && hv < lim));
          unsigned key = (open ? 0xfffu - (lim - hv) : 0xffffu) * 8 + s8;
          key = min(key, (unsigned)__shfl_xor((int)key, 1));
          key = min(key, (unsigned)__shfl_xor((int)key, 2));
          key = min(key, (unsigned)__shfl_xor((int)key, 4));
          key = __builtin_amdgcn_readfirstlane(key);
          if ((key >> 3) >= 0xfffu) break;
          ts = key & 7;
          resp |= 1u << ts;
        }
        draw(ts);
        asm volatile("s_waitcnt vmcnt(0)" : "+v"(tk) :: "memory");
        unsigned t = __builtin_amdgcn_readfirstlane(tk);
        unsigned us, ut;
        if (!map(ts, t, us, ut)) continue;
        Move mv = decode(ut, us, rank);
        v4u cur[4];
        if (hop == 0 && ut < 64) {
          const unsigned* const f1 = (const unsigned*)(my + OFF_SFLAG) + (us * 128 + (mv.soff >> 14)) * 32;
          for (unsigned spin = 0; spin < 4096; ++spin)
            if (__builtin_amdgcn_readfirstlane(ld_sys(f1)) >= seq) { ksrc = us; known = ut + 1; break; }
          asm volatile("" ::: "memory");
        }
        {
          const __amdgpu_buffer_rsrc_t rsrc = prep(us, ut);
          if (failed) break;
          if (hop == 0) STAMP(2);
          draw(ts);
#pragma unroll
          for (int j = 0; j < 4; ++j)
            cur[j] = __builtin_amdgcn_raw_buffer_load_b128(rsrc, lane16 + j * 1024, mv.soff, AUX_SYS);
        }
        asm volatile("s_waitcnt vmcnt(4)" : "+v"(tk) :: "memory");
        __builtin_amdgcn_raw_buffer_store_b32(seq, rflag, OOB, 0, AUX_SYS);
        t = __builtin_amdgcn_readfirstlane(tk);
        while (t + SNAP_AT < NT) {
          const __amdgpu_buffer_rsrc_t rsrc = prep(ts, t);
          if (failed) break;
          while (t + SNAP_AT < NT && t < known) {
            const Move nx = decode(t, ts, rank);
#pragma unroll
            for (int j = 0; j < 4; ++j) __builtin_amdgcn_raw_buffer_store_b128(cur[j], rdst, mv.doff + lane16 + j * 1024, 0, AUX_SYS);
            draw(ts);
#pragma unroll
            for (int j = 0; j < 4; ++j)
              cur[j] = __builtin_amdgcn_raw_buffer_load_b128(rsrc, lane16 + j * 1024, nx.soff, AUX_SYS);
            asm volatile("s_waitcnt vmcnt(4)" : "+v"(tk) :: "memory");
            __builtin_amdgcn_raw_buffer_store_b32(seq, rflag, lane == 0 ? mv.foff : (lane == 1 ? mv.goff : OOB), 0, AUX_SYS);
            t = __builtin_amdgcn_readfirstlane(tk);
            mv = nx;
          }
        }
        if (failed) break;
        while (map(ts, t, us, ut)) {
          const Move nx = decode(ut, us, rank);
          const __amdgpu_buffer_rsrc_t rsrc = prep(us, ut);
          if (failed) break;
          if (t + SNAP_AT == NT) take_snapshot();
#pragma unroll
          for (int j = 0; j < 4; ++j) __builtin_amdgcn_raw_buffer_store_b128(cur[j], rdst, mv.doff + lane16 + j * 1024, 0, AUX_SYS);
          draw(ts);
          unsigned sv = 0, vv = 0, wantv = 16;
          const bool want_plan = !planned && t + SNAP_AT - 16 >= NT;
          if (want_plan) sv = ld_agt(snap);
          if (planned && ts == src0 && t + XAT + 2 * PL >= NT && __builtin_amdgcn_readlane(p_ext, src0) != 0) {
            const unsigned g = __builtin_amdgcn_readlane(p_xs, src0);
            const unsigned m = (unsigned)__ballot(g >= p_es && g - p_es < p_e) & 0xffu;
            if (m) {
              const unsigned s1 = __builtin_ctz(m);
              const unsigned t1 = NT - RW + (g - __builtin_amdgcn_readlane(p_es, s1));
              const unsigned hb = t1 + 64 >= NT ? s1 : s1 + 8;
              if (!((tried >> hb) & 1)) { wantv = hb; vv = ld_sys(dense_ptr(s1, hb < 8 ? 64 : 128)); }
            }
          }
#pragma unroll
          for (int j = 0; j < 4; ++j)
            cur[j] = __builtin_amdgcn_raw_buffer_load_b128(rsrc, lane16 + j * 1024, nx.soff, AUX_SYS);
          asm volatile("s_waitcnt vmcnt(4)" : "+v"(tk) :: "memory");
          __builtin_amdgcn_raw_buffer_store_b32(seq, rflag, lane == 0 ? mv.foff : (lane == 1 ? mv.goff : OOB), 0, AUX_SYS);
          t = __builtin_amdgcn_readfirstlane(tk);
          mv = nx;
          if (want_plan) make_plan(sv);
          if (wantv < 16) {
            availv |= (vv >= seq ? 1u : 0u) << wantv;
            tried |= 1u << wantv;
          }
        }
        if (failed) break;
#pragma unroll
        for (int j = 0; j < 4; ++j) __builtin_amdgcn_raw_buffer_store_b128(cur[j], rdst, mv.doff + lane16 + j * 1024, 0, AUX_SYS);
        asm volatile("s_waitcnt vmcnt(0)" ::: "memory");
        __builtin_amdgcn_raw_buffer_store_b32(seq, rflag, lane == 0 ? mv.foff : (lane == 1 ? mv.goff : OOB), 0, AUX_SYS);
        asm volatile(".Lpf_pull_end:");
        if (hop == 0) STAMP(4);
        STAMP(3);
      }
    }
  }
}

}

extern "C" void kernel_launch(int rank, int nranks, ncclComm_t comm,
                              hipStream_t stream,
                              void* const* d_in, const int* in_sizes, int n_in,
                              void* d_out, int out_size,
                              const int* in_dtypes, int out_dtype,
                              void* const* d_out_all,
                              void* const* d_ws_all, size_t ws_bytes) {
  (void)comm; (void)in_dtypes; (void)out_dtype; (void)d_out_all;
  if (nranks != NR || (unsigned)rank >= (unsigned)NR || n_in != 4 || ws_bytes < WS_USED ||
      in_sizes[0] != NR * MS * KS || in_sizes[1] != KS * NN || out_size != MS * NN) {
    fprintf(stderr, "unexpected problem geometry\n");
    return;
  }
  Params p;
  p.x = (const int8_t*)d_in[0];
  p.w = (const int8_t*)d_in[1];
  p.sx = (const float*)d_in[2];
  p.sw = (const float*)d_in[3];
  p.out = (float*)d_out;
  for (int i = 0; i < NR; ++i) p.ws[i] = (char*)d_ws_all[i];
  p.rank = rank;
  p.my = p.ws[rank];
  p.pad = 0;
  p.pad2 = nullptr;
  fused_kernel<<<GRID, BLOCK, 0, stream>>>(p);
  HIP_CHECK(hipGetLastError());
}
```

```cpp
#include <hip/hip_runtime.h>
#include <rccl/rccl.h>
#include <cstdio>
#include <cstdint>

#define HIP_CHECK(expr) do { hipError_t _e = (expr); if (_e != hipSuccess) { \
    fprintf(stderr, "HIP error %s at %s:%d\n", hipGetErrorString(_e), __FILE__, __LINE__); \
    return; } } while (0)

namespace {

typedef int v4i __attribute__((ext_vector_type(4)));
typedef unsigned v4u __attribute__((ext_vector_type(4)));
typedef float v4f __attribute__((ext_vector_type(4)));

constexpr int NR = 8;
constexpr int NN = 4096;
constexpr int MS = 512;
constexpr int KS = 512;
constexpr unsigned UNIT = 4096;
constexpr unsigned SHARD = 2u << 20;
constexpr unsigned XBLK = 256u << 10;

#define TM 48
#define PL 14
#define PWIN 3
#define WDELAY 48
#define XSPLIT 6
#define RW 128
#define XAT 80
#define SNAP_AT (RW + 64)
#define PLAN_DEAD 2
#define HELP_MIN 64
#define POLLW 2
#define DYNORD 1
#define AFFINE 1

constexpr unsigned OFF_STG = 0;
constexpr unsigned OFF_X = 4u << 20;
constexpr unsigned OFF_WG = 8u << 20;
constexpr unsigned WGB = NR * SHARD;
constexpr unsigned OFF_FLG = 40u << 20;
constexpr unsigned OFF_WFLAG = OFF_FLG;
constexpr unsigned OFF_DENSE = OFF_WFLAG + 512 * 128;
constexpr unsigned OFF_SNAP = OFF_DENSE + 8 * 512 * 4;
constexpr unsigned OFF_SOWN = OFF_SNAP + 128;
constexpr unsigned OFF_XFLAG = OFF_SOWN + 128;
constexpr unsigned OFF_SFLAG = OFF_XFLAG + 128 * 128;
constexpr unsigned OFF_CTR = OFF_SFLAG + 1024 * 128;
constexpr unsigned OFF_QCTR = OFF_CTR + 256 * 128;
constexpr unsigned OFF_TMO = OFF_QCTR + 16 * 128;
constexpr unsigned WS_USED = OFF_TMO + 128;
static_assert(OFF_WG + 2 * WGB <= OFF_FLG, "layout");

constexpr unsigned NT = 512;
constexpr int GRID = 256;
constexpr int NWAVE = 6;
constexpr int BLOCK = NWAVE * 64;
constexpr int MF = TM / 16;
constexpr unsigned long long DEADLINE = 100000000ull;

constexpr int AUX_SYS = 17;
constexpr int AUX_AGT = 16;
constexpr unsigned OOB = 0xfffffff0u;

#define STAMP(i) do {} while (0)

struct Params {
  const int8_t* x;
  const int8_t* w;
  char* my;
  int rank;
  int pad;
  const float* sx;
  const float* sw;
  float* out;
  void* pad2;
  char* ws[NR];
};

__device__ __forceinline__ __amdgpu_buffer_rsrc_t mk_rsrc(const void* p, unsigned bytes) {
  return __builtin_amdgcn_make_buffer_rsrc(const_cast<void*>(p), 0, bytes, 0x00020000);
}
__device__ __forceinline__ unsigned ld_sys(const unsigned* p) {
  return __hip_atomic_load(p, __ATOMIC_RELAXED, __HIP_MEMORY_SCOPE_SYSTEM);
}
__device__ __forceinline__ void st_sys(unsigned* p, unsigned v) {
  __hip_atomic_store(p, v, __ATOMIC_RELAXED, __HIP_MEMORY_SCOPE_SYSTEM);
}
__device__ __forceinline__ unsigned ld_agt(const unsigned* p) {
  return __hip_atomic_load(p, __ATOMIC_RELAXED, __HIP_MEMORY_SCOPE_AGENT);
}
__device__ __forceinline__ void st_agt(unsigned* p, unsigned v) {
  __hip_atomic_store(p, v, __ATOMIC_RELAXED, __HIP_MEMORY_SCOPE_AGENT);
}
__device__ __forceinline__ unsigned ld_lds(const unsigned* p) {
  return __hip_atomic_load(p, __ATOMIC_RELAXED, __HIP_MEMORY_SCOPE_WORKGROUP);
}
__device__ __forceinline__ void st_lds(unsigned* p, unsigned v) {
  __hip_atomic_store(p, v, __ATOMIC_RELAXED, __HIP_MEMORY_SCOPE_WORKGROUP);
}

__device__ __forceinline__ unsigned slot_of(unsigned nb, unsigned ks) { return (nb >> 3) * 64 + ks * 8 + (nb & 7); }
__device__ __forceinline__ unsigned ord_dist(unsigned ord, unsigned nb) {
  const unsigned qn = nb >> 3;
  unsigned dd = (qn + 9 - ord) & 7;
  if (ord - 1 >= qn) dd = (dd + 7) & 7;
  return ord == 0 ? 0 : dd;
}

struct Move { unsigned soff, doff, foff, goff; };
__device__ __forceinline__ Move decode(unsigned t, unsigned src, unsigned rank) {
  const unsigned slot = (t + 64 * ((src - rank) & 7)) & 511;
  const unsigned nb = (slot >> 6) * 8 + (slot & 7), ks = (slot >> 3) & 7;
  Move m;
  m.soff = slot * UNIT;
  m.doff = src * SHARD + slot * UNIT;
  m.foff = (OFF_WFLAG - OFF_FLG) + ((nb * 8 + src) * 32 + ks) * 4;
  m.goff = (OFF_DENSE - OFF_FLG) + (src * 512 + slot) * 4;
  return m;
}

__global__ void __launch_bounds__(BLOCK, 2) fused_kernel(Params p) {
  __shared__ unsigned lds[21 * 1024];
  asm volatile("" :: "s"(p.x), "s"(p.w), "s"(p.my), "s"(p.rank), "s"(p.ws[0]));
  const unsigned b = blockIdx.x;
  const unsigned lane = threadIdx.x & 63;
  const unsigned wid = __builtin_amdgcn_readfirstlane(threadIdx.x >> 6);
  const unsigned rank = p.rank;
  char* const my = p.my;
  const unsigned role = (b >> 3) & 1;
  const unsigned idx = ((b >> 4) << 3) | (b & 7);
  const unsigned q = lane >> 4, c = lane & 15;

  unsigned* const ctr = (unsigned*)(my + OFF_CTR) + b * 32 + wid;
  unsigned* const tmo = (unsigned*)(my + OFF_TMO);
  const unsigned long long t_start = __builtin_amdgcn_s_memrealtime();
  unsigned seq;

  STAMP(0);
  if (role == 0 && wid == 0) {
    const unsigned xg = idx >> 3;
    const unsigned dst = (rank + (idx & 7) + xg) & 7;
    v4u v[4][4];
    const int8_t* src = p.x + (size_t)(dst * MS + (xg >> 1) * 64 + c) * KS + (xg & 1) * 256 + q * 16;
    const unsigned cv = ld_agt(ctr);
    asm volatile("" ::: "memory");
#pragma unroll
    for (int u = 0; u < 4; ++u) {
#pragma unroll
      for (int i = 0; i < 4; ++i) v[u][i] = *(const v4u*)(src + i * 16 * KS + u * 64);
      if (XSPLIT > 0 && u == 1) { asm volatile("" ::: "memory"); __builtin_amdgcn_s_sleep(XSPLIT); asm volatile("" ::: "memory"); }
    }
    char* const dws = p.ws[dst];
    __builtin_amdgcn_s_barrier();
    seq = __builtin_amdgcn_readfirstlane(cv) + 1;
    const __amdgpu_buffer_rsrc_t rs = mk_rsrc(dws + OFF_X + (seq & 1) * (NR * XBLK) + rank * XBLK, XBLK);
    const unsigned off = xg * 4 * UNIT + lane * 16;
#pragma unroll
    for (int k = 0; k < 16; ++k) {
      if (k >= PWIN) asm volatile("s_waitcnt vmcnt(%0)" :: "n"(PWIN - 1) : "memory");
      __builtin_amdgcn_raw_buffer_store_b128(v[k >> 2][k & 3], rs, off + (k & 3) * 1024, (k >> 2) * UNIT, AUX_SYS);
    }
    asm volatile("s_waitcnt vmcnt(0)" ::: "memory");
    STAMP(1);
    if (lane == 0) {
      st_sys((unsigned*)(dws + OFF_XFLAG) + (((xg >> 1) * 8 + rank) * 2 + (xg & 1)) * 32, seq);
      st_agt(ctr, seq);
    }
  } else if (role == 1 && wid < 4) {
    const unsigned sslot = idx * 4 + wid;
    const unsigned snb = (sslot >> 6) * 8 + (sslot & 7), sks = (sslot >> 3) & 7;
    if (WDELAY > 0) __builtin_amdgcn_s_sleep(WDELAY);
    char* const peer_ws = p.ws[(rank + lane) & (NR - 1)];
    unsigned d[16];
    {
      const unsigned* src = (const unsigned*)(p.w + (size_t)(sks * 64 + q * 16) * NN + snb * 64 + c * 4);
#pragma unroll
      for (int r = 0; r < 16; ++r) d[r] = src[r * (NN / 4)];
    }
    const unsigned cv = ld_agt(ctr);
    __builtin_amdgcn_s_barrier();
    seq = __builtin_amdgcn_readfirstlane(cv) + 1;
    v4u o[4];
#pragma unroll
    for (int g = 0; g < 4; ++g) {
      const unsigned t0 = __builtin_amdgcn_perm(d[4 * g + 1], d[4 * g + 0], 0x05010400u);
      const unsigned t1 = __builtin_amdgcn_perm(d[4 * g + 1], d[4 * g + 0], 0x07030602u);
      const unsigned t2 = __builtin_amdgcn_perm(d[4 * g + 3], d[4 * g + 2], 0x05010400u);
      const unsigned t3 = __builtin_amdgcn_perm(d[4 * g + 3], d[4 * g + 2], 0x07030602u);
      o[0][g] = __builtin_amdgcn_perm(t2, t0, 0x05040100u);
      o[1][g] = __builtin_amdgcn_perm(t2, t0, 0x07060302u);
      o[2][g] = __builtin_amdgcn_perm(t3, t1, 0x05040100u);
      o[3][g] = __builtin_amdgcn_perm(t3, t1, 0x07060302u);
    }
    const __amdgpu_buffer_rsrc_t rs = mk_rsrc(my + OFF_STG + (seq & 1) * SHARD, SHARD);
    const unsigned off = sslot * UNIT + lane * 16;
#pragma unroll
    for (int j = 0; j < 4; ++j) __builtin_amdgcn_raw_buffer_store_b128(o[j], rs, off + j * 1024, 0, AUX_SYS);
    asm volatile("s_waitcnt vmcnt(0)" ::: "memory");
    STAMP(1);
    unsigned old = 0;
    if (lane == 0) old = __hip_atomic_fetch_add(&lds[3], 1u, __ATOMIC_RELAXED, __HIP_MEMORY_SCOPE_WORKGROUP);
    old = __builtin_amdgcn_readfirstlane(old);
    if (old == 3 && lane >= 1 && lane < NR)
      st_sys((unsigned*)(peer_ws + OFF_SFLAG) + (rank * 128 + idx) * 32, seq);
    if (lane == 0) {
      st_agt((unsigned*)(my + OFF_WFLAG) + (snb * 8 + rank) * 32 + sks, seq);
      st_agt(ctr, seq);
    }
  } else {
    if (wid == 4 && lane < 4) st_lds(&lds[lane], 0);
    asm volatile("s_waitcnt lgkmcnt(0)" ::: "memory");
    __builtin_amdgcn_s_barrier();
    asm volatile("" ::: "memory");
    if (wid == 5 && b < 8) {
      unsigned long long pc;
      int la, lb;
      asm volatile("s_getpc_b64 %0\n.Lpf_here:\n\ts_mov_b32 %1, .Lpf_gemm_end-.Lpf_here\n\ts_mov_b32 %2, .Lpf_pull_end-.Lpf_here"
                   : "=s"(pc), "=s"(la), "=s"(lb));
      const unsigned len = (unsigned)max(max(la, lb), 0);
      if (len < 32768u) {
        const __amdgpu_buffer_rsrc_t rc = mk_rsrc((const void*)pc, len & ~15u);
        v4u acc = {0, 0, 0, 0};
#pragma unroll
        for (int i = 0; i < 4; ++i) acc |= __builtin_amdgcn_raw_buffer_load_b128(rc, (lane + i * 64) * 128, 0, 0);
        asm volatile("" :: "v"(acc));
      }
    }
    seq = __builtin_amdgcn_readfirstlane(ld_agt(ctr)) + 1;
    if (lane == 0) st_agt(ctr, seq);
    if (b == 16 && wid == 5 && lane < 8) st_agt((unsigned*)(my + OFF_QCTR) + (((seq & 1) ^ 1) * 8 + lane) * 32, 0);
  }
  seq = __builtin_amdgcn_readfirstlane(seq);
  const unsigned par = seq & 1;

  const unsigned part = TM == 64 ? ((b >> 3) & 3) == 0 ? 0 : ((b >> 3) & 3) == 2 ? 1 : 3 : (b >> 3) & 3;
  const bool gemm_wg = part < 3;
  const unsigned nb = (b >> 5) * 8 + (b & 7);
  const unsigned f0 = part * (TM == 64 ? 16 : 11);
  const unsigned f1 = min(f0 + (TM == 64 ? 16 : 11), 32u);
  const bool mover_wave = TM == 64 ? (role == 1 && wid == 5) : (part == 3 && wid >= 4);

  if (wid == 4 && gemm_wg) {
    {
      const unsigned psrc = (rank + ord_dist(lane >> 3, nb)) & 7;
      const unsigned* const wf = (const unsigned*)(my + OFF_WFLAG) + (nb * 8 + psrc) * 32 + (lane & 7);
      const unsigned xmb = min((f0 >> 2) + ((lane >> 1) & 3), (f1 - 1) >> 2);
      const unsigned* const xf = (const unsigned*)(my + OFF_XFLAG) + ((xmb * 8 + psrc) * 2 + (lane & 1)) * 32;
      bool wok = false, xok = false;
      unsigned long long last = 0;
      for (unsigned spin = 0;; ++spin) {
        const unsigned front = (unsigned)__builtin_ctzll(~last) >> 3;
        const bool near = (lane >> 3) < front + POLLW;
        if (!wok && near) wok = ld_agt(wf) >= seq;
        if (!xok && near) xok = ld_sys(xf) >= seq;
        asm volatile("" ::: "memory");
        const unsigned long long xm = __ballot(xok);
        const unsigned long long m = __ballot(wok && ((xm >> (lane & 56)) & 0xff) == 0xff);
        if (m != last) {
          last = m;
          if (lane < 2) st_lds(&lds[lane], lane ? (unsigned)(m >> 32) : (unsigned)m);
          if (m == ~0ull) break;
        }
        if ((spin & 63) == 63 &&
            (ld_agt(tmo) == seq || __builtin_amdgcn_s_memrealtime() - t_start > DEADLINE)) {
          st_agt(tmo, seq);
          if (lane == 0) st_lds(&lds[2], 1);
          asm volatile("s_waitcnt lgkmcnt(0)" ::: "memory");
          if (lane < 2) st_lds(&lds[lane], 0xffffffffu);
          break;
        }
        __builtin_amdgcn_s_sleep(1);
      }
    }
  } else if (wid < 4) {
    if (gemm_wg) {
      const unsigned wf0 = f0 + wid * MF;
      unsigned avo[MF];
#pragma unroll
      for (int i = 0; i < MF; ++i)
        avo[i] = wf0 + i < f1 ? ((wf0 + i) >> 2) * (8 * UNIT) + ((wf0 + i) & 3) * 1024 + lane * 16 : 0x80000000u;
      const float scale0 = p.sx[0] * p.sw[0];
      v4i acc[MF][4];
#pragma unroll
      for (int i = 0; i < MF; ++i)
#pragma unroll
        for (int j = 0; j < 4; ++j) acc[i][j] = v4i{0, 0, 0, 0};
      v4i ra[4][MF], rb[4][4];
#pragma unroll
      for (int s = 0; s < 4; ++s) {
#pragma unroll
        for (int i = 0; i < MF; ++i) ra[s][i] = v4i{0, 0, 0, 0};
#pragma unroll
        for (int j = 0; j < 4; ++j) rb[s][j] = v4i{0, 0, 0, 0};
      }

      unsigned tmo_val = 0;
      const __amdgpu_buffer_rsrc_t rws = mk_rsrc(my, OFF_FLG);
      unsigned srcs = 0;
#pragma unroll
      for (unsigned ord = 0; ord < 8; ++ord) srcs |= ((rank + ord_dist(ord, nb)) & 7) << (ord * 4);
      const unsigned bbase = slot_of(nb, 0) * UNIT;
      unsigned long long todo = 0, done = 0;
      for (unsigned it = 0; it < 16; ++it) {
#pragma unroll
        for (int s = 0; s < 4; ++s) {
          while (!todo) {
            const unsigned lo = __builtin_amdgcn_readfirstlane(ld_lds(&lds[0]));
            const unsigned hi = __builtin_amdgcn_readfirstlane(ld_lds(&lds[1]));
            todo = (((unsigned long long)hi << 32) | lo) & ~done;
            if (todo) break;
            __builtin_amdgcn_s_sleep(1);
          }
          asm volatile("" ::: "memory");
          const unsigned u = (unsigned)__builtin_ctzll(todo);
          todo &= todo - 1;
          done |= 1ull << u;
          const unsigned src = (srcs >> ((u >> 3) * 4)) & 7, ks = u & 7;
          const unsigned aoff = OFF_X + par * (NR * XBLK) + src * XBLK + ks * UNIT;
          const unsigned boff = (u < 8 ? OFF_STG + par * SHARD : OFF_WG + par * WGB + src * SHARD) + bbase + ks * (8 * UNIT);
          if (s == 3 && (it & 1)) STAMP(3 + (it >> 1));
#pragma unroll
          for (int j = 0; j < 4; ++j)
            rb[s][j] = (v4i)__builtin_amdgcn_raw_buffer_load_b128(rws, lane * 16 + j * 1024, boff, 0);
#pragma unroll
          for (int i = 0; i < MF; ++i)
            ra[s][i] = (v4i)__builtin_amdgcn_raw_buffer_load_b128(rws, avo[i], aoff, 0);
          if (s == 3) tmo_val = ld_agt(tmo);
          const int s3 = (s + 1) & 3;
#pragma unroll
          for (int i = 0; i < MF; ++i)
#pragma unroll
            for (int j = 0; j < 4; ++j)
              acc[i][j] = __builtin_amdgcn_mfma_i32_16x16x64_i8(ra[s3][i], rb[s3][j], acc[i][j], 0, 0, 0);
        }
      }
#pragma unroll
      for (int s = 1; s < 4; ++s)
#pragma unroll
        for (int i = 0; i < MF; ++i)
#pragma unroll
          for (int j = 0; j < 4; ++j)
            acc[i][j] = __builtin_amdgcn_mfma_i32_16x16x64_i8(ra[s][i], rb[s][j], acc[i][j], 0, 0, 0);

      STAMP(11);
      const bool failed = ld_lds(&lds[2]) != 0;
      const float scale = (failed || tmo_val == seq) ? __builtin_nanf("") : scale0;
      const __amdgpu_buffer_rsrc_t ro = mk_rsrc(p.out, (unsigned)MS * NN * 4);
#pragma unroll
      for (int i = 0; i < MF; ++i)
#pragma unroll
        for (int r = 0; r < 4; ++r) {
          v4f v;
#pragma unroll
          for (int j = 0; j < 4; ++j) v[j] = (float)acc[i][j][r] * scale;
          const unsigned row = (wf0 + i) * 16 + q * 4 + r;
          const unsigned off = wf0 + i < f1 ? (row * NN + nb * 64 + c * 4) * 4 : OOB;
          __builtin_amdgcn_raw_buffer_store_b128((v4u)v, ro, off, 0, AUX_AGT);
        }
      asm volatile(".Lpf_gemm_end:");
    }
  } else if (mover_wave) {
    const unsigned xcc = __builtin_amdgcn_s_getreg((3 << 11) | 20) & 7;
    const unsigned row = TM == 64 ? b >> 4 : (wid - 4) * 8 + (b >> 5);
    const unsigned CLQ = 0xB2u;
    const unsigned same = (((CLQ >> rank) & 1) ? CLQ : (~CLQ & 0xffu)) & ~(1u << rank);
    const unsigned other = ~same & ~(1u << rank) & 0xffu;
    unsigned cand = AFFINE == 1 ? ((xcc & 4) ? other : same) : ((xcc & 4) ? same : other);
    unsigned li = (row + xcc) & 3;
    if (AFFINE == 0) { cand = 0xffu & ~(1u << rank); li = (row + xcc) & 7; }
    unsigned src0 = 8;
    {
      unsigned m = cand;
      for (unsigned i = 0; i < li && m; ++i) m &= m - 1;
      if (m) src0 = __builtin_ctz(m);
    }
    if (row < PL && src0 < 8) {
      unsigned* const qh0 = (unsigned*)(my + OFF_QCTR) + par * (8 * 32);
      const __amdgpu_buffer_rsrc_t rdst = mk_rsrc(my + OFF_WG + par * WGB, WGB);
      const __amdgpu_buffer_rsrc_t rflag = mk_rsrc(my + OFF_FLG, OFF_XFLAG - OFF_FLG);
      unsigned vzero = 0, vone = 1;
      asm volatile("" : "+v"(vzero), "+v"(vone));
      bool failed = false;
      const unsigned lane16 = lane * 16;
      char* const qws = p.ws[src0];
      unsigned tk = 0;
      unsigned resp = 1u << src0;
      unsigned availv = 0;
      unsigned tried = 0;
      unsigned ksrc = 8, known = 0;
      bool planned = false;
      bool need_scan = false;
      unsigned p_e = 0, p_ext = 0, p_xs = 0, p_es = 0;
      const unsigned* const snap = (const unsigned*)(my + OFF_SNAP) + (lane & 7);

      auto timed_out = [&]() {
        if (ld_agt(tmo) == seq || __builtin_amdgcn_s_memrealtime() - t_start > DEADLINE) {
          st_agt(tmo, seq);
          failed = true;
        }
        return failed;
      };
      auto make_plan = [&](unsigned sv) {
        const unsigned s8 = lane & 7;
        if (__ballot(s8 == rank || sv >= seq * 1024) != ~0ull) return;
        const unsigned h = s8 == rank ? 0 : sv - seq * 1024;
        unsigned sum = h;
        sum += __shfl_xor((int)sum, 1); sum += __shfl_xor((int)sum, 2); sum += __shfl_xor((int)sum, 4);
        const unsigned a = sum / 7;
        const unsigned e = (s8 != rank && h + PLAN_DEAD < a) ? min(a - h - PLAN_DEAD, RW - 8u) : 0;
        const unsigned sp = (s8 != rank && s8 != ((rank + 1) & 7) && h > a) ? h - a : 0;
        need_scan = __ballot(s8 != rank && (h == 0 || h + PLAN_DEAD + RW - 8u < a)) != 0;
        unsigned pe = e, ps = sp;
#pragma unroll
        for (int o = 1; o < 8; o <<= 1) {
          const unsigned ue = __shfl_up((int)pe, o, 8), us = __shfl_up((int)ps, o, 8);
          if (s8 >= (unsigned)o) { pe += ue; ps += us; }
        }
        const unsigned tot = min((unsigned)__shfl((int)pe, 7, 8), (unsigned)__shfl((int)ps, 7, 8));
        p_es = pe - e;
        p_xs = ps - sp;
        p_e = tot > p_es ? min(tot - p_es, e) : 0;
        p_ext = tot > p_xs ? min(tot - p_xs, sp) : 0;
        planned = true;
      };
      auto take_snapshot = [&]() {
        unsigned old = seq;
        if (lane == 0) old = __hip_atomic_fetch_max((unsigned*)(my + OFF_SOWN), seq, __ATOMIC_RELAXED, __HIP_MEMORY_SCOPE_AGENT);
        if (__builtin_amdgcn_readfirstlane(old) < seq) {
          const unsigned hv = ld_agt(qh0 + (lane & 7) * 32);
          __builtin_amdgcn_raw_buffer_store_b32(seq * 1024 + min(hv, NT), rflag,
                                                lane < 8 ? (OFF_SNAP - OFF_FLG) + lane * 4 : OOB, 0, AUX_AGT);
        }
        asm volatile("s_waitcnt vmcnt(0)" ::: "memory");
      };
      auto map = [&](unsigned qs, unsigned tt, unsigned& us, unsigned& ut) -> bool {
        us = qs; ut = tt;
        if (tt + RW < NT) return true;
        for (unsigned spin = 0; !planned; ++spin) {
          make_plan(ld_agt(snap));
          if (planned) break;
          if (spin == 0) take_snapshot();
          if ((spin & 63) == 63 && timed_out()) return false;
          __builtin_amdgcn_s_sleep(2);
        }
        const unsigned ex = __builtin_amdgcn_readlane(p_ext, qs);
        if (tt + XAT >= NT && tt + XAT - NT < ex) {
          const unsigned g = __builtin_amdgcn_readlane(p_xs, qs) + (tt + XAT - NT);
          const unsigned m = (unsigned)__ballot(g >= p_es && g - p_es < p_e) & 0xffu;
          us = __builtin_ctz(m | 0x80u);
          ut = NT - RW + (g - __builtin_amdgcn_readlane(p_es, us));
          return true;
        }
        ut = tt + __builtin_amdgcn_readlane(p_e, qs) - (tt + XAT >= NT ? ex : 0);
        return ut < NT;
      };
      auto draw = [&](unsigned qs) {
        unsigned* const qh = qh0 + qs * 32;
        if (lane == 0)
          asm volatile("global_atomic_add %0, %1, %2, %3 sc0" : "=v"(tk) : "v"(vzero), "v"(vone), "s"(qh) : "memory");
      };
      auto dense_ptr = [&](unsigned s, unsigned back) {
        return (const unsigned*)(qws + OFF_DENSE) + s * 512 + ((512 - back + 64 * ((s - rank) & 7) + lane) & 511);
      };
      auto prep = [&](unsigned s, unsigned tt) {
        const unsigned hb = tt + 64 >= NT ? s : s + 8;
        if (s != src0 && tt + RW >= NT && !((tried >> hb) & 1)) {
          availv |= (ld_sys(dense_ptr(s, hb < 8 ? 64 : 128)) >= seq ? 1u : 0u) << hb;
          tried |= 1u << hb;
        }
        const bool rel = s != src0 && tt + RW >= NT && ((__builtin_amdgcn_readlane(availv, tt & 63) >> hb) & 1);
        if (!rel && (s != ksrc || tt >= known)) {
          const unsigned sd = (s - rank) & 7;
          const unsigned* const sf = (const unsigned*)(my + OFF_SFLAG) + s * (128 * 32);
          const unsigned g0 = ((lane + 16 * sd) & 127) * 32, g1 = ((lane + 64 + 16 * sd) & 127) * 32;
          ksrc = s;
          for (unsigned spin = 0;; ++spin) {
            const unsigned f0 = ld_sys(sf + g0), f1 = ld_sys(sf + g1);
            const unsigned long long m0 = __ballot(f0 >= seq), m1 = __ballot(f1 >= seq);
            known = 4 * (~m0 ? (unsigned)__builtin_ctzll(~m0) : 64 + (~m1 ? (unsigned)__builtin_ctzll(~m1) : 64));
            if (tt < known) break;
            if ((spin & 63) == 63 && timed_out()) break;
            __builtin_amdgcn_s_sleep(1);
          }
          asm volatile("" ::: "memory");
        }
        return mk_rsrc(rel ? qws + OFF_WG + par * WGB + s * SHARD : p.ws[s] + OFF_STG + par * SHARD, SHARD);
      };

      unsigned ts = src0;
      for (unsigned hop = 0; !failed; ++hop) {
        if (hop > 0) {
          if (!need_scan) break;
          const unsigned s8 = lane & 7;
          const unsigned hv = ld_agt(qh0 + s8 * 32);
          const unsigned lim = NT + p_ext - p_e;
          const bool open = s8 != rank && (hv + HELP_MIN < lim || (((resp >> s8) & 1) && hv < lim));
          unsigned key = (open ? 0xfffu - (lim - hv) : 0xffffu) * 8 + s8;
          key = min(key, (unsigned)__shfl_xor((int)key, 1));
          key = min(key, (unsigned)__shfl_xor((int)key, 2));
          key = min(key, (unsigned)__shfl_xor((int)key, 4));
          key = __builtin_amdgcn_readfirstlane(key);
          if ((key >> 3) >= 0xfffu) break;
          ts = key & 7;
          resp |= 1u << ts;
        }
        draw(ts);
        asm volatile("s_waitcnt vmcnt(0)" : "+v"(tk) :: "memory");
        unsigned t = __builtin_amdgcn_readfirstlane(tk);
        unsigned us, ut;
        if (!map(ts, t, us, ut)) continue;
        Move mv = decode(ut, us, rank);
        v4u cur[4];
        if (hop == 0 && ut < 64) {
          const unsigned* const f1 = (const unsigned*)(my + OFF_SFLAG) + (us * 128 + (mv.soff >> 14)) * 32;
          for (unsigned spin = 0; spin < 4096; ++spin)
            if (__builtin_amdgcn_readfirstlane(ld_sys(f1)) >= seq) { ksrc = us; known = ut + 1; break; }
          asm volatile("" ::: "memory");
        }
        {
          const __amdgpu_buffer_rsrc_t rsrc = prep(us, ut);
          if (failed) break;
          if (hop == 0) STAMP(2);
          draw(ts);
#pragma unroll
          for (int j = 0; j < 4; ++j)
            cur[j] = __builtin_amdgcn_raw_buffer_load_b128(rsrc, lane16 + j * 1024, mv.soff, AUX_SYS);
        }
        asm volatile("s_waitcnt vmcnt(4)" : "+v"(tk) :: "memory");
        __builtin_amdgcn_raw_buffer_store_b32(seq, rflag, OOB, 0, AUX_SYS);
        t = __builtin_amdgcn_readfirstlane(tk);
        while (t + SNAP_AT < NT) {
          const __amdgpu_buffer_rsrc_t rsrc = prep(ts, t);
          if (failed) break;
          while (t + SNAP_AT < NT && t < known) {
            const Move nx = decode(t, ts, rank);
#pragma unroll
            for (int j = 0; j < 4; ++j) __builtin_amdgcn_raw_buffer_store_b128(cur[j], rdst, mv.doff + lane16 + j * 1024, 0, AUX_SYS);
            draw(ts);
#pragma unroll
            for (int j = 0; j < 4; ++j)
              cur[j] = __builtin_amdgcn_raw_buffer_load_b128(rsrc, lane16 + j * 1024, nx.soff, AUX_SYS);
            asm volatile("s_waitcnt vmcnt(4)" : "+v"(tk) :: "memory");
            __builtin_amdgcn_raw_buffer_store_b32(seq, rflag, lane == 0 ? mv.foff : (lane == 1 ? mv.goff : OOB), 0, AUX_SYS);
            t = __builtin_amdgcn_readfirstlane(tk);
            mv = nx;
          }
        }
        if (failed) break;
        while (map(ts, t, us, ut)) {
          const Move nx = decode(ut, us, rank);
          const __amdgpu_buffer_rsrc_t rsrc = prep(us, ut);
          if (failed) break;
          if (t + SNAP_AT == NT) take_snapshot();
#pragma unroll
          for (int j = 0; j < 4; ++j) __builtin_amdgcn_raw_buffer_store_b128(cur[j], rdst, mv.doff + lane16 + j * 1024, 0, AUX_SYS);
          draw(ts);
          unsigned sv = 0, vv = 0, wantv = 16;
          const bool want_plan = !planned && t + SNAP_AT - 16 >= NT;
          if (want_plan) sv = ld_agt(snap);
          if (planned && ts == src0 && t + XAT + 2 * PL >= NT && __builtin_amdgcn_readlane(p_ext, src0) != 0) {
            const unsigned g = __builtin_amdgcn_readlane(p_xs, src0);
            const unsigned m = (unsigned)__ballot(g >= p_es && g - p_es < p_e) & 0xffu;
            if (m) {
              const unsigned s1 = __builtin_ctz(m);
              const unsigned t1 = NT - RW + (g - __builtin_amdgcn_readlane(p_es, s1));
              const unsigned hb = t1 + 64 >= NT ? s1 : s1 + 8;
              if (!((tried >> hb) & 1)) { wantv = hb; vv = ld_sys(dense_ptr(s1, hb < 8 ? 64 : 128)); }
            }
          }
#pragma unroll
          for (int j = 0; j < 4; ++j)
            cur[j] = __builtin_amdgcn_raw_buffer_load_b128(rsrc, lane16 + j * 1024, nx.soff, AUX_SYS);
          asm volatile("s_waitcnt vmcnt(4)" : "+v"(tk) :: "memory");
          __builtin_amdgcn_raw_buffer_store_b32(seq, rflag, lane == 0 ? mv.foff : (lane == 1 ? mv.goff : OOB), 0, AUX_SYS);
          t = __builtin_amdgcn_readfirstlane(tk);
          mv = nx;
          if (want_plan) make_plan(sv);
          if (wantv < 16) {
            availv |= (vv >= seq ? 1u : 0u) << wantv;
            tried |= 1u << wantv;
          }
        }
        if (failed) break;
#pragma unroll
        for (int j = 0; j < 4; ++j) __builtin_amdgcn_raw_buffer_store_b128(cur[j], rdst, mv.doff + lane16 + j * 1024, 0, AUX_SYS);
        asm volatile("s_waitcnt vmcnt(0)" ::: "memory");
        __builtin_amdgcn_raw_buffer_store_b32(seq, rflag, lane == 0 ? mv.foff : (lane == 1 ? mv.goff : OOB), 0, AUX_SYS);
        asm volatile(".Lpf_pull_end:");
        if (hop == 0) STAMP(4);
        STAMP(3);
      }
    }
  }
}

}

extern "C" void kernel_launch(int rank, int nranks, ncclComm_t comm,
                              hipStream_t stream,
                              void* const* d_in, const int* in_sizes, int n_in,
                              void* d_out, int out_size,
                              const int* in_dtypes, int out_dtype,
                              void* const* d_out_all,
                              void* const* d_ws_all, size_t ws_bytes) {
  (void)comm; (void)in_dtypes; (void)out_dtype; (void)d_out_all;
  if (nranks != NR || (unsigned)rank >= (unsigned)NR || n_in != 4 || ws_bytes < WS_USED ||
      in_sizes[0] != NR * MS * KS || in_sizes[1] != KS * NN || out_size != MS * NN) {
    fprintf(stderr, "unexpected problem geometry\n");
    return;
  }
  Params p;
  p.x = (const int8_t*)d_in[0];
  p.w = (const int8_t*)d_in[1];
  p.sx = (const float*)d_in[2];
  p.sw = (const float*)d_in[3];
  p.out = (float*)d_out;
  for (int i = 0; i < NR; ++i) p.ws[i] = (char*)d_ws_all[i];
  p.rank = rank;
  p.my = p.ws[rank];
  p.pad = 0;
  p.pad2 = nullptr;
  fused_kernel<<<GRID, BLOCK, 0, stream>>>(p);
  HIP_CHECK(hipGetLastError());
}
```
